```python
import numpy as np

kernel_name = "gpu_gemm_rs_m4096_k4096_n16384_f32_none"


ABSMAX_RTOL = 0.001

COMPUTE_DTYPE = "float32"
EPILOGUE = "none"
IN_QUANT = None
OUT_QUANT = None
IN_DTYPES = {"x": "float32", "w_mat": "float32"}
WORLD_SIZE = 8

def setup_inputs():

    x = np.random.randn(4096, 4096).astype(np.float32)
    w_mat = (np.random.randn(4096, 16384) / np.sqrt(4096)).astype(np.float32)
    return {"x": x, "w_mat": w_mat}

def _epilogue(y):
    return y

def reference(x, w_mat):
    y = (x @ w_mat).astype(np.float32)
    return _epilogue(y)
```

```cpp
#include <hip/hip_runtime.h>
#include <rccl/rccl.h>
#include <cstdio>
#include <cstdint>

#define HIP_CHECK(expr) do { hipError_t _e = (expr); if (_e != hipSuccess) { \
    fprintf(stderr, "HIP error %s at %s:%d\n", hipGetErrorString(_e), __FILE__, __LINE__); \
    return; } } while (0)

namespace {

#define LAS __attribute__((address_space(3)))
typedef _Float16 h8 __attribute__((ext_vector_type(8)));
typedef float f32x4 __attribute__((ext_vector_type(4)));
typedef unsigned u32x4 __attribute__((ext_vector_type(4)));
typedef __amdgpu_buffer_rsrc_t rsrc_t;
typedef unsigned long long u64;

constexpr int NR = 8;
constexpr int KSH = 512;
constexpr int NG = 16384;
constexpr int BLOB = 32768;
#define CFG_NW 6
#define CFG_PW 4
#define CFG_PU 2
#define CFG_D2 4
#define CFG_D3 4
constexpr int NW = CFG_NW;
constexpr int PW = CFG_PW;
constexpr int PU = CFG_PU;
constexpr int SLICE = BLOB / PW;
constexpr int SPI = SLICE / (PU * 1024);
constexpr int GRID = 256, NGEMM = 128, NPULL = (NR - 1) * NW, NCONV = GRID - NPULL, NHELP = GRID - NGEMM - NPULL;
constexpr int N_IN = 14 * 8 + 8 * 8;
constexpr int N_OUT = 128;
constexpr int N_ITEMS = N_IN + N_OUT;
static_assert((N_ITEMS + NW - 1) / NW <= 64, "one lane per item");
static_assert(4 % PU == 0 && SPI * PU * 1024 * PW == BLOB, "puller geometry");
static_assert(NHELP > 0, "need helper workgroups");

constexpr int ID_XR = 0;
constexpr int ID_WR = 1024;
constexpr int ID_P = 1536;
constexpr int ID_PAR = 2560;
constexpr int PO_X = 0, PO_W = 128, PO_C = 640, PAR_SZ = 1664;
constexpr int N_IDS = ID_PAR + 2 * PAR_SZ;
constexpr unsigned OFF_CTR = 0;
constexpr unsigned OFF_TMO = 32768;
constexpr unsigned OFF_RF = 65536;
constexpr unsigned OFF_LC = 131072;
constexpr unsigned OFF_DBG = 262144;
constexpr unsigned OFF_DATA = 1u << 20;
constexpr size_t WS_NEED = (size_t)OFF_DATA + (size_t)N_IDS * BLOB;
constexpr unsigned OUT_BYTES = 512u * NG * 4u;
constexpr u64 DEADLINE = 50000000ull;
constexpr int AUX_SC1 = 16, AUX_SYS = 17;
constexpr int LDS_BYTES = 131072;

struct Params {
    const float* x; const float* w; float* out;
    char* ws[NR];
    int rank;
    unsigned short order[N_IN];
};

#define MEMFENCE() asm volatile("" ::: "memory")
#define WAITV_IMM(n) (((n) & 15) | (7 << 4) | (15 << 8) | (((n) >> 4) << 14))
#define WAITV(n) do { MEMFENCE(); __builtin_amdgcn_s_waitcnt(WAITV_IMM(n)); MEMFENCE(); } while (0)

#define STAMP(slot) do {} while (0)

__device__ __forceinline__ unsigned ld_agent(const void* p) { return __hip_atomic_load((const unsigned*)p, __ATOMIC_RELAXED, __HIP_MEMORY_SCOPE_AGENT); }
__device__ __forceinline__ unsigned ld_sys(const void* p) { return __hip_atomic_load((const unsigned*)p, __ATOMIC_RELAXED, __HIP_MEMORY_SCOPE_SYSTEM); }
__device__ __forceinline__ void st_agent(void* p, unsigned v) { __hip_atomic_store((unsigned*)p, v, __ATOMIC_RELAXED, __HIP_MEMORY_SCOPE_AGENT); }
__device__ __forceinline__ void st_sys(void* p, unsigned v) { __hip_atomic_store((unsigned*)p, v, __ATOMIC_RELAXED, __HIP_MEMORY_SCOPE_SYSTEM); }
__device__ __forceinline__ rsrc_t mk_rsrc(const void* p, unsigned bytes) { return __builtin_amdgcn_make_buffer_rsrc((void*)p, 0, bytes, 0x00020000); }
__device__ __forceinline__ unsigned uni(unsigned v) { return (unsigned)__builtin_amdgcn_readfirstlane((int)v); }

__device__ __forceinline__ bool give_up(char* wsl, unsigned call, u64 t0) {
    if (uni(ld_agent(wsl + OFF_TMO)) == call || __builtin_amdgcn_s_memrealtime() - t0 > DEADLINE) {
        st_agent(wsl + OFF_TMO, call);
        return true;
    }
    return false;
}

__device__ __forceinline__ void conv_load(const Params& p, int n, int wid, int lane, f32x4 (&v)[8]) {
    const int tt = n / 80, r = n % 80, g = lane >> 4, c = lane & 15;
    if (r < 16) {
        const float* src = p.x + (size_t)(r * 256 + wid * 32 + c) * KSH + tt * 64 + 4 * g;
#pragma unroll
        for (int f = 0; f < 2; ++f)
#pragma unroll
            for (int ii = 0; ii < 4; ++ii) v[f * 4 + ii] = __builtin_nontemporal_load((const f32x4*)(src + f * 16 * KSH + ii * 16));
    } else {
        const int ks = wid & 1, wc = wid >> 1;
        const float* src = p.w + (size_t)(tt * 64 + ks * 32 + 4 * g) * NG + (r - 16) * 256 + wc * 64 + 4 * c;
#pragma unroll
        for (int e = 0; e < 8; ++e) v[e] = __builtin_nontemporal_load((const f32x4*)(src + (size_t)((e >> 2) * 16 + (e & 3)) * NG));
    }
}
__device__ __forceinline__ void conv_store(rsrc_t rsl, int n, int pb, int wid, int lane, const f32x4 (&v)[8]) {
    const int tt = n / 80, r = n % 80;
    if (r < 16) {
        const unsigned base = OFF_DATA + (unsigned)(pb + PO_X + r * 8 + tt) * BLOB + wid * 4096;
#pragma unroll
        for (int f = 0; f < 2; ++f)
#pragma unroll
            for (int ks = 0; ks < 2; ++ks) {
                h8 o;
#pragma unroll
                for (int e = 0; e < 8; ++e) o[e] = (_Float16)v[f * 4 + ks * 2 + (e >> 2)][e & 3];
                __builtin_amdgcn_raw_buffer_store_b128(__builtin_bit_cast(u32x4, o), rsl, lane * 16, base + f * 2048 + ks * 1024, AUX_SYS);
            }
    } else {
        const int ks = wid & 1, wc = wid >> 1;
        const unsigned base = OFF_DATA + (unsigned)(pb + PO_W + (r - 16) * 8 + tt) * BLOB + wc * 4096 + ks * 1024;
#pragma unroll
        for (int f = 0; f < 4; ++f) {
            h8 o;
#pragma unroll
            for (int e = 0; e < 8; ++e) o[e] = (_Float16)v[e][f];
            __builtin_amdgcn_raw_buffer_store_b128(__builtin_bit_cast(u32x4, o), rsl, lane * 16, base + (f >> 1) * 16384 + (f & 1) * 2048, AUX_SYS);
        }
    }
}
__device__ __forceinline__ void conv_publish(int rank, char* wst  , int n, int pb, unsigned call) {
    const int tt = n / 80, r = n % 80, t = threadIdx.x;
    if (t < NR) {
        if (t == rank) st_agent(wst + OFF_LC + 4 * (pb + (r < 16 ? PO_X + r * 8 : PO_W + (r - 16) * 8) + tt), PW * call);
        else if (r < 16) { if ((r >> 1) != rank) st_sys(wst + OFF_RF + 4 * (ID_XR + r * 64 + rank * 8 + tt), call); }
        else if (((r - 16) >> 3) == t) st_sys(wst + OFF_RF + 4 * (ID_WR + ((r - 16) & 7) * 64 + rank * 8 + tt), call);
    }
}

struct PStep { unsigned src, dst, did; bool valid, last; };
constexpr int VOFF_OOB = 0x7ffffff0;

__device__ __forceinline__ void pull_items(char* wsl, rsrc_t rsl, rsrc_t rsp, rsrc_t rso, unsigned desc  , u64 todo,
                                           unsigned call, int wid, int lane, int peer) {
    const unsigned did_l = desc & 0xffffu;
    const int lane16 = lane * 16, lane_out = (lane >> 4) * 4 * (NG * 4) + (lane & 15) * 16;
    const u64 t0 = __builtin_amdgcn_s_memrealtime();
    u64 ready = 0;
    bool dead = false;
    int sub = SPI;
    unsigned cur = 0, pend = 0xffffffffu, spins = 0;
    auto step = [&](const PStep& cu, const u32x4 (&rc)[PU], const u32x4 (&qc)[PU], PStep& nx, u32x4 (&rn)[PU], u32x4 (&qn)[PU]) -> bool {
        MEMFENCE();
        if (sub == SPI && (ready & todo)) {
            const int it = __builtin_ctzll(ready & todo);
            todo &= ~(1ull << it);
            cur = (unsigned)__builtin_amdgcn_readlane((int)desc, it);
            sub = 0;
        }
        nx.valid = sub < SPI;
        const unsigned inb = wid * SLICE + (nx.valid ? sub : 0) * (PU * 1024);
        nx.did = cur & 0xffffu;
        nx.src = OFF_DATA + (cur >> 16) * BLOB + inb;
        nx.dst = OFF_DATA + nx.did * BLOB + inb;
        if (nx.valid) ++sub;
        nx.last = sub == SPI;
        const u64 av = ready & todo;
        const bool need_poll = (todo & ~ready) != 0 && (av & (av - 1)) == 0 && (sub == 1 || !nx.valid);
        unsigned f1 = 0, f2 = 0;
        if (need_poll) { f1 = ld_sys(wsl + OFF_RF + 4 * did_l); f2 = ld_agent(wsl + OFF_LC + 4 * did_l); }
        MEMFENCE();
        const int vr = nx.valid ? lane16 : VOFF_OOB, vq = (nx.valid && nx.did >= (unsigned)ID_P) ? lane16 : VOFF_OOB;
#pragma unroll
        for (int u = 0; u < PU; ++u) rn[u] = __builtin_amdgcn_raw_buffer_load_b128(rsp, vr, nx.src + u * 1024, AUX_SYS);
#pragma unroll
        for (int u = 0; u < PU; ++u) qn[u] = __builtin_amdgcn_raw_buffer_load_b128(rsl, vq, nx.dst + u * 1024, AUX_SC1);
        WAITV(2 * PU);
        ready |= __ballot(f1 >= call && (did_l < (unsigned)ID_P || f2 >= PW * call));
        if (pend != 0xffffffffu) {
            if (lane == 0) __hip_atomic_fetch_add((unsigned*)(wsl + OFF_LC + 4 * pend), 1u, __ATOMIC_RELAXED, __HIP_MEMORY_SCOPE_AGENT);
            pend = 0xffffffffu;
        }
        if (cu.valid) {
            if (cu.did >= (unsigned)ID_P) {
                const unsigned cb = (cu.did - ID_P) & 127u, e = ((cu.dst - OFF_DATA) & (BLOB - 1)) >> 10;
                const unsigned row = (cb >> 6) * 256 + (e >> 4) * 128 + ((cb >> 2) & 1) * 64 + ((e >> 2) & 3) * 16 + (e & 3);
                const unsigned col = peer * 2048 + ((cb >> 3) & 7) * 256 + (cb & 3) * 64;
                const unsigned so = row * (NG * 4) + col * 4;
#pragma unroll
                for (int u = 0; u < PU; ++u) {
                    f32x4 v = __builtin_bit_cast(f32x4, rc[u]) + __builtin_bit_cast(f32x4, qc[u]);
                    if (dead) v = (f32x4){__builtin_nanf(""), __builtin_nanf(""), __builtin_nanf(""), __builtin_nanf("")};
                    __builtin_amdgcn_raw_buffer_store_b128(__builtin_bit_cast(u32x4, v), rso, lane_out, so + u * (NG * 4), AUX_SC1);
                }
            } else {
#pragma unroll
                for (int u = 0; u < PU; ++u) __builtin_amdgcn_raw_buffer_store_b128(rc[u], rsl, lane16, cu.dst + u * 1024, AUX_SC1);
                if (cu.last) pend = cu.did;
            }
        } else if (!nx.valid && todo && !(ready & todo)) {
            if ((++spins & 63u) == 0 && give_up(wsl, call, t0)) { dead = true; ready = ~0ull; }
            __builtin_amdgcn_s_sleep(4);
        }
        return !cu.valid && !nx.valid && !todo;
    };
    PStep sa = {0, 0, 0, false, false}, sb;
    u32x4 ra[PU] = {}, qa[PU] = {}, rb[PU], qb[PU];
    bool done;
    do {
        step(sa, ra, qa, sb, rb, qb);
        done = step(sb, rb, qb, sa, ra, qa);
    } while (!done);
}

__global__ __launch_bounds__(512, 2) void gemm_rs_fused(const Params p) {
    extern __shared__ __attribute__((aligned(16))) char shm[];
    LAS char* lds = (LAS char*)shm;
    const int b = blockIdx.x, tid = threadIdx.x, lane = tid & 63, wid = __builtin_amdgcn_readfirstlane(tid >> 6);
    const int rank = p.rank;
    char* wsl = p.ws[rank];
    const rsrc_t rsl = mk_rsrc(wsl, (unsigned)WS_NEED);
    const rsrc_t rso = mk_rsrc(p.out, OUT_BYTES);
    STAMP(0);

    if (b >= NGEMM && b < NGEMM + NPULL) {
        const unsigned call = uni(ld_agent(wsl + OFF_CTR + b * 128)) + 1u;
        asm volatile("s_waitcnt vmcnt(0)" ::: "memory");
        __builtin_amdgcn_s_barrier();
        MEMFENCE();
        if (tid == 0) st_agent(wsl + OFF_CTR + b * 128, call);
        if (wid >= PW) return;
        const int pi = b - NGEMM, ds = 1 + pi % (NR - 1), k = pi / (NR - 1), peer = (rank + ds) & (NR - 1);
        const int pb = ID_PAR + (int)(call & 1u) * PAR_SZ;
        const rsrc_t rsp = mk_rsrc(p.ws[peer], (unsigned)WS_NEED);
        const int n = k + lane * NW;
        unsigned did, sid;
        if (n < N_IN) {
            const unsigned code = p.order[n];
            const unsigned tt = code & 7u, a = (code >> 3) & 31u;
            if (code >> 8) { did = ID_WR + a * 64 + peer * 8 + tt; sid = pb + PO_W + rank * 64 + a * 8 + tt; }
            else {
                unsigned d = a >> 1;
                if (d >= (unsigned)ds) ++d;
                const unsigned i = (((rank + d) & (NR - 1)) << 1) | (a & 1u);
                did = ID_XR + i * 64 + peer * 8 + tt; sid = pb + PO_X + i * 8 + tt;
            }
        } else {
            const unsigned cb = n < N_ITEMS ? n - N_IN : 0;
            did = ID_P + peer * 128 + cb; sid = pb + PO_C + rank * 128 + cb;
        }
        pull_items(wsl, rsl, rsp, rso, (sid << 16) | did, __ballot(n < N_ITEMS), call, wid, lane, peer);
        STAMP(2);
        return;
    }

    unsigned call;
    int pb;
    {
        const int ci = b < NGEMM ? b : b - NPULL;
        static_assert(2 * NCONV <= 640 && 3 * NCONV >= 640, "three blobs per converter at most, two at least");
        const int n2 = ci + 2 * NCONV < 640 ? ci + 2 * NCONV : ci;
        f32x4 v0[8], v1[8], v2[8];
        const unsigned c0 = ld_agent(wsl + OFF_CTR + b * 128);
        char* wst = p.ws[0];
#pragma unroll
        for (int r = 1; r < NR; ++r) wst = tid == r ? p.ws[r] : wst;
        MEMFENCE();
        conv_load(p, ci, wid, lane, v0);
        call = uni(c0) + 1u;
        pb = ID_PAR + (int)(call & 1u) * PAR_SZ;
        conv_store(rsl, ci, pb, wid, lane, v0);
        MEMFENCE();
        conv_load(p, ci + NCONV, wid, lane, v1);
        conv_load(p, n2, wid, lane, v2);
        WAITV(16);
        __builtin_amdgcn_s_barrier();
        MEMFENCE();
        conv_publish(rank, wst, ci, pb, call);
        if (tid == 0) st_agent(wsl + OFF_CTR + b * 128, call);
        conv_store(rsl, ci + NCONV, pb, wid, lane, v1);
        if (n2 != ci) conv_store(rsl, n2, pb, wid, lane, v2);
        WAITV(0);
        __builtin_amdgcn_s_barrier();
        MEMFENCE();
        conv_publish(rank, wst, ci + NCONV, pb, call);
        if (n2 != ci) conv_publish(rank, wst, n2, pb, call);
    }
    STAMP(1);

    const unsigned tgt = PW * call;
    const int wr = wid >> 2, wc = wid & 3;
    for (int job = 0;; ++job) {
        int nsteps, cid, fl_rank;
        unsigned ids, fl_idx, out_off = 0;
        if (b < NGEMM) {
            if (job) break;
            const int xcd = b & 7, kk = b >> 3, i = (xcd >> 1) * 4 + (kk & 3), j = (xcd & 1) * 4 + (kk >> 2), own = i >> 1;
            const int nrem = own == rank ? 7 : 6;
            nsteps = 8 + 8 * nrem;
            const int m = (lane < nsteps ? lane : 0) - 8;
            if (m < 0) ids = (unsigned)(pb + PO_X + i * 8 + m + 8) | ((unsigned)(pb + PO_W + rank * 64 + j * 8 + m + 8) << 16);
            else {
                const int tt = m / nrem; int d = m % nrem + 1;
                if (own != rank && d >= ((own - rank) & (NR - 1))) ++d;
                const int s = (rank + d) & (NR - 1);
                ids = (unsigned)(ID_XR + i * 64 + s * 8 + tt) | ((unsigned)(ID_WR + j * 64 + s * 8 + tt) << 16);
            }
            const int cb = (i & 1) * 64 + j * 8;
            cid = own == rank ? -1 : pb + PO_C + own * 128 + cb;
            fl_rank = own; fl_idx = ID_P + rank * 128 + cb;
            out_off = (unsigned)((i & 1) * 256) * (NG * 4) + (unsigned)(rank * 2048 + j * 256) * 4;
        } else {
            const int pt = (b - NGEMM - NPULL) + job * NHELP;
            if (pt >= 112) break;
            const int h = pt / 56, rem = pt % 56, q = (rank + 1 + (rem >> 3)) & (NR - 1), j = rem & 7, m = lane & 7;
            nsteps = 8;
            ids = (unsigned)(pb + PO_X + (rank * 2 + h) * 8 + m) | ((unsigned)(pb + PO_W + q * 64 + j * 8 + m) << 16);
            cid = ID_P + q * 128 + h * 64 + j * 8;
            fl_rank = rank; fl_idx = cid;
        }

        u64 mask = 0;
        auto ensure = [&](int n) {
            if (!((mask >> n) & 1ull)) {
                const u64 t0 = __builtin_amdgcn_s_memrealtime();
                for (unsigned spins = 1;; ++spins) {
                    const unsigned fa = ld_agent(wsl + OFF_LC + 4 * (ids & 0xffffu)), fb = ld_agent(wsl + OFF_LC + 4 * (ids >> 16));
                    mask = __ballot(fa >= tgt && fb >= tgt);
                    if ((mask >> n) & 1ull) break;
                    if ((spins & 63u) == 0 && give_up(wsl, call, t0)) { mask = ~0ull; break; }
                    __builtin_amdgcn_s_sleep(8);
                }
                __builtin_amdgcn_fence(__ATOMIC_ACQUIRE, "agent");
            }
        };
        const int lane16 = lane * 16;
        const unsigned wofs = OFF_DATA + wid * 2048;
#define SA(bf, h) (((bf) * 2 + (h)) * 16384)
#define SB(bf, h) ((4 + (bf) * 2 + (h)) * 16384)
#define STAGE(bufoff, goff) do { \
        __builtin_amdgcn_raw_ptr_buffer_load_lds(rsl, (LAS void*)(lds + (bufoff) + wid * 2048), 16, lane16, (goff), 0, 0); \
        __builtin_amdgcn_raw_ptr_buffer_load_lds(rsl, (LAS void*)(lds + (bufoff) + wid * 2048), 16, lane16, (goff), 1024, 0); } while (0)
#define LDA(dst, bf, h) do { _Pragma("unroll") for (int m = 0; m < 4; ++m) _Pragma("unroll") for (int k = 0; k < 2; ++k) \
        dst[m][k] = *(const LAS h8*)(lds + SA(bf, h) + aoff + m * 2048 + k * 1024); } while (0)
#define LDB(dst, bf, h) do { _Pragma("unroll") for (int n = 0; n < 2; ++n) _Pragma("unroll") for (int k = 0; k < 2; ++k) \
        dst[n][k] = *(const LAS h8*)(lds + SB(bf, h) + boff + n * 2048 + k * 1024); } while (0)
#define MMA(ai, bj, At, Bt) do { __builtin_amdgcn_s_setprio(1); \
        _Pragma("unroll") for (int m = 0; m < 4; ++m) _Pragma("unroll") for (int n = 0; n < 2; ++n) _Pragma("unroll") for (int k = 0; k < 2; ++k) \
            acc[ai][bj][m][n] = __builtin_amdgcn_mfma_f32_16x16x32_f16(At[m][k], Bt[n][k], acc[ai][bj][m][n], 0, 0, 0); \
        __builtin_amdgcn_s_setprio(0); } while (0)
#define WAIT_V(n) asm volatile("s_waitcnt vmcnt(" #n ")" ::: "memory")
#define WAIT_L(n) asm volatile("s_waitcnt lgkmcnt(" #n ")" ::: "memory")
#define BAR __builtin_amdgcn_s_barrier()
#define SCHED __builtin_amdgcn_sched_barrier(0)
#define OFFA(c) (wofs + ((c) & 0xffffu) * BLOB)
#define OFFB(c) (wofs + ((c) >> 16) * BLOB)
        const int aoff = wr * 8192 + lane16, boff = wc * 4096 + lane16;
        f32x4 acc[2][2][4][2];
#pragma unroll
        for (int a = 0; a < 2; ++a)
#pragma unroll
            for (int c = 0; c < 2; ++c)
#pragma unroll
                for (int m = 0; m < 4; ++m)
#pragma unroll
                    for (int n = 0; n < 2; ++n) acc[a][c][m][n] = (f32x4){0.f, 0.f, 0.f, 0.f};
        h8 At[4][2], B0[2][2], B1[2][2];
        ensure(0); ensure(1);
        unsigned a1;
        {
            const unsigned c0 = (unsigned)__builtin_amdgcn_readlane((int)ids, 0), c1 = (unsigned)__builtin_amdgcn_readlane((int)ids, 1);
            STAGE(SB(0, 0), OFFB(c0)); STAGE(SA(0, 0), OFFA(c0)); STAGE(SB(0, 1), OFFB(c0) + 16384); STAGE(SA(0, 1), OFFA(c0) + 16384);
            if (wr == 1) BAR;
            WAIT_V(4); BAR;
            STAGE(SB(1, 0), OFFB(c1)); STAGE(SA(1, 0), OFFA(c1)); STAGE(SB(1, 1), OFFB(c1) + 16384);
            WAIT_V(6); BAR;
            a1 = OFFA(c1);
        }
        STAMP(2);
        for (int t = 0; t < nsteps; t += 2) {
            const int t2 = t + 2 < nsteps ? t + 2 : 0;
            const unsigned c2 = (unsigned)__builtin_amdgcn_readlane((int)ids, t2), c3 = (unsigned)__builtin_amdgcn_readlane((int)ids, t2 + 1);
            const unsigned a2 = OFFA(c2), b2 = OFFB(c2), a3 = OFFA(c3), b3 = OFFB(c3);
            LDB(B0, 0, 0); SCHED; LDA(At, 0, 0); STAGE(SA(1, 1), a1 + 16384);
            WAIT_L(8); BAR; WAIT_L(0); MMA(0, 0, At, B0); BAR; SCHED;
            ensure(t2);
            LDB(B1, 0, 1); STAGE(SB(0, 0), b2);
            BAR; WAIT_L(0); MMA(0, 1, At, B1); BAR;
            LDA(At, 0, 1); STAGE(SA(0, 0), a2);
            BAR; WAIT_L(0); MMA(1, 0, At, B0); BAR; SCHED;
            STAGE(SB(0, 1), b2 + 16384);
            WAIT_V(6); BAR; MMA(1, 1, At, B1); BAR;
            LDB(B0, 1, 0); SCHED; LDA(At, 1, 0); STAGE(SA(0, 1), a2 + 16384);
            WAIT_L(8); BAR; WAIT_L(0); MMA(0, 0, At, B0); BAR; SCHED;
            ensure(t2 + 1);
            LDB(B1, 1, 1); STAGE(SB(1, 0), b3);
            BAR; WAIT_L(0); MMA(0, 1, At, B1); BAR;
            LDA(At, 1, 1); STAGE(SA(1, 0), a3);
            BAR; WAIT_L(0); MMA(1, 0, At, B0); BAR; SCHED;
            STAGE(SB(1, 1), b3 + 16384);
            WAIT_V(6); BAR; MMA(1, 1, At, B1); BAR;
            a1 = a3;
        }
        STAMP(3);
        const bool dead = uni(ld_agent(wsl + OFF_TMO)) == call;
        const float qn = __builtin_nanf("");
        if (cid >= 0) {
            const unsigned base = OFF_DATA + (unsigned)(cid + wid) * BLOB;
#pragma unroll
            for (int ai = 0; ai < 2; ++ai)
#pragma unroll
                for (int m = 0; m < 4; ++m)
#pragma unroll
                    for (int r = 0; r < 4; ++r) {
                        f32x4 v = {acc[ai][0][m][0][r], acc[ai][0][m][1][r], acc[ai][1][m][0][r], acc[ai][1][m][1][r]};
                        if (dead) v = (f32x4){qn, qn, qn, qn};
                        __builtin_amdgcn_raw_buffer_store_b128(__builtin_bit_cast(u32x4, v), rsl, lane16, base + (ai * 16 + m * 4 + r) * 1024, AUX_SYS);
                    }
            WAIT_V(0);
            if (lane == 0) {
                if (fl_rank == rank) st_agent(wsl + OFF_LC + 4 * (fl_idx + wid), tgt);
                else if ((unsigned)fl_rank < (unsigned)NR) st_sys(p.ws[fl_rank] + OFF_RF + 4 * (fl_idx + wid), call);
            }
        } else {
            const int lane_out = (lane >> 4) * 4 * (NG * 4) + (lane & 15) * 16;
            const unsigned base = out_off + (unsigned)(wr * 64) * (NG * 4) + wc * 256;
#pragma unroll
            for (int ai = 0; ai < 2; ++ai)
#pragma unroll
                for (int m = 0; m < 4; ++m)
#pragma unroll
                    for (int r = 0; r < 4; ++r) {
                        f32x4 v = {acc[ai][0][m][0][r], acc[ai][0][m][1][r], acc[ai][1][m][0][r], acc[ai][1][m][1][r]};
                        if (dead) v = (f32x4){qn, qn, qn, qn};
                        __builtin_amdgcn_raw_buffer_store_b128(__builtin_bit_cast(u32x4, v), rso, lane_out, base + (unsigned)(ai * 128 + m * 16 + r) * (NG * 4), AUX_SC1);
                    }
            WAIT_V(0);
        }
        STAMP(4);
        if (wr == 0) BAR;
        BAR;
    }
}

}

extern "C" void kernel_launch(int rank, int nranks, ncclComm_t comm,
                              hipStream_t stream,
                              void* const* d_in, const int* in_sizes, int n_in,
                              void* d_out, int out_size,
                              const int* in_dtypes, int out_dtype,
                              void* const* d_out_all,
                              void* const* d_ws_all, size_t ws_bytes) {
    (void)comm; (void)in_sizes; (void)n_in; (void)in_dtypes; (void)out_dtype; (void)d_out_all;
    if (nranks != NR || (unsigned)rank >= (unsigned)NR || ws_bytes < WS_NEED || (size_t)out_size * 4 != OUT_BYTES) {
        fprintf(stderr, "kernel_launch: unexpected configuration\n");
        return;
    }
    Params p;
    p.x = (const float*)d_in[0];
    p.w = (const float*)d_in[1];
    p.out = (float*)d_out;
    for (int r = 0; r < NR; ++r) p.ws[r] = (char*)d_ws_all[r];
    p.rank = rank;
    int n = 0;
    for (int tt = 0; tt < 8; ++tt) {
        for (int j = 0; j < 8; ++j) p.order[n++] = (unsigned short)(0x100 | (j << 3) | tt);
        for (int u = 1; u < 7; ++u) p.order[n++] = (unsigned short)(((2 * u) << 3) | tt);
        if (tt < 8 - CFG_D2) for (int u = 1; u < 7; ++u) p.order[n++] = (unsigned short)(((2 * u + 1) << 3) | tt);
        if (tt < 8 - CFG_D3) for (int h = 0; h < 2; ++h) p.order[n++] = (unsigned short)((h << 3) | tt);
    }
    for (int tt = 0; tt < 8; ++tt) {
        if (tt >= 8 - CFG_D2) for (int u = 1; u < 7; ++u) p.order[n++] = (unsigned short)(((2 * u + 1) << 3) | tt);
        if (tt >= 8 - CFG_D3) for (int h = 0; h < 2; ++h) p.order[n++] = (unsigned short)((h << 3) | tt);
    }
    HIP_CHECK(hipFuncSetAttribute((const void*)gemm_rs_fused, hipFuncAttributeMaxDynamicSharedMemorySize, LDS_BYTES));
    hipLaunchKernelGGL(gemm_rs_fused, dim3(GRID), dim3(512), LDS_BYTES, stream, p);
    HIP_CHECK(hipGetLastError());
}
```

```cpp
#include <hip/hip_runtime.h>
#include <rccl/rccl.h>
#include <cstdio>
#include <cstdint>

#define HIP_CHECK(expr) do { hipError_t _e = (expr); if (_e != hipSuccess) { \
    fprintf(stderr, "HIP error %s at %s:%d\n", hipGetErrorString(_e), __FILE__, __LINE__); \
    return; } } while (0)

namespace {

#define LAS __attribute__((address_space(3)))
typedef _Float16 h8 __attribute__((ext_vector_type(8)));
typedef float f32x4 __attribute__((ext_vector_type(4)));
typedef unsigned u32x4 __attribute__((ext_vector_type(4)));
typedef __amdgpu_buffer_rsrc_t rsrc_t;
typedef unsigned long long u64;

constexpr int NR = 8;
constexpr int KSH = 512;
constexpr int NG = 16384;
constexpr int BLOB = 32768;
#define CFG_NW 6
#define CFG_PW 4
#define CFG_PU 2
#define CFG_SL 4
#define CFG_D2 4
#define CFG_D3 4
constexpr int NW = CFG_NW;
constexpr int PW = CFG_PW;
constexpr int PU = CFG_PU;
constexpr int SL = CFG_SL;
constexpr int SLICE = BLOB / SL;
constexpr int SPI = SLICE / (PU * 1024);
constexpr int GRID = 256, NGEMM = 128, NPULL = (NR - 1) * NW, NCONV = GRID - NPULL, NHELP = GRID - NGEMM - NPULL;
constexpr int N_IN = 14 * 8 + 8 * 8;
constexpr int N_OUT = 128;
constexpr int N_ITEMS = N_IN + N_OUT;
constexpr unsigned N_UNITS = N_ITEMS * SL, TICKETS = N_UNITS + NW * PW;
static_assert(4 % PU == 0 && SPI * PU * 1024 * SL == BLOB && SPI >= 2, "puller geometry");
static_assert(NHELP > 0 && N_ITEMS <= 512, "need helper workgroups");

constexpr int ID_XR = 0;
constexpr int ID_WR = 1024;
constexpr int ID_P = 1536;
constexpr int ID_PAR = 2560;
constexpr int PO_X = 0, PO_W = 128, PO_C = 640, PAR_SZ = 1664;
constexpr int N_IDS = ID_PAR + 2 * PAR_SZ;
constexpr unsigned OFF_CTR = 0;
constexpr unsigned OFF_TMO = 32768;
constexpr unsigned OFF_TK = 36864;
constexpr unsigned OFF_RF = 65536;
constexpr unsigned OFF_LC = 131072;
constexpr unsigned OFF_DBG = 262144;
constexpr unsigned OFF_DATA = 1u << 20;
constexpr size_t WS_NEED = (size_t)OFF_DATA + (size_t)N_IDS * BLOB;
constexpr unsigned OUT_BYTES = 512u * NG * 4u;
constexpr u64 DEADLINE = 50000000ull;
constexpr int AUX_SC1 = 16, AUX_SYS = 17;
constexpr int LDS_BYTES = 131072;

struct Params {
    const float* x; const float* w; float* out;
    char* ws[NR];
    int rank;
    unsigned short order[N_IN];
};

#define MEMFENCE() asm volatile("" ::: "memory")
#define WAITV_IMM(n) (((n) & 15) | (7 << 4) | (15 << 8) | (((n) >> 4) << 14))
#define WAITV(n) do { MEMFENCE(); __builtin_amdgcn_s_waitcnt(WAITV_IMM(n)); MEMFENCE(); } while (0)

#define STAMP(slot) do {} while (0)

__device__ __forceinline__ unsigned ld_agent(const void* p) { return __hip_atomic_load((const unsigned*)p, __ATOMIC_RELAXED, __HIP_MEMORY_SCOPE_AGENT); }
__device__ __forceinline__ unsigned ld_sys(const void* p) { return __hip_atomic_load((const unsigned*)p, __ATOMIC_RELAXED, __HIP_MEMORY_SCOPE_SYSTEM); }
__device__ __forceinline__ void st_agent(void* p, unsigned v) { __hip_atomic_store((unsigned*)p, v, __ATOMIC_RELAXED, __HIP_MEMORY_SCOPE_AGENT); }
__device__ __forceinline__ void st_sys(void* p, unsigned v) { __hip_atomic_store((unsigned*)p, v, __ATOMIC_RELAXED, __HIP_MEMORY_SCOPE_SYSTEM); }
__device__ __forceinline__ rsrc_t mk_rsrc(const void* p, unsigned bytes) { return __builtin_amdgcn_make_buffer_rsrc((void*)p, 0, bytes, 0x00020000); }
__device__ __forceinline__ unsigned uni(unsigned v) { return (unsigned)__builtin_amdgcn_readfirstlane((int)v); }

__device__ __forceinline__ bool give_up(char* wsl, unsigned call, u64 t0) {
    if (uni(ld_agent(wsl + OFF_TMO)) == call || __builtin_amdgcn_s_memrealtime() - t0 > DEADLINE) {
        st_agent(wsl + OFF_TMO, call);
        return true;
    }
    return false;
}

#define CONV_LD(p) (*(p))
__device__ __forceinline__ void conv_load(const Params& p, int n, int wid, int lane, f32x4 (&v)[8]) {
    const int tt = n / 80, r = n % 80, g = lane >> 4, c = lane & 15;
    if (r < 16) {
        const float* src = p.x + (size_t)(r * 256 + wid * 32 + c) * KSH + tt * 64 + 4 * g;
#pragma unroll
        for (int f = 0; f < 2; ++f)
#pragma unroll
            for (int ii = 0; ii < 4; ++ii) v[f * 4 + ii] = CONV_LD((const f32x4*)(src + f * 16 * KSH + ii * 16));
    } else {
        const int ks = wid & 1, wc = wid >> 1;
        const float* src = p.w + (size_t)(tt * 64 + ks * 32 + 4 * g) * NG + (r - 16) * 256 + wc * 64 + 4 * c;
#pragma unroll
        for (int e = 0; e < 8; ++e) v[e] = CONV_LD((const f32x4*)(src + (size_t)((e >> 2) * 16 + (e & 3)) * NG));
    }
}
__device__ __forceinline__ void conv_store(rsrc_t rsl, int n, int pb, int wid, int lane, const f32x4 (&v)[8]) {
    const int tt = n / 80, r = n % 80;
    if (r < 16) {
        const unsigned base = OFF_DATA + (unsigned)(pb + PO_X + r * 8 + tt) * BLOB + wid * 4096;
#pragma unroll
        for (int f = 0; f < 2; ++f)
#pragma unroll
            for (int ks = 0; ks < 2; ++ks) {
                h8 o;
#pragma unroll
                for (int e = 0; e < 8; ++e) o[e] = (_Float16)v[f * 4 + ks * 2 + (e >> 2)][e & 3];
                __builtin_amdgcn_raw_buffer_store_b128(__builtin_bit_cast(u32x4, o), rsl, lane * 16, base + f * 2048 + ks * 1024, AUX_SYS);
            }
    } else {
        const int ks = wid & 1, wc = wid >> 1;
        const unsigned base = OFF_DATA + (unsigned)(pb + PO_W + (r - 16) * 8 + tt) * BLOB + wc * 4096 + ks * 1024;
#pragma unroll
        for (int f = 0; f < 4; ++f) {
            h8 o;
#pragma unroll
            for (int e = 0; e < 8; ++e) o[e] = (_Float16)v[e][f];
            __builtin_amdgcn_raw_buffer_store_b128(__builtin_bit_cast(u32x4, o), rsl, lane * 16, base + (f >> 1) * 16384 + (f & 1) * 2048, AUX_SYS);
        }
    }
}
__device__ __forceinline__ void conv_publish(int rank, char* wst  , int n, int pb, unsigned call) {
    const int tt = n / 80, r = n % 80, t = threadIdx.x;
    if (t < NR) {
        if (t == rank) st_agent(wst + OFF_LC + 4 * (pb + (r < 16 ? PO_X + r * 8 : PO_W + (r - 16) * 8) + tt), SL * call);
        else if (r < 16) { if ((r >> 1) != rank) st_sys(wst + OFF_RF + 4 * (ID_XR + r * 64 + rank * 8 + tt), call); }
        else if (((r - 16) >> 3) == t) st_sys(wst + OFF_RF + 4 * (ID_WR + ((r - 16) & 7) * 64 + rank * 8 + tt), call);
    }
}

struct PStep { unsigned src, dst, did; bool valid, last; };
constexpr int VOFF_OOB = 0x7ffffff0;

__device__ __forceinline__ void pull_items(char* wsl, rsrc_t rsl, rsrc_t rsp, rsrc_t rso, const LAS unsigned* desc  ,
                                           unsigned call, int lane, int peer) {
    const int lane16 = lane * 16, lane_out = (lane >> 4) * 4 * (NG * 4) + (lane & 15) * 16;
    const u64 t0 = __builtin_amdgcn_s_memrealtime();
    const int lane0 = lane == 0 ? 0 : VOFF_OOB;
    const unsigned tk_base = (call - 1u) * TICKETS;
    bool dead = false;
    int sub = SPI;
    int nstate = 0;
    unsigned cur = 0, cofs = 0, nxt = 0, nofs = 0, pend = 0xffffffffu, spins = 0;
    auto step = [&](const PStep& cu, const u32x4 (&rc)[PU], const u32x4 (&qc)[PU], PStep& nx, u32x4 (&rn)[PU], u32x4 (&qn)[PU]) -> bool {
        MEMFENCE();
        if (sub == SPI && nstate == 2) { cur = nxt; cofs = nofs; sub = 0; nstate = 0; }
        nx.valid = sub < SPI;
        const unsigned inb = cofs + (nx.valid ? sub : 0) * (PU * 1024);
        nx.did = cur & 0xffffu;
        nx.src = OFF_DATA + (cur >> 16) * BLOB + inb;
        nx.dst = OFF_DATA + nx.did * BLOB + inb;
        if (nx.valid) ++sub;
        nx.last = sub == SPI;
        const int vt = nstate == 0 ? lane0 : VOFF_OOB, vf = nstate == 1 ? lane0 : VOFF_OOB;
        const unsigned f0 = (unsigned)__builtin_amdgcn_raw_ptr_buffer_atomic_add_i32(1, rsl, vt, OFF_TK + peer * 128, 0);
        const unsigned f1 = __builtin_amdgcn_raw_buffer_load_b32(rsl, vf, OFF_RF + 4 * (nxt & 0xffffu), AUX_SYS);
        const unsigned f2 = __builtin_amdgcn_raw_buffer_load_b32(rsl, vf, OFF_LC + 4 * (nxt & 0xffffu), AUX_SC1);
        MEMFENCE();
        const int vr = nx.valid ? lane16 : VOFF_OOB, vq = (nx.valid && nx.did >= (unsigned)ID_P) ? lane16 : VOFF_OOB;
#pragma unroll
        for (int u = 0; u < PU; ++u) rn[u] = __builtin_amdgcn_raw_buffer_load_b128(rsp, vr, nx.src + u * 1024, AUX_SYS);
#pragma unroll
        for (int u = 0; u < PU; ++u) qn[u] = __builtin_amdgcn_raw_buffer_load_b128(rsl, vq, nx.dst + u * 1024, AUX_SC1);
        WAITV(2 * PU);
        if (pend != 0xffffffffu) {
            if (lane == 0) __hip_atomic_fetch_add((unsigned*)(wsl + OFF_LC + 4 * pend), 1u, __ATOMIC_RELAXED, __HIP_MEMORY_SCOPE_AGENT);
            pend = 0xffffffffu;
        }
        if (cu.valid) {
            if (cu.did >= (unsigned)ID_P) {
                const unsigned cb = (cu.did - ID_P) & 127u, e = ((cu.dst - OFF_DATA) & (BLOB - 1)) >> 10;
                const unsigned row = (cb >> 6) * 256 + (e >> 4) * 128 + ((cb >> 2) & 1) * 64 + ((e >> 2) & 3) * 16 + (e & 3);
                const unsigned col = peer * 2048 + ((cb >> 3) & 7) * 256 + (cb & 3) * 64;
                const unsigned so = row * (NG * 4) + col * 4;
#pragma unroll
                for (int u = 0; u < PU; ++u) {
                    f32x4 v = __builtin_bit_cast(f32x4, rc[u]) + __builtin_bit_cast(f32x4, qc[u]);
                    if (dead) v = (f32x4){__builtin_nanf(""), __builtin_nanf(""), __builtin_nanf(""), __builtin_nanf("")};
                    __builtin_amdgcn_raw_buffer_store_b128(__builtin_bit_cast(u32x4, v), rso, lane_out, so + u * (NG * 4), AUX_SC1);
                }
            } else {
#pragma unroll
                for (int u = 0; u < PU; ++u) __builtin_amdgcn_raw_buffer_store_b128(rc[u], rsl, lane16, cu.dst + u * 1024, AUX_SC1);
                if (cu.last) pend = cu.did;
            }
        }
        const unsigned g0 = uni(f0), g1 = uni(f1), g2 = uni(f2);
        if (nstate == 0) {
            const unsigned un = g0 - tk_base;
            if (un < N_UNITS) {
                nxt = uni(desc[un / SL]); nofs = (un % SL) * SLICE; nstate = dead ? 2 : 1;
            } else nstate = 3;
        } else if (nstate == 1) {
            if (g1 >= call && ((nxt & 0xffffu) < (unsigned)ID_P || g2 >= SL * call)) nstate = 2;
            else if (!cu.valid && !nx.valid) {
                if ((++spins & 63u) == 0 && give_up(wsl, call, t0)) { dead = true; nstate = 2; }
                __builtin_amdgcn_s_sleep(2);
            }
        }
        return !cu.valid && !nx.valid && nstate == 3;
    };
    PStep sa = {0, 0, 0, false, false}, sb;
    u32x4 ra[PU] = {}, qa[PU] = {}, rb[PU], qb[PU];
    bool done;
    do {
        step(sa, ra, qa, sb, rb, qb);
        done = step(sb, rb, qb, sa, ra, qa);
    } while (!done);
}

__global__ __launch_bounds__(512, 2) void gemm_rs_fused(const Params p) {
    extern __shared__ __attribute__((aligned(16))) char shm[];
    LAS char* lds = (LAS char*)shm;
    const int b = blockIdx.x, tid = threadIdx.x, lane = tid & 63, wid = __builtin_amdgcn_readfirstlane(tid >> 6);
    const int rank = p.rank;
    char* wsl = p.ws[rank];
    const rsrc_t rsl = mk_rsrc(wsl, (unsigned)WS_NEED);
    const rsrc_t rso = mk_rsrc(p.out, OUT_BYTES);
    STAMP(0);

    if (b >= NGEMM && b < NGEMM + NPULL) {
        const unsigned call = uni(ld_agent(wsl + OFF_CTR + b * 128)) + 1u;
        const int pi = b - NGEMM, ds = 1 + pi % (NR - 1), peer = (rank + ds) & (NR - 1);
        const int pb = ID_PAR + (int)(call & 1u) * PAR_SZ;
        const rsrc_t rsp = mk_rsrc(p.ws[peer], (unsigned)WS_NEED);
        LAS unsigned* desc = (LAS unsigned*)lds;
        if (tid < N_ITEMS) {
            unsigned did, sid;
            if (tid < N_IN) {
                const unsigned code = p.order[tid];
                const unsigned tt = code & 7u, a = (code >> 3) & 31u;
                if (code >> 8) { did = ID_WR + a * 64 + peer * 8 + tt; sid = pb + PO_W + rank * 64 + a * 8 + tt; }
                else {
                    unsigned d = a >> 1;
                    if (d >= (unsigned)ds) ++d;
                    const unsigned i = (((rank + d) & (NR - 1)) << 1) | (a & 1u);
                    did = ID_XR + i * 64 + peer * 8 + tt; sid = pb + PO_X + i * 8 + tt;
                }
            } else {
                const unsigned cb = tid - N_IN;
                did = ID_P + peer * 128 + cb; sid = pb + PO_C + rank * 128 + cb;
            }
            desc[tid] = (sid << 16) | did;
        }
        asm volatile("s_waitcnt vmcnt(0) lgkmcnt(0)" ::: "memory");
        __builtin_amdgcn_s_barrier();
        MEMFENCE();
        if (tid == 0) st_agent(wsl + OFF_CTR + b * 128, call);
        if (wid >= PW) return;
        pull_items(wsl, rsl, rsp, rso, desc, call, lane, peer);
        STAMP(2);
        return;
    }

    unsigned call;
    int pb;
    {
        const int ci = b < NGEMM ? b : b - NPULL;
        static_assert(2 * NCONV <= 640 && 3 * NCONV >= 640, "three blobs per converter at most, two at least");
        const int n2 = ci + 2 * NCONV < 640 ? ci + 2 * NCONV : ci;
        f32x4 v0[8], v1[8], v2[8];
        const unsigned c0 = ld_agent(wsl + OFF_CTR + b * 128);
        char* wst = p.ws[0];
#pragma unroll
        for (int r = 1; r < NR; ++r) wst = tid == r ? p.ws[r] : wst;
        MEMFENCE();
        conv_load(p, ci, wid, lane, v0);
        call = uni(c0) + 1u;
        pb = ID_PAR + (int)(call & 1u) * PAR_SZ;
        conv_store(rsl, ci, pb, wid, lane, v0);
        STAMP(5);
        MEMFENCE();
        conv_load(p, ci + NCONV, wid, lane, v1);
        conv_load(p, n2, wid, lane, v2);
        WAITV(16);
        __builtin_amdgcn_s_barrier();
        MEMFENCE();
        conv_publish(rank, wst, ci, pb, call);
        STAMP(6);
        if (tid == 0) st_agent(wsl + OFF_CTR + b * 128, call);
        conv_store(rsl, ci + NCONV, pb, wid, lane, v1);
        if (n2 != ci) conv_store(rsl, n2, pb, wid, lane, v2);
        STAMP(7);
        WAITV(0);
        __builtin_amdgcn_s_barrier();
        MEMFENCE();
        conv_publish(rank, wst, ci + NCONV, pb, call);
        if (n2 != ci) conv_publish(rank, wst, n2, pb, call);
    }
    STAMP(1);

    const unsigned tgt = SL * call;
    const int wr = wid >> 2, wc = wid & 3;
    for (int job = 0;; ++job) {
        int nsteps, cid, fl_rank;
        unsigned ids, fl_idx, out_off = 0;
        if (b < NGEMM) {
            if (job) break;
            const int xcd = b & 7, kk = b >> 3, i = (xcd >> 1) * 4 + (kk & 3), j = (xcd & 1) * 4 + (kk >> 2), own = i >> 1;
            const int nrem = own == rank ? 7 : 6;
            nsteps = 8 + 8 * nrem;
            const int m = (lane < nsteps ? lane : 0) - 8;
            if (m < 0) ids = (unsigned)(pb + PO_X + i * 8 + m + 8) | ((unsigned)(pb + PO_W + rank * 64 + j * 8 + m + 8) << 16);
            else {
                const int tt = m / nrem; int d = m % nrem + 1;
                if (own != rank && d >= ((own - rank) & (NR - 1))) ++d;
                const int s = (rank + d) & (NR - 1);
                ids = (unsigned)(ID_XR + i * 64 + s * 8 + tt) | ((unsigned)(ID_WR + j * 64 + s * 8 + tt) << 16);
            }
            const int cb = (i & 1) * 64 + j * 8;
            cid = own == rank ? -1 : pb + PO_C + own * 128 + cb;
            fl_rank = own; fl_idx = ID_P + rank * 128 + cb;
            out_off = (unsigned)((i & 1) * 256) * (NG * 4) + (unsigned)(rank * 2048 + j * 256) * 4;
        } else {
            const int pt = (b - NGEMM - NPULL) + job * NHELP;
            if (pt >= 112) break;
            const int h = pt / 56, rem = pt % 56, q = (rank + 1 + (rem >> 3)) & (NR - 1), j = rem & 7, m = lane & 7;
            nsteps = 8;
            ids = (unsigned)(pb + PO_X + (rank * 2 + h) * 8 + m) | ((unsigned)(pb + PO_W + q * 64 + j * 8 + m) << 16);
            cid = ID_P + q * 128 + h * 64 + j * 8;
            fl_rank = rank; fl_idx = cid;
        }

        u64 mask = 0;
        auto ensure = [&](int n) {
            if (!((mask >> n) & 1ull)) {
                const u64 t0 = __builtin_amdgcn_s_memrealtime();
                for (unsigned spins = 1;; ++spins) {
                    const unsigned fa = ld_agent(wsl + OFF_LC + 4 * (ids & 0xffffu)), fb = ld_agent(wsl + OFF_LC + 4 * (ids >> 16));
                    mask = __ballot(fa >= tgt && fb >= tgt);
                    if ((mask >> n) & 1ull) break;
                    if ((spins & 63u) == 0 && give_up(wsl, call, t0)) { mask = ~0ull; break; }
                    __builtin_amdgcn_s_sleep(8);
                }
                MEMFENCE();
            }
        };
        const int lane16 = lane * 16;
        const unsigned wofs = OFF_DATA + wid * 2048;
#define SA(bf, h) (((bf) * 2 + (h)) * 16384)
#define SB(bf, h) ((4 + (bf) * 2 + (h)) * 16384)
#define STAGE(bufoff, goff) do { \
        __builtin_amdgcn_raw_ptr_buffer_load_lds(rsl, (LAS void*)(lds + (bufoff) + wid * 2048), 16, lane16, (goff), 0, AUX_SC1); \
        __builtin_amdgcn_raw_ptr_buffer_load_lds(rsl, (LAS void*)(lds + (bufoff) + wid * 2048), 16, lane16, (goff), 1024, AUX_SC1); } while (0)
#define LDA(dst, bf, h) do { _Pragma("unroll") for (int m = 0; m < 4; ++m) _Pragma("unroll") for (int k = 0; k < 2; ++k) \
        dst[m][k] = *(const LAS h8*)(lds + SA(bf, h) + aoff + m * 2048 + k * 1024); } while (0)
#define LDB(dst, bf, h) do { _Pragma("unroll") for (int n = 0; n < 2; ++n) _Pragma("unroll") for (int k = 0; k < 2; ++k) \
        dst[n][k] = *(const LAS h8*)(lds + SB(bf, h) + boff + n * 2048 + k * 1024); } while (0)
#define MMA(ai, bj, At, Bt) do { __builtin_amdgcn_s_setprio(1); \
        _Pragma("unroll") for (int m = 0; m < 4; ++m) _Pragma("unroll") for (int n = 0; n < 2; ++n) _Pragma("unroll") for (int k = 0; k < 2; ++k) \
            acc[ai][bj][m][n] = __builtin_amdgcn_mfma_f32_16x16x32_f16(At[m][k], Bt[n][k], acc[ai][bj][m][n], 0, 0, 0); \
        __builtin_amdgcn_s_setprio(0); } while (0)
#define WAIT_V(n) asm volatile("s_waitcnt vmcnt(" #n ")" ::: "memory")
#define WAIT_L(n) asm volatile("s_waitcnt lgkmcnt(" #n ")" ::: "memory")
#define BAR __builtin_amdgcn_s_barrier()
#define SCHED __builtin_amdgcn_sched_barrier(0)
#define OFFA(c) (wofs + ((c) & 0xffffu) * BLOB)
#define OFFB(c) (wofs + ((c) >> 16) * BLOB)
        const int aoff = wr * 8192 + lane16, boff = wc * 4096 + lane16;
        f32x4 acc[2][2][4][2];
#pragma unroll
        for (int a = 0; a < 2; ++a)
#pragma unroll
            for (int c = 0; c < 2; ++c)
#pragma unroll
                for (int m = 0; m < 4; ++m)
#pragma unroll
                    for (int n = 0; n < 2; ++n) acc[a][c][m][n] = (f32x4){0.f, 0.f, 0.f, 0.f};
        h8 At[4][2], B0[2][2], B1[2][2];
        ensure(0); ensure(1);
        unsigned a1;
        {
            const unsigned c0 = (unsigned)__builtin_amdgcn_readlane((int)ids, 0), c1 = (unsigned)__builtin_amdgcn_readlane((int)ids, 1);
            STAGE(SB(0, 0), OFFB(c0)); STAGE(SA(0, 0), OFFA(c0)); STAGE(SB(0, 1), OFFB(c0) + 16384); STAGE(SA(0, 1), OFFA(c0) + 16384);
            if (wr == 1) BAR;
            WAIT_V(4); BAR;
            STAGE(SB(1, 0), OFFB(c1)); STAGE(SA(1, 0), OFFA(c1)); STAGE(SB(1, 1), OFFB(c1) + 16384);
            WAIT_V(6); BAR;
            a1 = OFFA(c1);
        }
        STAMP(2);
        for (int t = 0; t < nsteps; t += 2) {
            const int t2 = t + 2 < nsteps ? t + 2 : 0;
            const unsigned c2 = (unsigned)__builtin_amdgcn_readlane((int)ids, t2), c3 = (unsigned)__builtin_amdgcn_readlane((int)ids, t2 + 1);
            const unsigned a2 = OFFA(c2), b2 = OFFB(c2), a3 = OFFA(c3), b3 = OFFB(c3);
            LDB(B0, 0, 0); SCHED; LDA(At, 0, 0); STAGE(SA(1, 1), a1 + 16384);
            WAIT_L(8); BAR; WAIT_L(0); MMA(0, 0, At, B0); BAR; SCHED;
            ensure(t2);
            LDB(B1, 0, 1); STAGE(SB(0, 0), b2);
            BAR; WAIT_L(0); MMA(0, 1, At, B1); BAR;
            LDA(At, 0, 1); STAGE(SA(0, 0), a2);
            BAR; WAIT_L(0); MMA(1, 0, At, B0); BAR; SCHED;
            STAGE(SB(0, 1), b2 + 16384);
            WAIT_V(6); BAR; MMA(1, 1, At, B1); BAR;
            LDB(B0, 1, 0); SCHED; LDA(At, 1, 0); STAGE(SA(0, 1), a2 + 16384);
            WAIT_L(8); BAR; WAIT_L(0); MMA(0, 0, At, B0); BAR; SCHED;
            ensure(t2 + 1);
            LDB(B1, 1, 1); STAGE(SB(1, 0), b3);
            BAR; WAIT_L(0); MMA(0, 1, At, B1); BAR;
            LDA(At, 1, 1); STAGE(SA(1, 0), a3);
            BAR; WAIT_L(0); MMA(1, 0, At, B0); BAR; SCHED;
            STAGE(SB(1, 1), b3 + 16384);
            WAIT_V(6); BAR; MMA(1, 1, At, B1); BAR;
            a1 = a3;
        }
        STAMP(3);
        const bool dead = uni(ld_agent(wsl + OFF_TMO)) == call;
        const float qn = __builtin_nanf("");
        if (cid >= 0) {
            const unsigned base = OFF_DATA + (unsigned)(cid + wid) * BLOB;
#pragma unroll
            for (int ai = 0; ai < 2; ++ai)
#pragma unroll
                for (int m = 0; m < 4; ++m)
#pragma unroll
                    for (int r = 0; r < 4; ++r) {
                        f32x4 v = {acc[ai][0][m][0][r], acc[ai][0][m][1][r], acc[ai][1][m][0][r], acc[ai][1][m][1][r]};
                        if (dead) v = (f32x4){qn, qn, qn, qn};
                        __builtin_amdgcn_raw_buffer_store_b128(__builtin_bit_cast(u32x4, v), rsl, lane16, base + (ai * 16 + m * 4 + r) * 1024, AUX_SYS);
                    }
            WAIT_V(0);
            if (lane == 0) {
                if (fl_rank == rank) st_agent(wsl + OFF_LC + 4 * (fl_idx + wid), tgt);
                else if ((unsigned)fl_rank < (unsigned)NR) st_sys(p.ws[fl_rank] + OFF_RF + 4 * (fl_idx + wid), call);
            }
        } else {
            const int lane_out = (lane >> 4) * 4 * (NG * 4) + (lane & 15) * 16;
            const unsigned base = out_off + (unsigned)(wr * 64) * (NG * 4) + wc * 256;
#pragma unroll
            for (int ai = 0; ai < 2; ++ai)
#pragma unroll
                for (int m = 0; m < 4; ++m)
#pragma unroll
                    for (int r = 0; r < 4; ++r) {
                        f32x4 v = {acc[ai][0][m][0][r], acc[ai][0][m][1][r], acc[ai][1][m][0][r], acc[ai][1][m][1][r]};
                        if (dead) v = (f32x4){qn, qn, qn, qn};
                        __builtin_amdgcn_raw_buffer_store_b128(__builtin_bit_cast(u32x4, v), rso, lane_out, base + (unsigned)(ai * 128 + m * 16 + r) * (NG * 4), AUX_SC1);
                    }
            WAIT_V(0);
        }
        STAMP(4);
        if (wr == 0) BAR;
        BAR;
    }
}

}

extern "C" void kernel_launch(int rank, int nranks, ncclComm_t comm,
                              hipStream_t stream,
                              void* const* d_in, const int* in_sizes, int n_in,
                              void* d_out, int out_size,
                              const int* in_dtypes, int out_dtype,
                              void* const* d_out_all,
                              void* const* d_ws_all, size_t ws_bytes) {
    (void)comm; (void)in_sizes; (void)n_in; (void)in_dtypes; (void)out_dtype; (void)d_out_all;
    if (nranks != NR || (unsigned)rank >= (unsigned)NR || ws_bytes < WS_NEED || (size_t)out_size * 4 != OUT_BYTES) {
        fprintf(stderr, "kernel_launch: unexpected configuration\n");
        return;
    }
    Params p;
    p.x = (const float*)d_in[0];
    p.w = (const float*)d_in[1];
    p.out = (float*)d_out;
    for (int r = 0; r < NR; ++r) p.ws[r] = (char*)d_ws_all[r];
    p.rank = rank;
    int n = 0;
    for (int tt = 0; tt < 8; ++tt) {
        for (int j = 0; j < 8; ++j) p.order[n++] = (unsigned short)(0x100 | (j << 3) | tt);
        for (int u = 1; u < 7; ++u) p.order[n++] = (unsigned short)(((2 * u) << 3) | tt);
        if (tt < 8 - CFG_D2) for (int u = 1; u < 7; ++u) p.order[n++] = (unsigned short)(((2 * u + 1) << 3) | tt);
        if (tt < 8 - CFG_D3) for (int h = 0; h < 2; ++h) p.order[n++] = (unsigned short)((h << 3) | tt);
    }
    for (int tt = 0; tt < 8; ++tt) {
        if (tt >= 8 - CFG_D2) for (int u = 1; u < 7; ++u) p.order[n++] = (unsigned short)(((2 * u + 1) << 3) | tt);
        if (tt >= 8 - CFG_D3) for (int h = 0; h < 2; ++h) p.order[n++] = (unsigned short)((h << 3) | tt);
    }
    HIP_CHECK(hipFuncSetAttribute((const void*)gemm_rs_fused, hipFuncAttributeMaxDynamicSharedMemorySize, LDS_BYTES));
    hipLaunchKernelGGL(gemm_rs_fused, dim3(GRID), dim3(512), LDS_BYTES, stream, p);
    HIP_CHECK(hipGetLastError());
}
```

```cpp
#include <hip/hip_runtime.h>
#include <rccl/rccl.h>
#include <cstdio>
#include <cstdint>

#define HIP_CHECK(expr) do { hipError_t _e = (expr); if (_e != hipSuccess) { \
    fprintf(stderr, "HIP error %s at %s:%d\n", hipGetErrorString(_e), __FILE__, __LINE__); \
    return; } } while (0)

namespace {

#define LAS __attribute__((address_space(3)))
typedef _Float16 h8 __attribute__((ext_vector_type(8)));
typedef float f32x4 __attribute__((ext_vector_type(4)));
typedef unsigned u32x4 __attribute__((ext_vector_type(4)));
typedef __amdgpu_buffer_rsrc_t rsrc_t;
typedef unsigned long long u64;

constexpr int NR = 8;
constexpr int KSH = 512;
constexpr int NG = 16384;
constexpr int BLOB = 32768;
#define CFG_NW 6
#define CFG_PW 4
#define CFG_PU 2
#define CFG_NPT 1
#define CFG_D2 5
#define CFG_D3 5
constexpr int NW = CFG_NW;
constexpr int PW = CFG_PW;
constexpr int PU = CFG_PU;
constexpr int SL = 4;
constexpr int SLICE = BLOB / SL;
constexpr int SPI = SLICE / (PU * 1024);
constexpr int GRID = 256, NGEMM = 128, NPULL = (NR - 1) * NW, NCONV = GRID - NPULL, NHELP = GRID - NGEMM - NPULL;
constexpr int NPT = CFG_NPT;
constexpr int N_IN = 14 * 8 + 8 * (8 - NPT);
constexpr int N_OUT = 128;
constexpr int N_ITEMS = N_IN + N_OUT;
constexpr unsigned N_UNITS = N_ITEMS * SL, TICKETS = N_UNITS + NW * PW;
static_assert(4 % PU == 0 && SPI * PU * 1024 * SL == BLOB && SPI >= 2, "puller geometry");
static_assert(NHELP > 0 && N_ITEMS <= 512, "need helper workgroups");

constexpr int ID_XR = 0;
constexpr int ID_WR = 1024;
constexpr int ID_P = 1536;
constexpr int ID_PAR = 2560;
constexpr int PO_X = 0, PO_W = 128, PO_C = 640, PO_WP = 1664  , PAR_SZ = 1792;
constexpr int ID_ZERO = ID_PAR + 2 * PAR_SZ;
constexpr int N_IDS = ID_ZERO + 1;
constexpr unsigned OFF_CTR = 0;
constexpr unsigned OFF_TMO = 32768;
constexpr unsigned OFF_TK = 36864;
constexpr unsigned OFF_RF0 = 65536;
constexpr unsigned OFF_LC = 131072;
constexpr unsigned OFF_DBG = 262144;
constexpr unsigned OFF_DATA = 1u << 20;
static_assert(OFF_LC + 16u * N_IDS <= OFF_DBG && OFF_RF0 + 8u * ID_PAR <= OFF_LC, "control layout");
constexpr size_t WS_NEED = (size_t)OFF_DATA + (size_t)N_IDS * BLOB;
constexpr unsigned OUT_BYTES = 512u * NG * 4u;
constexpr u64 DEADLINE = 50000000ull;
constexpr int AUX_SC1 = 16, AUX_SYS = 17;
constexpr int LDS_BYTES = 131072;
constexpr int VOFF_OOB = 0x70000000;

struct Params {
    const float* x; const float* w; float* out;
    char* ws[NR];
    int rank;
    unsigned short order[N_IN];
};

#define MEMFENCE() asm volatile("" ::: "memory")
#define WAITV_IMM(n) (((n) & 15) | (7 << 4) | (15 << 8) | (((n) >> 4) << 14))
#define WAITV(n) do { MEMFENCE(); __builtin_amdgcn_s_waitcnt(WAITV_IMM(n)); MEMFENCE(); } while (0)

#define STAMP(slot) do {} while (0)

__device__ __forceinline__ unsigned ld_agent(const void* p) { return __hip_atomic_load((const unsigned*)p, __ATOMIC_RELAXED, __HIP_MEMORY_SCOPE_AGENT); }
__device__ __forceinline__ unsigned ld_sys(const void* p) { return __hip_atomic_load((const unsigned*)p, __ATOMIC_RELAXED, __HIP_MEMORY_SCOPE_SYSTEM); }
__device__ __forceinline__ void st_agent(void* p, unsigned v) { __hip_atomic_store((unsigned*)p, v, __ATOMIC_RELAXED, __HIP_MEMORY_SCOPE_AGENT); }
__device__ __forceinline__ void st_sys(void* p, unsigned v) { __hip_atomic_store((unsigned*)p, v, __ATOMIC_RELAXED, __HIP_MEMORY_SCOPE_SYSTEM); }
__device__ __forceinline__ rsrc_t mk_rsrc(const void* p, unsigned bytes) { return __builtin_amdgcn_make_buffer_rsrc((void*)p, 0, bytes, 0x00020000); }
template <int AUX> __device__ __forceinline__ void st128(u32x4 v, rsrc_t rs, int voff, unsigned soff) {
    __builtin_amdgcn_raw_buffer_store_b128(v, rs, voff, soff, AUX);
    asm volatile("s_nop 1" :: "v"(v));
}
__device__ __forceinline__ unsigned off_rf(unsigned call) { return OFF_RF0 + (call & 1u) * (4u * ID_PAR); }
__device__ __forceinline__ unsigned uni(unsigned v) { return (unsigned)__builtin_amdgcn_readfirstlane((int)v); }

__device__ __forceinline__ bool give_up(char* wsl, unsigned call, u64 t0) {
    if (uni(ld_agent(wsl + OFF_TMO)) == call || __builtin_amdgcn_s_memrealtime() - t0 > DEADLINE) {
        st_agent(wsl + OFF_TMO, call);
        return true;
    }
    return false;
}

#define CONV_LD(p) (*(p))
__device__ __forceinline__ void conv_load(const Params& p, int n, int wid, int lane, f32x4 (&v)[8]) {
    const int tt = n / 80, r = n % 80, g = lane >> 4, c = lane & 15;
    if (r < 16) {
        const float* src = p.x + (size_t)(r * 256 + wid * 32 + c) * KSH + tt * 64 + 4 * g;
#pragma unroll
        for (int f = 0; f < 2; ++f)
#pragma unroll
            for (int ii = 0; ii < 4; ++ii) v[f * 4 + ii] = CONV_LD((const f32x4*)(src + f * 16 * KSH + ii * 16));
    } else {
        const int ks = wid & 1, wc = wid >> 1;
        const float* src = p.w + (size_t)(tt * 64 + ks * 32 + 4 * g) * NG + (r - 16) * 256 + wc * 64 + 4 * c;
#pragma unroll
        for (int e = 0; e < 8; ++e) v[e] = CONV_LD((const f32x4*)(src + (size_t)((e >> 2) * 16 + (e & 3)) * NG));
    }
}
__device__ __forceinline__ unsigned conv_class(const f32x4 (&v)[8]) {
    float m = 0.f;
#pragma unroll
    for (int e = 0; e < 8; ++e)
#pragma unroll
        for (int j = 0; j < 4; ++j) m = __builtin_fmaxf(m, __builtin_fabsf(v[e][j]));
    const unsigned c = ((__builtin_bit_cast(unsigned, m) >> 23) + 3u) >> 4;
    unsigned c0 = uni(c);
    for (;;) {
        const u64 up = __ballot(c > c0);
        if (!up) break;
        c0 = (unsigned)__builtin_amdgcn_readlane((int)c, __builtin_ctzll(up));
    }
    return c0;
}
template <bool MAY_PUSH>
__device__ __forceinline__ void conv_store(const Params& p, rsrc_t rsl, int n, int pb, int wid, int lane, const f32x4 (&v)[8], unsigned cls) {
    const int tt = n / 80, r = n % 80, sh = 128 - 16 * (int)cls;
    if (r < 16) {
        const unsigned base = OFF_DATA + (unsigned)(pb + PO_X + r * 8 + tt) * BLOB + wid * 4096;
#pragma unroll
        for (int f = 0; f < 2; ++f)
#pragma unroll
            for (int ks = 0; ks < 2; ++ks) {
                h8 o;
#pragma unroll
                for (int e = 0; e < 8; ++e) o[e] = (_Float16)__builtin_ldexpf(v[f * 4 + ks * 2 + (e >> 2)][e & 3], sh);
                st128<AUX_SYS>(__builtin_bit_cast(u32x4, o), rsl, lane * 16, base + f * 2048 + ks * 1024);
            }
    } else {
        const int ks = wid & 1, wc = wid >> 1;
        const unsigned inb = wc * 4096 + ks * 1024, base = OFF_DATA + (unsigned)(pb + PO_W + (r - 16) * 8 + tt) * BLOB + inb;
        h8 o[4];
#pragma unroll
        for (int f = 0; f < 4; ++f)
#pragma unroll
            for (int e = 0; e < 8; ++e) o[f][e] = (_Float16)__builtin_ldexpf(v[e][f], sh);
        const int q = ((r - 16) >> 3) & (NR - 1);
        if (MAY_PUSH && tt < NPT && q != p.rank) {
            const rsrc_t rsq = mk_rsrc(p.ws[q], (unsigned)WS_NEED);
            const unsigned rb = OFF_DATA + (unsigned)(pb + PO_WP + (((r - 16) & 7) * 8 + p.rank) * 2 + tt) * BLOB + inb;
#pragma unroll
            for (int f = 0; f < 4; ++f) st128<AUX_SYS>(__builtin_bit_cast(u32x4, o[f]), rsq, lane * 16, rb + (f >> 1) * 16384 + (f & 1) * 2048);
        }
#pragma unroll
        for (int f = 0; f < 4; ++f) st128<AUX_SYS>(__builtin_bit_cast(u32x4, o[f]), rsl, lane * 16, base + (f >> 1) * 16384 + (f & 1) * 2048);
    }
}
template <bool MAY_PUSH>
__device__ __forceinline__ unsigned conv_agree(const Params& p, rsrc_t rsl, int n, int pb, int wid, int lane, const f32x4 (&v)[8], unsigned mine,
                                               const LAS unsigned* cls) {
    const u32x4 a = *(const LAS u32x4*)cls, b = *(const LAS u32x4*)(cls + 4);
    const unsigned hi = uni(max(max(max(a[0], a[1]), max(a[2], a[3])), max(max(b[0], b[1]), max(b[2], b[3]))));
    const unsigned lo = uni(min(min(min(a[0], a[1]), min(a[2], a[3])), min(min(b[0], b[1]), min(b[2], b[3]))));
    if (lo != hi) {
        if (mine != hi) conv_store<MAY_PUSH>(p, rsl, n, pb, wid, lane, v, hi);
        WAITV(0);
        __builtin_amdgcn_s_barrier();
        MEMFENCE();
    }
    return hi;
}
__device__ __forceinline__ void conv_publish(int rank, char* wst  , int n, int pb, unsigned tag) {
    const int tt = n / 80, r = n % 80, t = threadIdx.x >> 2, wd = threadIdx.x & 3;
    const unsigned OFF_RF = off_rf(tag >> 8);
    if (t < NR) {
        if (t == rank) st_agent(wst + OFF_LC + 16 * (pb + (r < 16 ? PO_X + r * 8 : PO_W + (r - 16) * 8) + tt) + 4 * wd, tag);
        else if (r < 16) { if ((r >> 1) != rank && wd == 0) st_sys(wst + OFF_RF + 4 * (ID_XR + r * 64 + rank * 8 + tt), tag); }
        else if (((r - 16) >> 3) == t) {
            if (tt < NPT) st_sys(wst + OFF_LC + 16 * (pb + PO_WP + (((r - 16) & 7) * 8 + rank) * 2 + tt) + 4 * wd, tag);
            else if (wd == 0) st_sys(wst + OFF_RF + 4 * (ID_WR + ((r - 16) & 7) * 64 + rank * 8 + tt), tag);
        }
    }
}

struct PStep { unsigned src, dst, did, tag; bool valid, last; };

__device__ __forceinline__ void pull_items(char* wsl, rsrc_t rsl, rsrc_t rsp, rsrc_t rso, const LAS unsigned* desc  ,
                                           unsigned call, int lane, int peer) {
    const int lane16 = lane * 16, lane_out = (lane >> 4) * 4 * (NG * 4) + (lane & 15) * 16;
    const u64 t0 = __builtin_amdgcn_s_memrealtime();
    const int lane0 = lane == 0 ? 0 : VOFF_OOB;
    const unsigned tk_base = (call - 1u) * TICKETS;
    bool dead = false;
    int sub = SPI;
    int nstate = 0;
    unsigned cur = 0, cofs = 0, ctag = 0, nxt = 0, nofs = 0, ntag = 0, pend = 0, ptag = 0, spins = 0;
    const unsigned tag0 = call << 8, OFF_RF = off_rf(call);
    auto step = [&](const PStep& cu, const u32x4 (&rc)[PU], const u32x4 (&qc)[PU], PStep& nx, u32x4 (&rn)[PU], u32x4 (&qn)[PU]) -> bool {
        MEMFENCE();
        if (sub == SPI && nstate == 2) { cur = nxt; cofs = nofs; ctag = ntag; sub = 0; nstate = 0; }
        nx.valid = sub < SPI;
        const unsigned inb = cofs + (nx.valid ? sub : 0) * (PU * 1024);
        nx.did = cur & 0xffffu;
        nx.tag = ctag;
        nx.src = OFF_DATA + (cur >> 16) * BLOB + inb;
        nx.dst = OFF_DATA + nx.did * BLOB + inb;
        if (nx.valid) ++sub;
        nx.last = sub == SPI;
        const int vt = nstate == 0 ? lane0 : VOFF_OOB, vf = nstate == 1 ? lane0 : VOFF_OOB;
        const unsigned f0 = (unsigned)__builtin_amdgcn_raw_ptr_buffer_atomic_add_i32(1, rsl, vt, OFF_TK + peer * 128, 0);
        const unsigned f1 = __builtin_amdgcn_raw_buffer_load_b32(rsl, vf, OFF_RF + 4 * (nxt & 0xffffu), AUX_SYS);
        const unsigned f2 = __builtin_amdgcn_raw_buffer_load_b32(rsl, vf, OFF_LC + 16 * (nxt & 0xffffu), AUX_SC1);
        MEMFENCE();
        const int vr = nx.valid ? lane16 : VOFF_OOB, vq = (nx.valid && nx.did >= (unsigned)ID_P) ? lane16 : VOFF_OOB;
#pragma unroll
        for (int u = 0; u < PU; ++u) rn[u] = __builtin_amdgcn_raw_buffer_load_b128(rsp, vr, nx.src + u * 1024, AUX_SYS);
#pragma unroll
        for (int u = 0; u < PU; ++u) qn[u] = __builtin_amdgcn_raw_buffer_load_b128(rsl, vq, nx.dst + u * 1024, AUX_SC1);
        WAITV(2 * PU);
        if (pend) { __builtin_amdgcn_raw_buffer_store_b32(ptag, rsl, lane0, pend, AUX_SC1); pend = 0; }
        if (cu.valid) {
            if (cu.did >= (unsigned)ID_P) {
                const unsigned cb = (cu.did - ID_P) & 127u, e = ((cu.dst - OFF_DATA) & (BLOB - 1)) >> 10;
                const unsigned row = (cb >> 6) * 256 + (e >> 4) * 128 + ((cb >> 2) & 1) * 64 + ((e >> 2) & 3) * 16 + (e & 3);
                const unsigned col = peer * 2048 + ((cb >> 3) & 7) * 256 + (cb & 3) * 64;
                const unsigned so = row * (NG * 4) + col * 4;
#pragma unroll
                for (int u = 0; u < PU; ++u) {
                    f32x4 v = __builtin_bit_cast(f32x4, rc[u]) + __builtin_bit_cast(f32x4, qc[u]);
                    if (dead) v = (f32x4){__builtin_nanf(""), __builtin_nanf(""), __builtin_nanf(""), __builtin_nanf("")};
                    st128<AUX_SC1>(__builtin_bit_cast(u32x4, v), rso, lane_out, so + u * (NG * 4));
                }
            } else {
#pragma unroll
                for (int u = 0; u < PU; ++u) st128<AUX_SC1>(rc[u], rsl, lane16, cu.dst + u * 1024);
                if (cu.last) { pend = OFF_LC + 16 * cu.did + 4 * (((cu.dst - OFF_DATA) & (BLOB - 1)) / SLICE); ptag = cu.tag; }
            }
        }
        const unsigned g0 = uni(f0), g1 = uni(f1), g2 = uni(f2);
        if (nstate == 0) {
            const unsigned un = g0 - tk_base;
            if (un < N_UNITS) {
                nxt = uni(desc[un / SL]); nofs = (un % SL) * SLICE; ntag = tag0; nstate = dead ? 2 : 1;
            } else nstate = 3;
        } else if (nstate == 1) {
            if (g1 >= tag0 && ((nxt & 0xffffu) < (unsigned)ID_P || g2 >= tag0)) { ntag = g1; nstate = 2; }
            else if (!cu.valid && !nx.valid) {
                if ((++spins & 63u) == 0 && give_up(wsl, call, t0)) { dead = true; nstate = 2; }
                __builtin_amdgcn_s_sleep(2);
            }
        }
        return !cu.valid && !nx.valid && nstate == 3;
    };
    PStep sa = {0, 0, 0, 0, false, false}, sb;
    u32x4 ra[PU] = {}, qa[PU] = {}, rb[PU], qb[PU];
    bool done;
    do {
        step(sa, ra, qa, sb, rb, qb);
        done = step(sb, rb, qb, sa, ra, qa);
    } while (!done);
}

__global__ __launch_bounds__(512, 2) void gemm_rs_fused(const Params p) {
    extern __shared__ __attribute__((aligned(16))) char shm[];
    LAS char* lds = (LAS char*)shm;
    const int b = blockIdx.x, tid = threadIdx.x, lane = tid & 63, wid = __builtin_amdgcn_readfirstlane(tid >> 6);
    const int rank = p.rank;
    char* wsl = p.ws[rank];
    const rsrc_t rsl = mk_rsrc(wsl, (unsigned)WS_NEED);
    const rsrc_t rso = mk_rsrc(p.out, OUT_BYTES);
    STAMP(0);

    if (b >= NGEMM && b < NGEMM + NPULL) {
        const unsigned call = uni(ld_agent(wsl + OFF_CTR + b * 128)) + 1u;
        const int pi = b - NGEMM, ds = 1 + pi % (NR - 1), peer = (rank + ds) & (NR - 1);
        const int pb = ID_PAR + (int)(call & 1u) * PAR_SZ;
        const rsrc_t rsp = mk_rsrc(p.ws[peer], (unsigned)WS_NEED);
        LAS unsigned* desc = (LAS unsigned*)lds;
        if (tid < N_ITEMS) {
            unsigned did, sid;
            if (tid < N_IN) {
                const unsigned code = p.order[tid];
                const unsigned tt = code & 7u, a = (code >> 3) & 31u;
                if (code >> 8) { did = ID_WR + a * 64 + peer * 8 + tt; sid = pb + PO_W + rank * 64 + a * 8 + tt; }
                else {
                    unsigned d = a >> 1;
                    if (d >= (unsigned)ds) ++d;
                    const unsigned i = (((rank + d) & (NR - 1)) << 1) | (a & 1u);
                    did = ID_XR + i * 64 + peer * 8 + tt; sid = pb + PO_X + i * 8 + tt;
                }
            } else {
                const unsigned cb = tid - N_IN;
                did = ID_P + peer * 128 + cb; sid = pb + PO_C + rank * 128 + cb;
            }
            desc[tid] = (sid << 16) | did;
        }
        asm volatile("s_waitcnt vmcnt(0) lgkmcnt(0)" ::: "memory");
        __builtin_amdgcn_s_barrier();
        MEMFENCE();
        if (tid == 0) st_agent(wsl + OFF_CTR + b * 128, call);
        if (wid >= PW) return;
        pull_items(wsl, rsl, rsp, rso, desc, call, lane, peer);
        STAMP(2);
        return;
    }

    unsigned call;
    int pb;
    {
        const int ci = b < NGEMM ? b : b - NPULL;
        static_assert(2 * NCONV <= 640 && 3 * NCONV >= 640 && NPT * 80 <= NCONV && NPT <= 2, "three blobs per converter at most, two at least");
        const int n2 = ci + 2 * NCONV < 640 ? ci + 2 * NCONV : ci;
        f32x4 v0[8], v1[8], v2[8];
        const unsigned c0 = ld_agent(wsl + OFF_CTR + b * 128);
        char* wst = p.ws[0];
#pragma unroll
        for (int r = 1; r < NR; ++r) wst = (tid >> 2) == r ? p.ws[r] : wst;
        LAS unsigned* cls = (LAS unsigned*)lds;
        MEMFENCE();
        conv_load(p, ci, wid, lane, v0);
        WAITV(0);
        call = uni(c0) + 1u;
        pb = ID_PAR + (int)(call & 1u) * PAR_SZ;
        const unsigned k0 = conv_class(v0);
        conv_store<true>(p, rsl, ci, pb, wid, lane, v0, k0);
        if (lane == 0) cls[wid] = k0;
        STAMP(5);
        MEMFENCE();
        conv_load(p, ci + NCONV, wid, lane, v1);
        conv_load(p, n2, wid, lane, v2);
        asm volatile("s_waitcnt lgkmcnt(0)" ::: "memory");
        WAITV(16);
        __builtin_amdgcn_s_barrier();
        MEMFENCE();
        conv_publish(rank, wst, ci, pb, (call << 8) | conv_agree<true>(p, rsl, ci, pb, wid, lane, v0, k0, cls));
        STAMP(6);
        if (tid == 0) st_agent(wsl + OFF_CTR + b * 128, call);
        WAITV(0);
        const unsigned k1 = conv_class(v1), k2 = conv_class(v2);
        conv_store<false>(p, rsl, ci + NCONV, pb, wid, lane, v1, k1);
        if (n2 != ci) conv_store<false>(p, rsl, n2, pb, wid, lane, v2, k2);
        if (lane == 0) { cls[8 + wid] = k1; cls[16 + wid] = k2; }
        STAMP(7);
        asm volatile("s_waitcnt lgkmcnt(0)" ::: "memory");
        WAITV(0);
        __builtin_amdgcn_s_barrier();
        MEMFENCE();
        conv_publish(rank, wst, ci + NCONV, pb, (call << 8) | conv_agree<false>(p, rsl, ci + NCONV, pb, wid, lane, v1, k1, cls + 8));
        if (n2 != ci) conv_publish(rank, wst, n2, pb, (call << 8) | conv_agree<false>(p, rsl, n2, pb, wid, lane, v2, k2, cls + 16));
        asm volatile("s_waitcnt lgkmcnt(0)" ::: "memory");
        __builtin_amdgcn_s_barrier();
        MEMFENCE();
    }
    STAMP(1);

    const unsigned tag0 = call << 8;
    const int wr = wid >> 2, wc = wid & 3;
    for (int job = 0;; ++job) {
        int nsteps, cid, fl_rank;
        unsigned ids, fl_idx, out_off = 0;
        if (b < NGEMM) {
            if (job) break;
            const int xcd = b & 7, kk = b >> 3, i = (xcd >> 1) * 4 + (kk & 3), j = (xcd & 1) * 4 + (kk >> 2), own = i >> 1;
            const int nrem = own == rank ? 7 : 6;
            nsteps = 8 + 8 * nrem;
            const int m = (lane < nsteps ? lane : 0) - 8;
            if (m < 0) ids = (unsigned)(pb + PO_X + i * 8 + m + 8) | ((unsigned)(pb + PO_W + rank * 64 + j * 8 + m + 8) << 16);
            else {
                const int tt = m / nrem; int d = m % nrem + 1;
                if (own != rank && d >= ((own - rank) & (NR - 1))) ++d;
                const int s = (rank + d) & (NR - 1);
                ids = (unsigned)(ID_XR + i * 64 + s * 8 + tt) | ((unsigned)(tt < NPT ? pb + PO_WP + (j * 8 + s) * 2 + tt : ID_WR + j * 64 + s * 8 + tt) << 16);
            }
            const int cb = (i & 1) * 64 + j * 8;
            cid = own == rank ? -1 : pb + PO_C + own * 128 + cb;
            fl_rank = own; fl_idx = ID_P + rank * 128 + cb;
            out_off = (unsigned)((i & 1) * 256) * (NG * 4) + (unsigned)(rank * 2048 + j * 256) * 4;
        } else {
            const int pt = (b - NGEMM - NPULL) + job * NHELP;
            if (pt >= 112) break;
            const int h = pt / 56, rem = pt % 56, q = (rank + 1 + (rem >> 3)) & (NR - 1), j = rem & 7, m = lane & 7;
            nsteps = 8;
            ids = (unsigned)(pb + PO_X + (rank * 2 + h) * 8 + m) | ((unsigned)(pb + PO_W + q * 64 + j * 8 + m) << 16);
            cid = ID_P + q * 128 + h * 64 + j * 8;
            fl_rank = rank; fl_idx = cid;
        }

        u64 mask = 0;
        unsigned cl = 0;
        int umax = -64;
        auto fetch = [&](int n, int uprev, int& unit) -> unsigned {
            if (!((mask >> n) & 1ull)) {
                const u64 t0 = __builtin_amdgcn_s_memrealtime();
                const int vo = (unsigned)(lane - n) < 16u ? 0 : VOFF_OOB;
                for (unsigned spins = 1;; ++spins) {
                    MEMFENCE();
                    const u32x4 fa = __builtin_amdgcn_raw_buffer_load_b128(rsl, 16 * (ids & 0xffffu) + vo, OFF_LC, AUX_SC1);
                    const u32x4 fb = __builtin_amdgcn_raw_buffer_load_b128(rsl, 16 * (ids >> 16) + vo, OFF_LC, AUX_SC1);
                    const bool ok = min(min(min(fa[0], fa[1]), min(fa[2], fa[3])), min(min(fb[0], fb[1]), min(fb[2], fb[3]))) >= tag0;
                    if (ok) cl = (fa[0] & 255u) + (fb[0] & 255u);
                    mask |= __ballot(ok);
                    if ((mask >> n) & 1ull) break;
                    if ((spins & 63u) == 0 && give_up(wsl, call, t0)) { mask = ~0ull; break; }
                    __builtin_amdgcn_s_sleep(8);
                }
                MEMFENCE();
            }
            const int u = __builtin_amdgcn_readlane((int)cl, n);
            const bool tiny = u < umax - 5;
            unit = tiny ? uprev : u;
            umax = max(umax, u);
            return tiny ? (unsigned)ID_ZERO * 0x10001u : (unsigned)__builtin_amdgcn_readlane((int)ids, n);
        };
        const int lane16 = lane * 16;
        const unsigned wofs = OFF_DATA + wid * 2048;
#define SA(bf, h) (((bf) * 2 + (h)) * 16384)
#define SB(bf, h) ((4 + (bf) * 2 + (h)) * 16384)
#define STAGE(bufoff, goff) do { \
        __builtin_amdgcn_raw_ptr_buffer_load_lds(rsl, (LAS void*)(lds + (bufoff) + wid * 2048), 16, lane16, (goff), 0, AUX_SC1); \
        __builtin_amdgcn_raw_ptr_buffer_load_lds(rsl, (LAS void*)(lds + (bufoff) + wid * 2048), 16, lane16, (goff), 1024, AUX_SC1); } while (0)
#define LDA(dst, bf, h) do { _Pragma("unroll") for (int m = 0; m < 4; ++m) _Pragma("unroll") for (int k = 0; k < 2; ++k) \
        dst[m][k] = *(const LAS h8*)(lds + SA(bf, h) + aoff + m * 2048 + k * 1024); } while (0)
#define LDB(dst, bf, h) do { _Pragma("unroll") for (int n = 0; n < 2; ++n) _Pragma("unroll") for (int k = 0; k < 2; ++k) \
        dst[n][k] = *(const LAS h8*)(lds + SB(bf, h) + boff + n * 2048 + k * 1024); } while (0)
#define MMA(ai, bj, At, Bt) do { __builtin_amdgcn_s_setprio(1); \
        _Pragma("unroll") for (int m = 0; m < 4; ++m) _Pragma("unroll") for (int n = 0; n < 2; ++n) _Pragma("unroll") for (int k = 0; k < 2; ++k) \
            acc[ai][bj][m][n] = __builtin_amdgcn_mfma_f32_16x16x32_f16(At[m][k], Bt[n][k], acc[ai][bj][m][n], 0, 0, 0); \
        __builtin_amdgcn_s_setprio(0); } while (0)
#define WAIT_V(n) asm volatile("s_waitcnt vmcnt(" #n ")" ::: "memory")
#define WAIT_L(n) asm volatile("s_waitcnt lgkmcnt(" #n ")" ::: "memory")
#define BAR __builtin_amdgcn_s_barrier()
#define SCHED __builtin_amdgcn_sched_barrier(0)
#define RESCALE(from, to) do { if ((from) != (to)) { const float f_ = __builtin_ldexpf(1.0f, 16 * ((from) - (to))); \
        _Pragma("unroll") for (int a_ = 0; a_ < 2; ++a_) _Pragma("unroll") for (int c_ = 0; c_ < 2; ++c_) _Pragma("unroll") for (int m_ = 0; m_ < 4; ++m_) \
            _Pragma("unroll") for (int n_ = 0; n_ < 2; ++n_) acc[a_][c_][m_][n_] *= f_; } } while (0)
#define OFFA(c) (wofs + ((c) & 0xffffu) * BLOB)
#define OFFB(c) (wofs + ((c) >> 16) * BLOB)
        const int aoff = wr * 8192 + lane16, boff = wc * 4096 + lane16;
        f32x4 acc[2][2][4][2];
#pragma unroll
        for (int a = 0; a < 2; ++a)
#pragma unroll
            for (int c = 0; c < 2; ++c)
#pragma unroll
                for (int m = 0; m < 4; ++m)
#pragma unroll
                    for (int n = 0; n < 2; ++n) acc[a][c][m][n] = (f32x4){0.f, 0.f, 0.f, 0.f};
        h8 At[4][2], B0[2][2], B1[2][2];
        unsigned a1;
        int ucur, u0, u1;
        {
            const unsigned c0 = fetch(0, 0, u0), c1 = fetch(1, u0, u1);
            ucur = u0;
            STAGE(SB(0, 0), OFFB(c0)); STAGE(SA(0, 0), OFFA(c0)); STAGE(SB(0, 1), OFFB(c0) + 16384); STAGE(SA(0, 1), OFFA(c0) + 16384);
            if (wr == 1) BAR;
            WAIT_V(4); BAR;
            STAGE(SB(1, 0), OFFB(c1)); STAGE(SA(1, 0), OFFA(c1)); STAGE(SB(1, 1), OFFB(c1) + 16384);
            WAIT_V(6); BAR;
            a1 = OFFA(c1);
        }
        STAMP(2);
        for (int t = 0; t < nsteps; t += 2) {
            const int t2 = t + 2 < nsteps ? t + 2 : 0;
            int u2, u3;
            RESCALE(ucur, u0);
            LDB(B0, 0, 0); SCHED; LDA(At, 0, 0); STAGE(SA(1, 1), a1 + 16384);
            WAIT_L(8); BAR; WAIT_L(0); MMA(0, 0, At, B0); BAR; SCHED;
            const unsigned c2 = fetch(t2, u1, u2), a2 = OFFA(c2), b2 = OFFB(c2);
            LDB(B1, 0, 1); STAGE(SB(0, 0), b2);
            BAR; WAIT_L(0); MMA(0, 1, At, B1); BAR;
            LDA(At, 0, 1); STAGE(SA(0, 0), a2);
            BAR; WAIT_L(0); MMA(1, 0, At, B0); BAR; SCHED;
            STAGE(SB(0, 1), b2 + 16384);
            WAIT_V(6); BAR; MMA(1, 1, At, B1); BAR;
            RESCALE(u0, u1);
            LDB(B0, 1, 0); SCHED; LDA(At, 1, 0); STAGE(SA(0, 1), a2 + 16384);
            WAIT_L(8); BAR; WAIT_L(0); MMA(0, 0, At, B0); BAR; SCHED;
            const unsigned c3 = fetch(t2 + 1, u2, u3), a3 = OFFA(c3), b3 = OFFB(c3);
            LDB(B1, 1, 1); STAGE(SB(1, 0), b3);
            BAR; WAIT_L(0); MMA(0, 1, At, B1); BAR;
            LDA(At, 1, 1); STAGE(SA(1, 0), a3);
            BAR; WAIT_L(0); MMA(1, 0, At, B0); BAR; SCHED;
            STAGE(SB(1, 1), b3 + 16384);
            WAIT_V(6); BAR; MMA(1, 1, At, B1); BAR;
            a1 = a3; ucur = u1; u0 = u2; u1 = u3;
        }
        STAMP(3);
        const bool dead = uni(ld_agent(wsl + OFF_TMO)) == call;
        const float qn = __builtin_nanf("");
        const int sh = 16 * (ucur - 16);
        if (cid >= 0) {
            const unsigned base = OFF_DATA + (unsigned)(cid + wid) * BLOB;
#pragma unroll
            for (int ai = 0; ai < 2; ++ai)
#pragma unroll
                for (int m = 0; m < 4; ++m)
#pragma unroll
                    for (int r = 0; r < 4; ++r) {
                        f32x4 v = {__builtin_ldexpf(acc[ai][0][m][0][r], sh), __builtin_ldexpf(acc[ai][0][m][1][r], sh),
                                   __builtin_ldexpf(acc[ai][1][m][0][r], sh), __builtin_ldexpf(acc[ai][1][m][1][r], sh)};
                        if (dead) v = (f32x4){qn, qn, qn, qn};
                        st128<AUX_SYS>(__builtin_bit_cast(u32x4, v), rsl, lane16, base + (ai * 16 + m * 4 + r) * 1024);
                    }
            WAIT_V(0);
            if (lane == 0) {
                if (fl_rank == rank) st_agent(wsl + OFF_LC + 16 * (fl_idx + wid), tag0);
                else if ((unsigned)fl_rank < (unsigned)NR) st_sys(p.ws[fl_rank] + off_rf(call) + 4 * (fl_idx + wid), tag0);
            }
        } else {
            const int lane_out = (lane >> 4) * 4 * (NG * 4) + (lane & 15) * 16;
            const unsigned base = out_off + (unsigned)(wr * 64) * (NG * 4) + wc * 256;
#pragma unroll
            for (int ai = 0; ai < 2; ++ai)
#pragma unroll
                for (int m = 0; m < 4; ++m)
#pragma unroll
                    for (int r = 0; r < 4; ++r) {
                        f32x4 v = {__builtin_ldexpf(acc[ai][0][m][0][r], sh), __builtin_ldexpf(acc[ai][0][m][1][r], sh),
                                   __builtin_ldexpf(acc[ai][1][m][0][r], sh), __builtin_ldexpf(acc[ai][1][m][1][r], sh)};
                        if (dead) v = (f32x4){qn, qn, qn, qn};
                        st128<AUX_SC1>(__builtin_bit_cast(u32x4, v), rso, lane_out, base + (unsigned)(ai * 128 + m * 16 + r) * (NG * 4));
                    }
            WAIT_V(0);
        }
        STAMP(4);
        if (wr == 0) BAR;
        BAR;
    }
}

}

extern "C" void kernel_launch(int rank, int nranks, ncclComm_t comm,
                              hipStream_t stream,
                              void* const* d_in, const int* in_sizes, int n_in,
                              void* d_out, int out_size,
                              const int* in_dtypes, int out_dtype,
                              void* const* d_out_all,
                              void* const* d_ws_all, size_t ws_bytes) {
    (void)comm; (void)in_sizes; (void)n_in; (void)in_dtypes; (void)out_dtype; (void)d_out_all;
    if (nranks != NR || (unsigned)rank >= (unsigned)NR || ws_bytes < WS_NEED || (size_t)out_size * 4 != OUT_BYTES) {
        fprintf(stderr, "kernel_launch: unexpected configuration\n");
        return;
    }
    Params p;
    p.x = (const float*)d_in[0];
    p.w = (const float*)d_in[1];
    p.out = (float*)d_out;
    for (int r = 0; r < NR; ++r) p.ws[r] = (char*)d_ws_all[r];
    p.rank = rank;
    int n = 0;
    for (int tt = 0; tt < 8; ++tt) {
        if (tt >= NPT) for (int j = 0; j < 8; ++j) p.order[n++] = (unsigned short)(0x100 | (j << 3) | tt);
        for (int u = 1; u < 7; ++u) p.order[n++] = (unsigned short)(((2 * u) << 3) | tt);
        if (tt < 8 - CFG_D2) for (int u = 1; u < 7; ++u) p.order[n++] = (unsigned short)(((2 * u + 1) << 3) | tt);
        if (tt < 8 - CFG_D3) for (int h = 0; h < 2; ++h) p.order[n++] = (unsigned short)((h << 3) | tt);
    }
    for (int tt = 0; tt < 8; ++tt) {
        if (tt >= 8 - CFG_D2) for (int u = 1; u < 7; ++u) p.order[n++] = (unsigned short)(((2 * u + 1) << 3) | tt);
        if (tt >= 8 - CFG_D3) for (int h = 0; h < 2; ++h) p.order[n++] = (unsigned short)((h << 3) | tt);
    }
    HIP_CHECK(hipFuncSetAttribute((const void*)gemm_rs_fused, hipFuncAttributeMaxDynamicSharedMemorySize, LDS_BYTES));
    hipLaunchKernelGGL(gemm_rs_fused, dim3(GRID), dim3(512), LDS_BYTES, stream, p);
    HIP_CHECK(hipGetLastError());
}
```

```cpp
#include <hip/hip_runtime.h>
#include <rccl/rccl.h>
#include <cstdio>
#include <cstdint>

#define HIP_CHECK(expr) do { hipError_t _e = (expr); if (_e != hipSuccess) { \
    fprintf(stderr, "HIP error %s at %s:%d\n", hipGetErrorString(_e), __FILE__, __LINE__); \
    return; } } while (0)

namespace {

#define LAS __attribute__((address_space(3)))
typedef _Float16 h8 __attribute__((ext_vector_type(8)));
typedef float f32x4 __attribute__((ext_vector_type(4)));
typedef unsigned u32x4 __attribute__((ext_vector_type(4)));
typedef __amdgpu_buffer_rsrc_t rsrc_t;
typedef unsigned long long u64;

constexpr int NR = 8;
constexpr int KSH = 512;
constexpr int NG = 16384;
constexpr int BLOB = 32768;
#define CFG_NW 6
#define CFG_PW 5
#define CFG_PU 2
#define CFG_NPT 1
#define CFG_SLEEP 48
#define CFG_FINE 16
#define CFG_D2 5
#define CFG_D3 5
constexpr int NW = CFG_NW;
constexpr int PW = CFG_PW;
constexpr int PU = CFG_PU;
constexpr int SL = 4;
constexpr int SLICE = BLOB / SL;
constexpr int SPI = SLICE / (PU * 1024);
constexpr int GRID = 256, NGEMM = 128, NPULL = (NR - 1) * NW, NCONV = GRID - NPULL, NHELP = GRID - NGEMM - NPULL;
constexpr int NPT = CFG_NPT;
constexpr int N_IN = 14 * 8 + 8 * (8 - NPT);
constexpr int N_OUT = 128;
constexpr int N_ITEMS = N_IN + N_OUT;
constexpr unsigned N_FINE = CFG_FINE;
constexpr unsigned N_FULL = (N_ITEMS - N_FINE) * SL, N_UNITS = N_FULL + N_FINE * SL * 2;
constexpr unsigned TICKETS = N_UNITS + NW * PW;
static_assert(4 % PU == 0 && SPI * PU * 1024 * SL == BLOB && SPI >= 2 && SPI % 2 == 0 && N_FINE <= N_OUT, "puller geometry");
static_assert(NHELP > 0 && N_ITEMS <= 512, "need helper workgroups");

constexpr int ID_XR = 0;
constexpr int ID_WR = 1024;
constexpr int ID_P = 1536;
constexpr int ID_PAR = 2560;
constexpr int PO_X = 0, PO_W = 128, PO_C = 640, PO_WP = 1664  , PAR_SZ = 1792;
constexpr int ID_ZERO = ID_PAR + 2 * PAR_SZ;
constexpr int N_IDS = ID_ZERO + 1;
constexpr unsigned OFF_CTR = 0;
constexpr unsigned OFF_TMO = 32768;
constexpr unsigned OFF_TK = 36864;
constexpr unsigned OFF_RF0 = 65536;
constexpr unsigned OFF_LC = 131072;
constexpr unsigned OFF_DBG = 262144;
constexpr unsigned OFF_DATA = 1u << 20;
static_assert(OFF_LC + 16u * N_IDS <= OFF_DBG && OFF_RF0 + 8u * ID_PAR <= OFF_LC, "control layout");
constexpr size_t WS_NEED = (size_t)OFF_DATA + (size_t)N_IDS * BLOB;
constexpr unsigned OUT_BYTES = 512u * NG * 4u;
constexpr u64 DEADLINE = 50000000ull;
constexpr int AUX_SC1 = 16, AUX_SYS = 17;
constexpr int LDS_BYTES = 131072;
constexpr int VOFF_OOB = 0x70000000;

struct Params {
    const float* x; const float* w; float* out;
    char* ws[NR];
    int rank;
    unsigned short order[N_IN];
};

#define MEMFENCE() asm volatile("" ::: "memory")
#define WAITV_IMM(n) (((n) & 15) | (7 << 4) | (15 << 8) | (((n) >> 4) << 14))
#define WAITV(n) do { MEMFENCE(); __builtin_amdgcn_s_waitcnt(WAITV_IMM(n)); MEMFENCE(); } while (0)

#define STAMP(slot) do {} while (0)

__device__ __forceinline__ unsigned ld_agent(const void* p) { return __hip_atomic_load((const unsigned*)p, __ATOMIC_RELAXED, __HIP_MEMORY_SCOPE_AGENT); }
__device__ __forceinline__ unsigned ld_sys(const void* p) { return __hip_atomic_load((const unsigned*)p, __ATOMIC_RELAXED, __HIP_MEMORY_SCOPE_SYSTEM); }
__device__ __forceinline__ void st_agent(void* p, unsigned v) { __hip_atomic_store((unsigned*)p, v, __ATOMIC_RELAXED, __HIP_MEMORY_SCOPE_AGENT); }
__device__ __forceinline__ void st_sys(void* p, unsigned v) { __hip_atomic_store((unsigned*)p, v, __ATOMIC_RELAXED, __HIP_MEMORY_SCOPE_SYSTEM); }
__device__ __forceinline__ rsrc_t mk_rsrc(const void* p, unsigned bytes) { return __builtin_amdgcn_make_buffer_rsrc((void*)p, 0, bytes, 0x00020000); }
template <int AUX> __device__ __forceinline__ void st128(u32x4 v, rsrc_t rs, int voff, unsigned soff) {
    __builtin_amdgcn_raw_buffer_store_b128(v, rs, voff, soff, AUX);
    asm volatile("s_nop 1" :: "v"(v));
}
__device__ __forceinline__ unsigned off_rf(unsigned call) { return OFF_RF0 + (call & 1u) * (4u * ID_PAR); }
__device__ __forceinline__ unsigned uni(unsigned v) { return (unsigned)__builtin_amdgcn_readfirstlane((int)v); }

__device__ __forceinline__ bool give_up(char* wsl, unsigned call, u64 t0) {
    if (uni(ld_agent(wsl + OFF_TMO)) == call || __builtin_amdgcn_s_memrealtime() - t0 > DEADLINE) {
        st_agent(wsl + OFF_TMO, call);
        return true;
    }
    return false;
}

#define CONV_LD(p) (*(p))
__device__ __forceinline__ void conv_load(const Params& p, int n, int wid, int lane, f32x4 (&v)[8]) {
    const int tt = n / 80, r = n % 80, g = lane >> 4, c = lane & 15;
    if (r < 16) {
        const float* src = p.x + (size_t)(r * 256 + wid * 32 + c) * KSH + tt * 64 + 4 * g;
#pragma unroll
        for (int f = 0; f < 2; ++f)
#pragma unroll
            for (int ii = 0; ii < 4; ++ii) v[f * 4 + ii] = CONV_LD((const f32x4*)(src + f * 16 * KSH + ii * 16));
    } else {
        const int ks = wid & 1, wc = wid >> 1;
        const float* src = p.w + (size_t)(tt * 64 + ks * 32 + 4 * g) * NG + (r - 16) * 256 + wc * 64 + 4 * c;
#pragma unroll
        for (int e = 0; e < 8; ++e) v[e] = CONV_LD((const f32x4*)(src + (size_t)((e >> 2) * 16 + (e & 3)) * NG));
    }
}
__device__ __forceinline__ unsigned conv_class(const f32x4 (&v)[8]) {
    float m = 0.f;
#pragma unroll
    for (int e = 0; e < 8; ++e)
#pragma unroll
        for (int j = 0; j < 4; ++j) m = __builtin_fmaxf(m, __builtin_fabsf(v[e][j]));
    const unsigned c = ((__builtin_bit_cast(unsigned, m) >> 23) + 3u) >> 4;
    unsigned c0 = uni(c);
    for (;;) {
        const u64 up = __ballot(c > c0);
        if (!up) break;
        c0 = (unsigned)__builtin_amdgcn_readlane((int)c, __builtin_ctzll(up));
    }
    return c0;
}
template <bool MAY_PUSH>
__device__ __forceinline__ void conv_store(const Params& p, rsrc_t rsl, int n, int pb, int wid, int lane, const f32x4 (&v)[8], unsigned cls) {
    const int tt = n / 80, r = n % 80, sh = 128 - 16 * (int)cls;
    if (r < 16) {
        const unsigned base = OFF_DATA + (unsigned)(pb + PO_X + r * 8 + tt) * BLOB + wid * 4096;
#pragma unroll
        for (int f = 0; f < 2; ++f)
#pragma unroll
            for (int ks = 0; ks < 2; ++ks) {
                h8 o;
#pragma unroll
                for (int e = 0; e < 8; ++e) o[e] = (_Float16)__builtin_ldexpf(v[f * 4 + ks * 2 + (e >> 2)][e & 3], sh);
                st128<AUX_SYS>(__builtin_bit_cast(u32x4, o), rsl, lane * 16, base + f * 2048 + ks * 1024);
            }
    } else {
        const int ks = wid & 1, wc = wid >> 1;
        const unsigned inb = wc * 4096 + ks * 1024, base = OFF_DATA + (unsigned)(pb + PO_W + (r - 16) * 8 + tt) * BLOB + inb;
        h8 o[4];
#pragma unroll
        for (int f = 0; f < 4; ++f)
#pragma unroll
            for (int e = 0; e < 8; ++e) o[f][e] = (_Float16)__builtin_ldexpf(v[e][f], sh);
        const int q = ((r - 16) >> 3) & (NR - 1);
        if (MAY_PUSH && tt < NPT && q != p.rank) {
            const rsrc_t rsq = mk_rsrc(p.ws[q], (unsigned)WS_NEED);
            const unsigned rb = OFF_DATA + (unsigned)(pb + PO_WP + (((r - 16) & 7) * 8 + p.rank) * 2 + tt) * BLOB + inb;
#pragma unroll
            for (int f = 0; f < 4; ++f) st128<AUX_SYS>(__builtin_bit_cast(u32x4, o[f]), rsq, lane * 16, rb + (f >> 1) * 16384 + (f & 1) * 2048);
        }
#pragma unroll
        for (int f = 0; f < 4; ++f) st128<AUX_SYS>(__builtin_bit_cast(u32x4, o[f]), rsl, lane * 16, base + (f >> 1) * 16384 + (f & 1) * 2048);
    }
}
template <bool MAY_PUSH>
__device__ __forceinline__ unsigned conv_agree(const Params& p, rsrc_t rsl, int n, int pb, int wid, int lane, const f32x4 (&v)[8], unsigned mine,
                                               const LAS unsigned* cls) {
    const u32x4 a = *(const LAS u32x4*)cls, b = *(const LAS u32x4*)(cls + 4);
    const unsigned hi = uni(max(max(max(a[0], a[1]), max(a[2], a[3])), max(max(b[0], b[1]), max(b[2], b[3]))));
    const unsigned lo = uni(min(min(min(a[0], a[1]), min(a[2], a[3])), min(min(b[0], b[1]), min(b[2], b[3]))));
    if (lo != hi) {
        if (mine != hi) conv_store<MAY_PUSH>(p, rsl, n, pb, wid, lane, v, hi);
        WAITV(0);
        __builtin_amdgcn_s_barrier();
        MEMFENCE();
    }
    return hi;
}
__device__ __forceinline__ void conv_publish(int rank, char* wst  , int n, int pb, unsigned tag) {
    const int tt = n / 80, r = n % 80, t = threadIdx.x >> 2, wd = threadIdx.x & 3;
    const unsigned OFF_RF = off_rf(tag >> 8);
    if (t < NR) {
        if (t == rank) st_agent(wst + OFF_LC + 16 * (pb + (r < 16 ? PO_X + r * 8 : PO_W + (r - 16) * 8) + tt) + 4 * wd, tag);
        else if (r < 16) { if ((r >> 1) != rank && wd == 0) st_sys(wst + OFF_RF + 4 * (ID_XR + r * 64 + rank * 8 + tt), tag); }
        else if (((r - 16) >> 3) == t) {
            if (tt < NPT) st_sys(wst + OFF_LC + 16 * (pb + PO_WP + (((r - 16) & 7) * 8 + rank) * 2 + tt) + 4 * wd, tag);
            else if (wd == 0) st_sys(wst + OFF_RF + 4 * (ID_WR + ((r - 16) & 7) * 64 + rank * 8 + tt), tag);
        }
    }
}

struct PStep { unsigned src, dst, did, tag; bool valid, last; };

__device__ __forceinline__ void pull_items(char* wsl, rsrc_t rsl, rsrc_t rsp, rsrc_t rso, const LAS unsigned* desc  ,
                                           unsigned call, int lane, int peer) {
    const int lane16 = lane * 16, lane_out = (lane >> 4) * 4 * (NG * 4) + (lane & 15) * 16;
    const u64 t0 = __builtin_amdgcn_s_memrealtime();
    const int lane0 = lane == 0 ? 0 : VOFF_OOB;
    const unsigned tk_base = (call - 1u) * TICKETS;
    bool dead = false;
    int sub = SPI;
    int nsub = 0;
    int nstate = 0;
    unsigned cur = 0, cofs = 0, ctag = 0, nxt = 0, nofs = 0, ntag = 0, pend = 0, ptag = 0, spins = 0;
    const unsigned tag0 = call << 8, OFF_RF = off_rf(call);
    auto step = [&](const PStep& cu, const u32x4 (&rc)[PU], const u32x4 (&qc)[PU], PStep& nx, u32x4 (&rn)[PU], u32x4 (&qn)[PU]) -> bool {
        MEMFENCE();
        if (sub == SPI && nstate == 2) { cur = nxt; cofs = nofs; ctag = ntag; sub = nsub; nstate = 0; }
        nx.valid = sub < SPI;
        const unsigned inb = cofs + (nx.valid ? sub : SPI - 1) * (PU * 1024);
        nx.did = cur & 0xffffu;
        nx.tag = ctag;
        nx.src = OFF_DATA + (cur >> 16) * BLOB + inb;
        nx.dst = OFF_DATA + nx.did * BLOB + inb;
        if (nx.valid) ++sub;
        nx.last = sub == SPI;
        const int vt = nstate == 0 ? lane0 : VOFF_OOB, vf = nstate == 1 ? lane0 : VOFF_OOB;
        const unsigned f0 = (unsigned)__builtin_amdgcn_raw_ptr_buffer_atomic_add_i32(1, rsl, vt, OFF_TK + peer * 128, 0);
        const unsigned f1 = __builtin_amdgcn_raw_buffer_load_b32(rsl, vf, OFF_RF + 4 * (nxt & 0xffffu), AUX_SYS);
        const unsigned f2 = __builtin_amdgcn_raw_buffer_load_b32(rsl, vf, OFF_LC + 16 * (nxt & 0xffffu), AUX_SC1);
        MEMFENCE();
        const int vr = nx.valid ? lane16 : VOFF_OOB, vq = (nx.valid && nx.did >= (unsigned)ID_P) ? lane16 : VOFF_OOB;
#pragma unroll
        for (int u = 0; u < PU; ++u) rn[u] = __builtin_amdgcn_raw_buffer_load_b128(rsp, vr, nx.src + u * 1024, AUX_SYS);
#pragma unroll
        for (int u = 0; u < PU; ++u) qn[u] = __builtin_amdgcn_raw_buffer_load_b128(rsl, vq, nx.dst + u * 1024, AUX_SC1);
        WAITV(2 * PU);
        if (pend) {
            __builtin_amdgcn_raw_buffer_store_b32(ptag, rsl, lane0, pend, AUX_SC1);
            pend = 0;
        }
        if (cu.valid) {
            if (cu.did >= (unsigned)ID_P) {
                const unsigned cb = (cu.did - ID_P) & 127u, e = ((cu.dst - OFF_DATA) & (BLOB - 1)) >> 10;
                const unsigned row = (cb >> 6) * 256 + (e >> 4) * 128 + ((cb >> 2) & 1) * 64 + ((e >> 2) & 3) * 16 + (e & 3);
                const unsigned col = peer * 2048 + ((cb >> 3) & 7) * 256 + (cb & 3) * 64;
                const unsigned so = row * (NG * 4) + col * 4;
#pragma unroll
                for (int u = 0; u < PU; ++u) {
                    f32x4 v = __builtin_bit_cast(f32x4, rc[u]) + __builtin_bit_cast(f32x4, qc[u]);
                    if (dead) v = (f32x4){__builtin_nanf(""), __builtin_nanf(""), __builtin_nanf(""), __builtin_nanf("")};
                    st128<AUX_SC1>(__builtin_bit_cast(u32x4, v), rso, lane_out, so + u * (NG * 4));
                }
            } else {
#pragma unroll
                for (int u = 0; u < PU; ++u) st128<AUX_SC1>(rc[u], rsl, lane16, cu.dst + u * 1024);
                if (cu.last) { pend = OFF_LC + 16 * cu.did + 4 * (((cu.dst - OFF_DATA) & (BLOB - 1)) / SLICE); ptag = cu.tag; }
            }
        }
        const unsigned g0 = uni(f0), g1 = uni(f1), g2 = uni(f2);
        if (nstate == 0) {
            const unsigned un = g0 - tk_base;
            if (un < N_UNITS) {
                const bool fine = un >= N_FULL;
                const unsigned hu = fine ? un + N_FULL : 2 * un;
                nxt = uni(desc[hu / (2 * SL)]);
                nsub = fine ? SPI / 2 : 0;
                nofs = (hu % (2 * SL)) * (SLICE / 2) - nsub * (PU * 1024);
                ntag = tag0; nstate = dead ? 2 : 1;
            } else nstate = 3;
        } else if (nstate == 1) {
            if (g1 >= tag0 && ((nxt & 0xffffu) < (unsigned)ID_P || g2 >= tag0)) { ntag = g1; nstate = 2; }
            else if (!cu.valid && !nx.valid) {
                if ((++spins & 63u) == 0 && give_up(wsl, call, t0)) { dead = true; nstate = 2; }
                __builtin_amdgcn_s_sleep(2);
            }
        }
        return !cu.valid && !nx.valid && nstate == 3;
    };
    PStep sa = {0, 0, 0, 0, false, false}, sb;
    u32x4 ra[PU] = {}, qa[PU] = {}, rb[PU], qb[PU];
    bool done;
    do {
        step(sa, ra, qa, sb, rb, qb);
        done = step(sb, rb, qb, sa, ra, qa);
    } while (!done);
}

__global__ __launch_bounds__(512, 2) void gemm_rs_fused(const Params p) {
    extern __shared__ __attribute__((aligned(16))) char shm[];
    LAS char* lds = (LAS char*)shm;
    const int b = blockIdx.x, tid = threadIdx.x, lane = tid & 63, wid = __builtin_amdgcn_readfirstlane(tid >> 6);
    const int rank = p.rank;
    char* wsl = p.ws[rank];
    const rsrc_t rsl = mk_rsrc(wsl, (unsigned)WS_NEED);
    const rsrc_t rso = mk_rsrc(p.out, OUT_BYTES);
    STAMP(0);

    if (b >= NGEMM && b < NGEMM + NPULL) {
        const unsigned call = uni(ld_agent(wsl + OFF_CTR + b * 128)) + 1u;
        const int pi = b - NGEMM, ds = 1 + pi % (NR - 1), peer = (rank + ds) & (NR - 1);
        const int pb = ID_PAR + (int)(call & 1u) * PAR_SZ;
        const rsrc_t rsp = mk_rsrc(p.ws[peer], (unsigned)WS_NEED);
        LAS unsigned* desc = (LAS unsigned*)lds;
        if (tid < N_ITEMS) {
            unsigned did, sid;
            if (tid < N_IN) {
                const unsigned code = p.order[tid];
                const unsigned tt = code & 7u, a = (code >> 3) & 31u;
                if (code >> 8) { did = ID_WR + a * 64 + peer * 8 + tt; sid = pb + PO_W + rank * 64 + a * 8 + tt; }
                else {
                    unsigned d = a >> 1;
                    if (d >= (unsigned)ds) ++d;
                    const unsigned i = (((rank + d) & (NR - 1)) << 1) | (a & 1u);
                    did = ID_XR + i * 64 + peer * 8 + tt; sid = pb + PO_X + i * 8 + tt;
                }
            } else {
                const unsigned cb = tid - N_IN;
                did = ID_P + peer * 128 + cb; sid = pb + PO_C + rank * 128 + cb;
            }
            desc[tid] = (sid << 16) | did;
        }
        asm volatile("s_waitcnt vmcnt(0) lgkmcnt(0)" ::: "memory");
        __builtin_amdgcn_s_barrier();
        MEMFENCE();
        if (tid == 0) st_agent(wsl + OFF_CTR + b * 128, call);
        if (wid >= PW) return;
        pull_items(wsl, rsl, rsp, rso, desc, call, lane, peer);
        STAMP(2);
        return;
    }

    unsigned call;
    int pb;
    {
        const int ci = b < NGEMM ? b : b - NPULL;
        static_assert(2 * NCONV <= 640 && 3 * NCONV >= 640 && NPT * 80 <= NCONV && NPT <= 2, "three blobs per converter at most, two at least");
        const int n2 = ci + 2 * NCONV < 640 ? ci + 2 * NCONV : ci;
        f32x4 v0[8], v1[8], v2[8];
        const unsigned c0 = ld_agent(wsl + OFF_CTR + b * 128);
        char* wst = p.ws[0];
#pragma unroll
        for (int r = 1; r < NR; ++r) wst = (tid >> 2) == r ? p.ws[r] : wst;
        LAS unsigned* cls = (LAS unsigned*)lds;
        MEMFENCE();
        if (ci >= 96) __builtin_amdgcn_s_sleep(CFG_SLEEP);
        conv_load(p, ci, wid, lane, v0);
        WAITV(0);
        call = uni(c0) + 1u;
        pb = ID_PAR + (int)(call & 1u) * PAR_SZ;
        const unsigned k0 = conv_class(v0);
        conv_store<true>(p, rsl, ci, pb, wid, lane, v0, k0);
        if (lane == 0) cls[wid] = k0;
        STAMP(5);
        MEMFENCE();
        asm volatile("s_waitcnt lgkmcnt(0)" ::: "memory");
        WAITV(0);
        STAMP(8);
        __builtin_amdgcn_s_barrier();
        MEMFENCE();
        conv_publish(rank, wst, ci, pb, (call << 8) | conv_agree<true>(p, rsl, ci, pb, wid, lane, v0, k0, cls));
        STAMP(6);
        MEMFENCE();
        conv_load(p, ci + NCONV, wid, lane, v1);
        conv_load(p, n2, wid, lane, v2);
        if (tid == 0) st_agent(wsl + OFF_CTR + b * 128, call);
        WAITV(0);
        const unsigned k1 = conv_class(v1), k2 = conv_class(v2);
        conv_store<false>(p, rsl, ci + NCONV, pb, wid, lane, v1, k1);
        if (n2 != ci) conv_store<false>(p, rsl, n2, pb, wid, lane, v2, k2);
        if (lane == 0) { cls[8 + wid] = k1; cls[16 + wid] = k2; }
        STAMP(7);
        asm volatile("s_waitcnt lgkmcnt(0)" ::: "memory");
        WAITV(0);
        __builtin_amdgcn_s_barrier();
        MEMFENCE();
        conv_publish(rank, wst, ci + NCONV, pb, (call << 8) | conv_agree<false>(p, rsl, ci + NCONV, pb, wid, lane, v1, k1, cls + 8));
        if (n2 != ci) conv_publish(rank, wst, n2, pb, (call << 8) | conv_agree<false>(p, rsl, n2, pb, wid, lane, v2, k2, cls + 16));
        asm volatile("s_waitcnt lgkmcnt(0)" ::: "memory");
        __builtin_amdgcn_s_barrier();
        MEMFENCE();
    }
    STAMP(1);

    const unsigned tag0 = call << 8;
    const int wr = wid >> 2, wc = wid & 3;
    for (int job = 0;; ++job) {
        int nsteps, cid, fl_rank;
        unsigned ids, fl_idx, out_off = 0;
        if (b < NGEMM) {
            if (job) break;
            const int xcd = b & 7, kk = b >> 3, i = (xcd >> 1) * 4 + (kk & 3), j = (xcd & 1) * 4 + (kk >> 2), own = i >> 1;
            const int nrem = own == rank ? 7 : 6;
            nsteps = 8 + 8 * nrem;
            const int m = (lane < nsteps ? lane : 0) - 8;
            if (m < 0) ids = (unsigned)(pb + PO_X + i * 8 + m + 8) | ((unsigned)(pb + PO_W + rank * 64 + j * 8 + m + 8) << 16);
            else {
                const int tt = m / nrem; int d = m % nrem + 1;
                if (own != rank && d >= ((own - rank) & (NR - 1))) ++d;
                const int s = (rank + d) & (NR - 1);
                ids = (unsigned)(ID_XR + i * 64 + s * 8 + tt) | ((unsigned)(tt < NPT ? pb + PO_WP + (j * 8 + s) * 2 + tt : ID_WR + j * 64 + s * 8 + tt) << 16);
            }
            const int cb = (i & 1) * 64 + j * 8;
            cid = own == rank ? -1 : pb + PO_C + own * 128 + cb;
            fl_rank = own; fl_idx = ID_P + rank * 128 + cb;
            out_off = (unsigned)((i & 1) * 256) * (NG * 4) + (unsigned)(rank * 2048 + j * 256) * 4;
        } else {
            const int pt = (b - NGEMM - NPULL) + job * NHELP;
            if (pt >= 112) break;
            const int h = pt / 56, rem = pt % 56, q = (rank + 1 + (rem >> 3)) & (NR - 1), j = rem & 7, m = lane & 7;
            nsteps = 8;
            ids = (unsigned)(pb + PO_X + (rank * 2 + h) * 8 + m) | ((unsigned)(pb + PO_W + q * 64 + j * 8 + m) << 16);
            cid = ID_P + q * 128 + h * 64 + j * 8;
            fl_rank = rank; fl_idx = cid;
        }

        u64 mask = 0;
        unsigned cl = 0;
        int umax = -64;
        auto fetch = [&](int n, int uprev, int& unit) -> unsigned {
            if (!((mask >> n) & 1ull)) {
                const u64 t0 = __builtin_amdgcn_s_memrealtime();
                const int vo = (unsigned)(lane - n) < 16u ? 0 : VOFF_OOB;
                for (unsigned spins = 1;; ++spins) {
                    MEMFENCE();
                    const u32x4 fa = __builtin_amdgcn_raw_buffer_load_b128(rsl, 16 * (ids & 0xffffu) + vo, OFF_LC, AUX_SC1);
                    const u32x4 fb = __builtin_amdgcn_raw_buffer_load_b128(rsl, 16 * (ids >> 16) + vo, OFF_LC, AUX_SC1);
                    const bool ok = min(min(min(fa[0], fa[1]), min(fa[2], fa[3])), min(min(fb[0], fb[1]), min(fb[2], fb[3]))) >= tag0;
                    if (ok) cl = (fa[0] & 255u) + (fb[0] & 255u);
                    mask |= __ballot(ok);
                    if ((mask >> n) & 1ull) break;
                    if ((spins & 63u) == 0 && give_up(wsl, call, t0)) { mask = ~0ull; break; }
                    __builtin_amdgcn_s_sleep(8);
                }
                MEMFENCE();
            }
            const int u = __builtin_amdgcn_readlane((int)cl, n);
            const bool tiny = u < umax - 5;
            unit = tiny ? uprev : u;
            umax = max(umax, u);
            return tiny ? (unsigned)ID_ZERO * 0x10001u : (unsigned)__builtin_amdgcn_readlane((int)ids, n);
        };
        const int lane16 = lane * 16;
        const unsigned wofs = OFF_DATA + wid * 2048;
#define SA(bf, h) (((bf) * 2 + (h)) * 16384)
#define SB(bf, h) ((4 + (bf) * 2 + (h)) * 16384)
#define STAGE(bufoff, goff) do { \
        __builtin_amdgcn_raw_ptr_buffer_load_lds(rsl, (LAS void*)(lds + (bufoff) + wid * 2048), 16, lane16, (goff), 0, AUX_SC1); \
        __builtin_amdgcn_raw_ptr_buffer_load_lds(rsl, (LAS void*)(lds + (bufoff) + wid * 2048), 16, lane16, (goff), 1024, AUX_SC1); } while (0)
#define LDA(dst, bf, h) do { _Pragma("unroll") for (int m = 0; m < 4; ++m) _Pragma("unroll") for (int k = 0; k < 2; ++k) \
        dst[m][k] = *(const LAS h8*)(lds + SA(bf, h) + aoff + m * 2048 + k * 1024); } while (0)
#define LDB(dst, bf, h) do { _Pragma("unroll") for (int n = 0; n < 2; ++n) _Pragma("unroll") for (int k = 0; k < 2; ++k) \
        dst[n][k] = *(const LAS h8*)(lds + SB(bf, h) + boff + n * 2048 + k * 1024); } while (0)
#define MMA(ai, bj, At, Bt) do { __builtin_amdgcn_s_setprio(1); \
        _Pragma("unroll") for (int m = 0; m < 4; ++m) _Pragma("unroll") for (int n = 0; n < 2; ++n) _Pragma("unroll") for (int k = 0; k < 2; ++k) \
            acc[ai][bj][m][n] = __builtin_amdgcn_mfma_f32_16x16x32_f16(At[m][k], Bt[n][k], acc[ai][bj][m][n], 0, 0, 0); \
        __builtin_amdgcn_s_setprio(0); } while (0)
#define WAIT_V(n) asm volatile("s_waitcnt vmcnt(" #n ")" ::: "memory")
#define WAIT_L(n) asm volatile("s_waitcnt lgkmcnt(" #n ")" ::: "memory")
#define BAR __builtin_amdgcn_s_barrier()
#define SCHED __builtin_amdgcn_sched_barrier(0)
#define RESCALE(from, to) do { if ((from) != (to)) { const float f_ = __builtin_ldexpf(1.0f, 16 * ((from) - (to))); \
        _Pragma("unroll") for (int a_ = 0; a_ < 2; ++a_) _Pragma("unroll") for (int c_ = 0; c_ < 2; ++c_) _Pragma("unroll") for (int m_ = 0; m_ < 4; ++m_) \
            _Pragma("unroll") for (int n_ = 0; n_ < 2; ++n_) acc[a_][c_][m_][n_] *= f_; } } while (0)
#define OFFA(c) (wofs + ((c) & 0xffffu) * BLOB)
#define OFFB(c) (wofs + ((c) >> 16) * BLOB)
        const int aoff = wr * 8192 + lane16, boff = wc * 4096 + lane16;
        f32x4 acc[2][2][4][2];
#pragma unroll
        for (int a = 0; a < 2; ++a)
#pragma unroll
            for (int c = 0; c < 2; ++c)
#pragma unroll
                for (int m = 0; m < 4; ++m)
#pragma unroll
                    for (int n = 0; n < 2; ++n) acc[a][c][m][n] = (f32x4){0.f, 0.f, 0.f, 0.f};
        h8 At[4][2], B0[2][2], B1[2][2];
        unsigned a1;
        int ucur, u0, u1;
        {
            const unsigned c0 = fetch(0, 0, u0), c1 = fetch(1, u0, u1);
            ucur = u0;
            STAGE(SB(0, 0), OFFB(c0)); STAGE(SA(0, 0), OFFA(c0)); STAGE(SB(0, 1), OFFB(c0) + 16384); STAGE(SA(0, 1), OFFA(c0) + 16384);
            if (wr == 1) BAR;
            WAIT_V(4); BAR;
            STAGE(SB(1, 0), OFFB(c1)); STAGE(SA(1, 0), OFFA(c1)); STAGE(SB(1, 1), OFFB(c1) + 16384);
            WAIT_V(6); BAR;
            a1 = OFFA(c1);
        }
        STAMP(2);
        for (int t = 0; t < nsteps; t += 2) {
            const int t2 = t + 2 < nsteps ? t + 2 : 0;
            int u2, u3;
            RESCALE(ucur, u0);
            LDB(B0, 0, 0); SCHED; LDA(At, 0, 0); STAGE(SA(1, 1), a1 + 16384);
            WAIT_L(8); BAR; WAIT_L(0); MMA(0, 0, At, B0); BAR; SCHED;
            const unsigned c2 = fetch(t2, u1, u2), a2 = OFFA(c2), b2 = OFFB(c2);
            LDB(B1, 0, 1); STAGE(SB(0, 0), b2);
            BAR; WAIT_L(0); MMA(0, 1, At, B1); BAR;
            LDA(At, 0, 1); STAGE(SA(0, 0), a2);
            BAR; WAIT_L(0); MMA(1, 0, At, B0); BAR; SCHED;
            STAGE(SB(0, 1), b2 + 16384);
            WAIT_V(6); BAR; MMA(1, 1, At, B1); BAR;
            RESCALE(u0, u1);
            LDB(B0, 1, 0); SCHED; LDA(At, 1, 0); STAGE(SA(0, 1), a2 + 16384);
            WAIT_L(8); BAR; WAIT_L(0); MMA(0, 0, At, B0); BAR; SCHED;
            const unsigned c3 = fetch(t2 + 1, u2, u3), a3 = OFFA(c3), b3 = OFFB(c3);
            LDB(B1, 1, 1); STAGE(SB(1, 0), b3);
            BAR; WAIT_L(0); MMA(0, 1, At, B1); BAR;
            LDA(At, 1, 1); STAGE(SA(1, 0), a3);
            BAR; WAIT_L(0); MMA(1, 0, At, B0); BAR; SCHED;
            STAGE(SB(1, 1), b3 + 16384);
            WAIT_V(6); BAR; MMA(1, 1, At, B1); BAR;
            a1 = a3; ucur = u1; u0 = u2; u1 = u3;
        }
        STAMP(3);
        const bool dead = uni(ld_agent(wsl + OFF_TMO)) == call;
        const float qn = __builtin_nanf("");
        const int sh = 16 * (ucur - 16);
        if (cid >= 0) {
            const unsigned base = OFF_DATA + (unsigned)(cid + wid) * BLOB;
#pragma unroll
            for (int ai = 0; ai < 2; ++ai)
#pragma unroll
                for (int m = 0; m < 4; ++m)
#pragma unroll
                    for (int r = 0; r < 4; ++r) {
                        f32x4 v = {__builtin_ldexpf(acc[ai][0][m][0][r], sh), __builtin_ldexpf(acc[ai][0][m][1][r], sh),
                                   __builtin_ldexpf(acc[ai][1][m][0][r], sh), __builtin_ldexpf(acc[ai][1][m][1][r], sh)};
                        if (dead) v = (f32x4){qn, qn, qn, qn};
                        st128<AUX_SYS>(__builtin_bit_cast(u32x4, v), rsl, lane16, base + (ai * 16 + m * 4 + r) * 1024);
                    }
            WAIT_V(0);
            if (lane == 0) {
                if (fl_rank == rank) st_agent(wsl + OFF_LC + 16 * (fl_idx + wid), tag0);
                else if ((unsigned)fl_rank < (unsigned)NR) st_sys(p.ws[fl_rank] + off_rf(call) + 4 * (fl_idx + wid), tag0);
            }
        } else {
            const int lane_out = (lane >> 4) * 4 * (NG * 4) + (lane & 15) * 16;
            const unsigned base = out_off + (unsigned)(wr * 64) * (NG * 4) + wc * 256;
#pragma unroll
            for (int ai = 0; ai < 2; ++ai)
#pragma unroll
                for (int m = 0; m < 4; ++m)
#pragma unroll
                    for (int r = 0; r < 4; ++r) {
                        f32x4 v = {__builtin_ldexpf(acc[ai][0][m][0][r], sh), __builtin_ldexpf(acc[ai][0][m][1][r], sh),
                                   __builtin_ldexpf(acc[ai][1][m][0][r], sh), __builtin_ldexpf(acc[ai][1][m][1][r], sh)};
                        if (dead) v = (f32x4){qn, qn, qn, qn};
                        st128<AUX_SC1>(__builtin_bit_cast(u32x4, v), rso, lane_out, base + (unsigned)(ai * 128 + m * 16 + r) * (NG * 4));
                    }
            WAIT_V(0);
        }
        STAMP(4);
        if (wr == 0) BAR;
        BAR;
    }
}

}

extern "C" void kernel_launch(int rank, int nranks, ncclComm_t comm,
                              hipStream_t stream,
                              void* const* d_in, const int* in_sizes, int n_in,
                              void* d_out, int out_size,
                              const int* in_dtypes, int out_dtype,
                              void* const* d_out_all,
                              void* const* d_ws_all, size_t ws_bytes) {
    (void)comm; (void)in_sizes; (void)n_in; (void)in_dtypes; (void)out_dtype; (void)d_out_all;
    if (nranks != NR || (unsigned)rank >= (unsigned)NR || ws_bytes < WS_NEED || (size_t)out_size * 4 != OUT_BYTES) {
        fprintf(stderr, "kernel_launch: unexpected configuration\n");
        return;
    }
    Params p;
    p.x = (const float*)d_in[0];
    p.w = (const float*)d_in[1];
    p.out = (float*)d_out;
    for (int r = 0; r < NR; ++r) p.ws[r] = (char*)d_ws_all[r];
    p.rank = rank;
    int n = 0;
    for (int tt = 0; tt < 8; ++tt) {
        if (tt >= NPT) for (int j = 0; j < 8; ++j) p.order[n++] = (unsigned short)(0x100 | (j << 3) | tt);
        for (int u = 1; u < 7; ++u) p.order[n++] = (unsigned short)(((2 * u) << 3) | tt);
        if (tt < 8 - CFG_D2) for (int u = 1; u < 7; ++u) p.order[n++] = (unsigned short)(((2 * u + 1) << 3) | tt);
        if (tt < 8 - CFG_D3) for (int h = 0; h < 2; ++h) p.order[n++] = (unsigned short)((h << 3) | tt);
    }
    for (int tt = 0; tt < 8; ++tt) {
        if (tt >= 8 - CFG_D2) for (int u = 1; u < 7; ++u) p.order[n++] = (unsigned short)(((2 * u + 1) << 3) | tt);
        if (tt >= 8 - CFG_D3) for (int h = 0; h < 2; ++h) p.order[n++] = (unsigned short)((h << 3) | tt);
    }
    HIP_CHECK(hipFuncSetAttribute((const void*)gemm_rs_fused, hipFuncAttributeMaxDynamicSharedMemorySize, LDS_BYTES));
    hipLaunchKernelGGL(gemm_rs_fused, dim3(GRID), dim3(512), LDS_BYTES, stream, p);
    HIP_CHECK(hipGetLastError());
}
```

```cpp
#include <hip/hip_runtime.h>
#include <rccl/rccl.h>
#include <cstdio>
#include <cstdint>

#define HIP_CHECK(expr) do { hipError_t _e = (expr); if (_e != hipSuccess) { \
    fprintf(stderr, "HIP error %s at %s:%d\n", hipGetErrorString(_e), __FILE__, __LINE__); \
    return; } } while (0)

namespace {

#define LAS __attribute__((address_space(3)))
typedef _Float16 h8 __attribute__((ext_vector_type(8)));
typedef float f32x4 __attribute__((ext_vector_type(4)));
typedef unsigned u32x4 __attribute__((ext_vector_type(4)));
typedef __amdgpu_buffer_rsrc_t rsrc_t;
typedef unsigned long long u64;

constexpr int NR = 8;
constexpr int KSH = 512;
constexpr int NG = 16384;
constexpr int BLOB = 32768;
#define CFG_NW 6
#define CFG_PW 5
#define CFG_PU 2
#define CFG_PHI 14
#define CFG_WIN 32u
#define CFG_OWN0 1000
#define CFG_OWNSP 1
#define CFG_J2 4
#define CFG_SP 8
#define CFG_XL 8
#define CFG_SLEEP 48
#define CFG_FINE 16
#define CFG_D2 3
#define CFG_D3 3
constexpr int NW = CFG_NW;
constexpr int PW = CFG_PW;
constexpr int PU = CFG_PU;
constexpr int SL = 4;
constexpr int SLICE = BLOB / SL;
constexpr int SPI = SLICE / (PU * 1024);
constexpr int GRID = 256, NGEMM = 128, NPULL = (NR - 1) * NW, NCONV = GRID - NPULL, NHELP = GRID - NGEMM - NPULL;
constexpr int PHI = CFG_PHI;
constexpr int N_IND = 7 * 8 + 16 * 8 - 8 - PHI;
constexpr int N_INP = 8 * 8 + 16 * 8 - 8 + 6 * PHI;
constexpr int N_OUT = 128;
constexpr unsigned N_FINE = CFG_FINE;
static_assert(4 % PU == 0 && SPI * PU * 1024 * SL == BLOB && SPI >= 2 && SPI % 2 == 0 && N_FINE <= N_OUT, "puller geometry");
static_assert(NHELP > 0 && N_IND + N_OUT <= 512 && N_INP <= 512 && PHI <= 16, "geometry");

constexpr int ID_XR = 0;
constexpr int ID_P = 512;
constexpr int ID_PAR = 1536;
constexpr int PO_X = 0  , PO_W = 128  , PO_C = 640  ,
              PO_WR = 1664  , PAR_SZ = 2688;
constexpr int RF_WR = ID_PAR, N_RF = ID_PAR + 1024;
constexpr int ID_ZERO = ID_PAR + 2 * PAR_SZ;
constexpr int N_IDS = ID_ZERO + 1;
constexpr unsigned OFF_CTR = 0;
constexpr unsigned OFF_TMO = 32768;
constexpr unsigned OFF_TK = 36864;
constexpr unsigned OFF_RF0 = 65536;
constexpr unsigned OFF_LC = 163840;
constexpr unsigned OFF_DBG = 1u << 20;
constexpr unsigned OFF_DATA = 2u << 20;
static_assert(OFF_LC + 16u * N_IDS <= OFF_DBG && OFF_RF0 + 32u * N_RF <= OFF_LC && N_IDS < 32768, "control layout");
constexpr size_t WS_NEED = (size_t)OFF_DATA + (size_t)N_IDS * BLOB;
constexpr unsigned OUT_BYTES = 512u * NG * 4u;
constexpr u64 DEADLINE = 50000000ull;
constexpr int AUX_SC1 = 16, AUX_SYS = 17;
constexpr int LDS_FLAGS = 131072, LDS_BYTES = LDS_FLAGS + 8 * 2048;
constexpr int VOFF_OOB = 0x70000000;

struct Params {
    const float* x; const float* w; float* out;
    char* ws[NR];
    int rank;
    unsigned short pord[N_INP];
    unsigned char dord[N_IND];
    unsigned char kord[2][16][56];
};

#define MEMFENCE() asm volatile("" ::: "memory")
#define WAITV_IMM(n) (((n) & 15) | (7 << 4) | (15 << 8) | (((n) >> 4) << 14))
#define WAITV(n) do { MEMFENCE(); __builtin_amdgcn_s_waitcnt(WAITV_IMM(n)); MEMFENCE(); } while (0)

#define STAMP(slot) do {} while (0)

__device__ __forceinline__ unsigned ld_agent(const void* p) { return __hip_atomic_load((const unsigned*)p, __ATOMIC_RELAXED, __HIP_MEMORY_SCOPE_AGENT); }
__device__ __forceinline__ unsigned ld_sys(const void* p) { return __hip_atomic_load((const unsigned*)p, __ATOMIC_RELAXED, __HIP_MEMORY_SCOPE_SYSTEM); }
__device__ __forceinline__ void st_agent(void* p, unsigned v) { __hip_atomic_store((unsigned*)p, v, __ATOMIC_RELAXED, __HIP_MEMORY_SCOPE_AGENT); }
__device__ __forceinline__ void st_sys(void* p, unsigned v) { __hip_atomic_store((unsigned*)p, v, __ATOMIC_RELAXED, __HIP_MEMORY_SCOPE_SYSTEM); }
__device__ __forceinline__ rsrc_t mk_rsrc(const void* p, unsigned bytes) { return __builtin_amdgcn_make_buffer_rsrc((void*)p, 0, bytes, 0x00020000); }
template <int AUX> __device__ __forceinline__ void st128(u32x4 v, rsrc_t rs, int voff, unsigned soff) {
    __builtin_amdgcn_raw_buffer_store_b128(v, rs, voff, soff, AUX);
    asm volatile("s_nop 1" :: "v"(v));
}
__device__ __forceinline__ unsigned off_rf(unsigned call) { return OFF_RF0 + (call & 1u) * (16u * N_RF); }
__device__ __forceinline__ unsigned uni(unsigned v) { return (unsigned)__builtin_amdgcn_readfirstlane((int)v); }

__device__ __forceinline__ bool give_up(char* wsl, unsigned call, u64 t0) {
    if (uni(ld_agent(wsl + OFF_TMO)) == call || __builtin_amdgcn_s_memrealtime() - t0 > DEADLINE) {
        st_agent(wsl + OFF_TMO, call);
        return true;
    }
    return false;
}

#define CONV_LD(p) (*(p))
__device__ __forceinline__ void conv_load(const Params& p, int n, int wid, int lane, f32x4 (&v)[8]) {
    const int tt = n / 80, r = n % 80, g = lane >> 4, c = lane & 15;
    if (r < 16) {
        const float* src = p.x + (size_t)(r * 256 + wid * 32 + c) * KSH + tt * 64 + 4 * g;
#pragma unroll
        for (int f = 0; f < 2; ++f)
#pragma unroll
            for (int ii = 0; ii < 4; ++ii) v[f * 4 + ii] = CONV_LD((const f32x4*)(src + f * 16 * KSH + ii * 16));
    } else {
        const int ks = wid & 1, wc = wid >> 1;
        const float* src = p.w + (size_t)(tt * 64 + ks * 32 + 4 * g) * NG + (r - 16) * 256 + wc * 64 + 4 * c;
#pragma unroll
        for (int e = 0; e < 8; ++e) v[e] = CONV_LD((const f32x4*)(src + (size_t)((e >> 2) * 16 + (e & 3)) * NG));
    }
}
__device__ __forceinline__ unsigned conv_class(const f32x4 (&v)[8]) {
    float m = 0.f;
#pragma unroll
    for (int e = 0; e < 8; ++e)
#pragma unroll
        for (int j = 0; j < 4; ++j) m = __builtin_fmaxf(m, __builtin_fabsf(v[e][j]));
    const unsigned c = ((__builtin_bit_cast(unsigned, m) >> 23) + 3u) >> 4;
    unsigned c0 = uni(c);
    for (;;) {
        const u64 up = __ballot(c > c0);
        if (!up) break;
        c0 = (unsigned)__builtin_amdgcn_readlane((int)c, __builtin_ctzll(up));
    }
    return c0;
}
__device__ __forceinline__ int push_dest(int r) { return ((((r - 16) >> 4) << 1) | (r & 1)) & (NR - 1); }
__device__ __forceinline__ bool conv_pushes(int n, int rank) { return n >= 16 && n < 80 && push_dest(n) != rank; }
template <bool MAY_PUSH>
__device__ __forceinline__ void conv_store(const Params& p, rsrc_t rsl, int n, int pb, int wid, int lane, const f32x4 (&v)[8], unsigned cls) {
    const int tt = n / 80, r = n % 80, sh = 128 - 16 * (int)cls;
    if (r < 16) {
        const unsigned base = OFF_DATA + (unsigned)(pb + PO_X + r * 8 + tt) * BLOB + wid * 4096;
#pragma unroll
        for (int f = 0; f < 2; ++f)
#pragma unroll
            for (int ks = 0; ks < 2; ++ks) {
                h8 o;
#pragma unroll
                for (int e = 0; e < 8; ++e) o[e] = (_Float16)__builtin_ldexpf(v[f * 4 + ks * 2 + (e >> 2)][e & 3], sh);
                st128<AUX_SYS>(__builtin_bit_cast(u32x4, o), rsl, lane * 16, base + f * 2048 + ks * 1024);
            }
    } else {
        const int ks = wid & 1, wc = wid >> 1;
        const unsigned inb = wc * 4096 + ks * 1024, base = OFF_DATA + (unsigned)(pb + PO_W + (r - 16) * 8 + tt) * BLOB + inb;
        h8 o[4];
#pragma unroll
        for (int f = 0; f < 4; ++f)
#pragma unroll
            for (int e = 0; e < 8; ++e) o[f][e] = (_Float16)__builtin_ldexpf(v[e][f], sh);
#pragma unroll
        for (int f = 0; f < 4; ++f) st128<AUX_SYS>(__builtin_bit_cast(u32x4, o[f]), rsl, lane * 16, base + (f >> 1) * 16384 + (f & 1) * 2048);
        if (MAY_PUSH && conv_pushes(n, p.rank)) {
            MEMFENCE();
            const rsrc_t rsq = mk_rsrc(p.ws[push_dest(r)], (unsigned)WS_NEED);
            const unsigned rb = OFF_DATA + (unsigned)(pb + PO_WR + ((r - 16) & 15) * 64 + p.rank * 8) * BLOB + inb;
#pragma unroll
            for (int f = 0; f < 4; ++f) st128<AUX_SYS>(__builtin_bit_cast(u32x4, o[f]), rsq, lane * 16, rb + (f >> 1) * 16384 + (f & 1) * 2048);
        }
    }
}
template <bool MAY_PUSH>
__device__ __forceinline__ unsigned conv_agree(const Params& p, rsrc_t rsl, int n, int pb, int wid, int lane, const f32x4 (&v)[8], unsigned mine,
                                               const LAS unsigned* cls) {
    const u32x4 a = *(const LAS u32x4*)cls, b = *(const LAS u32x4*)(cls + 4);
    const unsigned hi = uni(max(max(max(a[0], a[1]), max(a[2], a[3])), max(max(b[0], b[1]), max(b[2], b[3]))));
    const unsigned lo = uni(min(min(min(a[0], a[1]), min(a[2], a[3])), min(min(b[0], b[1]), min(b[2], b[3]))));
    if (lo != hi) {
        WAITV(0);
        if (mine != hi) conv_store<MAY_PUSH>(p, rsl, n, pb, wid, lane, v, hi);
        WAITV(0);
        __builtin_amdgcn_s_barrier();
        MEMFENCE();
    }
    return hi;
}
__device__ __forceinline__ void conv_publish(int rank, char* wst  , int n, int pb, unsigned tag, bool pushed) {
    const int tt = n / 80, r = n % 80, t = threadIdx.x >> 2, wd = threadIdx.x & 3;
    const unsigned OFF_RF = off_rf(tag >> 8);
    if (pushed) {
        if (t == push_dest(r)) st_sys(wst + OFF_LC + 16 * (pb + PO_WR + ((r - 16) & 15) * 64 + rank * 8) + 4 * wd, tag);
    } else if (t < NR) {
        if (t == rank) st_agent(wst + OFF_LC + 16 * (pb + (r < 16 ? PO_X + r * 8 : PO_W + (r - 16) * 8) + tt) + 4 * wd, tag);
        else if (r < 16) {
            const int o = r >> 1, li = o == t ? (r & 1) : (o ^ t);
            if (o != rank && (o == t || (li >= 2 && ((r ^ t) & 1) == 0))) st_sys(wst + OFF_RF + 16 * (ID_XR + li * 64 + rank * 8 + tt) + 4 * wd, tag);
        } else if (((r - 16) >> 4) == (t >> 1)) {
            const int j = (r - 16) & 15;
            if (!(tt == 0 && ((j ^ t) & 1) == 0) && !(j < PHI && tt == 1 + (t & 1) && (rank ^ t) >= 2))
                st_sys(wst + OFF_RF + 16 * (RF_WR + j * 64 + rank * 8 + tt) + 4 * wd, tag);
        }
    }
}

struct PStep { unsigned src, dst, did, tag; bool valid, last, fwd; };

__device__ __forceinline__ bool is_out(unsigned did) { return did - (unsigned)ID_P < (unsigned)(ID_PAR - ID_P); }

__device__ __forceinline__ void pull_items(char* wsl, rsrc_t rsl, rsrc_t rsp, rsrc_t rso, rsrc_t rsq  ,
                                           const LAS unsigned* desc  ,
                                           unsigned n_full, unsigned n_units, unsigned call, int pb, int lane, int peer) {
    const int lane16 = lane * 16, lane_out = (lane >> 4) * 4 * (NG * 4) + (lane & 15) * 16;
    const u64 t0 = __builtin_amdgcn_s_memrealtime();
    const int lane0 = lane == 0 ? 0 : VOFF_OOB;
    const unsigned tk_base = (call - 1u) * (n_units + NW * PW);
    bool dead = false;
    int sub = SPI;
    int nsub = 0;
    int nstate = 0;
    unsigned cur = 0, cofs = 0, ctag = 0, nxt = 0, nofs = 0, ntag = 0, nrf = 0, pend = 0, pend2 = 0, ptag = 0, spins = 0;
    const unsigned tag0 = call << 8, OFF_RF = off_rf(call);
    auto step = [&](const PStep& cu, const u32x4 (&rc)[PU], const u32x4 (&qc)[PU], PStep& nx, u32x4 (&rn)[PU], u32x4 (&qn)[PU]) -> bool {
        MEMFENCE();
        if (sub == SPI && nstate == 2) { cur = nxt; cofs = nofs; ctag = ntag; sub = nsub; nstate = 0; }
        nx.valid = sub < SPI;
        const unsigned inb = cofs + (nx.valid ? sub : SPI - 1) * (PU * 1024);
        nx.did = cur & 0x7fffu;
        nx.fwd = (cur >> 15) & 1u;
        nx.tag = ctag;
        nx.src = OFF_DATA + (cur >> 16) * BLOB + inb;
        nx.dst = OFF_DATA + nx.did * BLOB + inb;
        if (nx.valid) ++sub;
        nx.last = sub == SPI;
        const int vt = nstate == 0 ? lane0 : VOFF_OOB, vf = nstate == 1 ? lane0 : VOFF_OOB;
        const unsigned f0 = (unsigned)__builtin_amdgcn_raw_ptr_buffer_atomic_add_i32(1, rsl, vt, OFF_TK + peer * 128, 0);
        const unsigned f1 = __builtin_amdgcn_raw_buffer_load_b32(rsl, vf, nrf, AUX_SYS);
        const unsigned f2 = __builtin_amdgcn_raw_buffer_load_b32(rsl, vf, OFF_LC + 16 * (nxt & 0x7fffu), AUX_SC1);
        MEMFENCE();
        const int vr = nx.valid ? lane16 : VOFF_OOB, vq = (nx.valid && is_out(nx.did)) ? lane16 : VOFF_OOB;
#pragma unroll
        for (int u = 0; u < PU; ++u) rn[u] = __builtin_amdgcn_raw_buffer_load_b128(rsp, vr, nx.src + u * 1024, AUX_SYS);
#pragma unroll
        for (int u = 0; u < PU; ++u) qn[u] = __builtin_amdgcn_raw_buffer_load_b128(rsl, vq, nx.dst + u * 1024, AUX_SC1);
        WAITV(2 * PU);
        if (pend) {
            __builtin_amdgcn_raw_buffer_store_b32(ptag, rsl, lane0, pend, AUX_SC1);
            if (pend2) { __builtin_amdgcn_raw_buffer_store_b32(ptag, rsq, lane0, pend2, AUX_SYS); pend2 = 0; }
            pend = 0;
        }
        if (cu.valid) {
            if (is_out(cu.did)) {
                const unsigned cb = (cu.did - ID_P) & 127u, e = ((cu.dst - OFF_DATA) & (BLOB - 1)) >> 10;
                const unsigned row = (peer & 1) * 256 + (e >> 4) * 128 + ((cb >> 2) & 1) * 64 + ((e >> 2) & 3) * 16 + (e & 3);
                const unsigned col = (peer >> 1) * 4096 + (cb >> 3) * 256 + (cb & 3) * 64;
                const unsigned so = row * (NG * 4) + col * 4;
#pragma unroll
                for (int u = 0; u < PU; ++u) {
                    f32x4 v = __builtin_bit_cast(f32x4, rc[u]) + __builtin_bit_cast(f32x4, qc[u]);
                    if (dead) v = (f32x4){__builtin_nanf(""), __builtin_nanf(""), __builtin_nanf(""), __builtin_nanf("")};
                    st128<AUX_SC1>(__builtin_bit_cast(u32x4, v), rso, lane_out, so + u * (NG * 4));
                }
            } else {
#pragma unroll
                for (int u = 0; u < PU; ++u) {
                    if (cu.fwd) st128<AUX_SYS>(rc[u], rsl, lane16, cu.dst + u * 1024);
                    else st128<AUX_SC1>(rc[u], rsl, lane16, cu.dst + u * 1024);
                }
                if (cu.last) {
                    const unsigned wd = 4 * (((cu.dst - OFF_DATA) & (BLOB - 1)) / SLICE);
                    pend = OFF_LC + 16 * cu.did + wd; ptag = cu.tag;
                    if (cu.fwd) pend2 = OFF_RF + 16 * (cu.did - pb - PO_WR + RF_WR) + wd;
                }
            }
        }
        const unsigned g0 = uni(f0), g1 = uni(f1), g2 = uni(f2);
        if (nstate == 0) {
            const unsigned un = g0 - tk_base;
            if (un < n_units) {
                const bool fine = un >= n_full;
                const unsigned hu = fine ? un + n_full : 2 * un;
                nxt = uni(desc[hu / (2 * SL)]);
                nsub = fine ? SPI / 2 : 0;
                nofs = (hu % (2 * SL)) * (SLICE / 2) - nsub * (PU * 1024);
                const unsigned nd = nxt & 0x7fffu;
                nrf = OFF_RF + 16 * (nd < (unsigned)ID_PAR ? nd : nd - pb - PO_WR + RF_WR) + 4 * ((hu % (2 * SL)) >> 1);
                ntag = tag0 | 255u; nstate = dead ? 2 : 1;
            } else nstate = 3;
        } else if (nstate == 1) {
            if (g1 >= tag0 && (!is_out(nxt & 0x7fffu) || g2 >= tag0)) { ntag = g1; nstate = 2; }
            else if (!cu.valid && !nx.valid) {
                if ((++spins & 63u) == 0 && give_up(wsl, call, t0)) { dead = true; nstate = 2; }
                __builtin_amdgcn_s_sleep(2);
            }
        }
        return !cu.valid && !nx.valid && nstate == 3;
    };
    PStep sa = {0, 0, 0, 0, false, false, false}, sb;
    u32x4 ra[PU] = {}, qa[PU] = {}, rb[PU], qb[PU];
    bool done;
    do {
        step(sa, ra, qa, sb, rb, qb);
        done = step(sb, rb, qb, sa, ra, qa);
    } while (!done);
}

__global__ __launch_bounds__(512, 2) void gemm_rs_fused(const Params p) {
    extern __shared__ __attribute__((aligned(16))) char shm[];
    LAS char* lds = (LAS char*)shm;
    const int b = blockIdx.x, tid = threadIdx.x, lane = tid & 63, wid = __builtin_amdgcn_readfirstlane(tid >> 6);
    const int rank = p.rank;
    char* wsl = p.ws[rank];
    const rsrc_t rsl = mk_rsrc(wsl, (unsigned)WS_NEED);
    const rsrc_t rso = mk_rsrc(p.out, OUT_BYTES);
    STAMP(0);

    if (b >= NGEMM && b < NGEMM + NPULL) {
        const unsigned call = uni(ld_agent(wsl + OFF_CTR + b * 128)) + 1u;
        const int pi = b - NGEMM, ds = 1 + pi % (NR - 1), peer = (rank ^ ds) & (NR - 1);
        const int pb = ID_PAR + (int)(call & 1u) * PAR_SZ;
        const rsrc_t rsp = mk_rsrc(p.ws[peer], (unsigned)WS_NEED), rsq = mk_rsrc(p.ws[rank ^ 1], (unsigned)WS_NEED);
        LAS unsigned* desc = (LAS unsigned*)lds;
        const int n_in = ds == 1 ? N_INP : N_IND, n_items = ds == 1 ? N_INP : N_IND + N_OUT;
        if (tid < n_items) {
            unsigned did, sid;
            if (tid < n_in) {
                unsigned kind, a, c;
                if (ds == 1) { const unsigned code = p.pord[tid]; kind = code >> 8; a = (code >> 3) & 15u; c = code & 7u; }
                else {
                    const unsigned code = p.dord[tid]; kind = code >> 7; a = (code >> 3) & 15u; c = code & 7u;
                    if (kind == 0 && a >= (unsigned)ds) ++a;
                }
                if (kind == 0) {
                    const unsigned i = a < 2 ? rank * 2 + a : ((rank ^ a) * 2 + (rank & 1));
                    did = ID_XR + a * 64 + peer * 8 + c; sid = pb + PO_X + i * 8 + c;
                } else if (kind == 1) {
                    did = pb + PO_WR + a * 64 + peer * 8 + c; sid = pb + PO_W + ((rank >> 1) * 16 + a) * 8 + c;
                    if (ds != 1 && a < (unsigned)PHI && c == 2u - (rank & 1)) did |= 0x8000u;
                } else did = sid = pb + PO_WR + a * 64 + (rank ^ c) * 8 + 1 + (rank & 1);
            } else {
                const unsigned cb = (tid - n_in) ^ ((peer & 1) << 3);
                did = ID_P + peer * 128 + cb; sid = pb + PO_C + rank * 128 + cb;
            }
            desc[tid] = (sid << 16) | did;
        }
        asm volatile("s_waitcnt vmcnt(0) lgkmcnt(0)" ::: "memory");
        __builtin_amdgcn_s_barrier();
        MEMFENCE();
        if (tid == 0) st_agent(wsl + OFF_CTR + b * 128, call);
        if (wid >= PW) return;
        const unsigned n_fine = ds == 1 ? 0 : N_FINE, n_full = (n_items - n_fine) * SL;
        pull_items(wsl, rsl, rsp, rso, rsq, desc, n_full, n_full + n_fine * SL * 2, call, pb, lane, peer);
        STAMP(2);
        return;
    }

    unsigned call;
    int pb;
    {
        const int ci = b < NGEMM ? b : b - NPULL;
        static_assert(2 * NCONV <= 640 && 3 * NCONV >= 640 && 80 <= NCONV, "three blobs per converter at most, two at least");
        const int n2 = ci + 2 * NCONV < 640 ? ci + 2 * NCONV : ci;
        f32x4 v0[8], v1[8], v2[8];
        const unsigned c0 = ld_agent(wsl + OFF_CTR + b * 128);
        char* wst = p.ws[0];
#pragma unroll
        for (int r = 1; r < NR; ++r) wst = (tid >> 2) == r ? p.ws[r] : wst;
        LAS unsigned* cls = (LAS unsigned*)lds;
        MEMFENCE();
        if (ci >= 80 && ci % 80 >= 16) __builtin_amdgcn_s_sleep(CFG_SLEEP);
        conv_load(p, ci, wid, lane, v0);
        WAITV(0);
        call = uni(c0) + 1u;
        pb = ID_PAR + (int)(call & 1u) * PAR_SZ;
        const unsigned k0 = conv_class(v0);
        conv_store<true>(p, rsl, ci, pb, wid, lane, v0, k0);
        if (lane == 0) cls[wid] = k0;
        STAMP(5);
        MEMFENCE();
        asm volatile("s_waitcnt lgkmcnt(0)" ::: "memory");
        const bool pushes = conv_pushes(ci, rank);
        if (pushes) WAITV(4); else WAITV(0);
        STAMP(8);
        __builtin_amdgcn_s_barrier();
        MEMFENCE();
        const unsigned tagc0 = (call << 8) | conv_agree<true>(p, rsl, ci, pb, wid, lane, v0, k0, cls);
        conv_publish(rank, wst, ci, pb, tagc0, false);
        STAMP(6);
        MEMFENCE();
        conv_load(p, ci + NCONV, wid, lane, v1);
        conv_load(p, n2, wid, lane, v2);
        if (tid == 0) st_agent(wsl + OFF_CTR + b * 128, call);
        WAITV(0);
        const unsigned k1 = conv_class(v1), k2 = conv_class(v2);
        conv_store<false>(p, rsl, ci + NCONV, pb, wid, lane, v1, k1);
        if (n2 != ci) conv_store<false>(p, rsl, n2, pb, wid, lane, v2, k2);
        if (lane == 0) { cls[8 + wid] = k1; cls[16 + wid] = k2; }
        STAMP(7);
        asm volatile("s_waitcnt lgkmcnt(0)" ::: "memory");
        WAITV(0);
        __builtin_amdgcn_s_barrier();
        MEMFENCE();
        if (pushes) conv_publish(rank, wst, ci, pb, tagc0, true);
        conv_publish(rank, wst, ci + NCONV, pb, (call << 8) | conv_agree<false>(p, rsl, ci + NCONV, pb, wid, lane, v1, k1, cls + 8), false);
        if (n2 != ci) conv_publish(rank, wst, n2, pb, (call << 8) | conv_agree<false>(p, rsl, n2, pb, wid, lane, v2, k2, cls + 16), false);
        asm volatile("s_waitcnt lgkmcnt(0)" ::: "memory");
        __builtin_amdgcn_s_barrier();
        MEMFENCE();
    }
    STAMP(1);

    const unsigned tag0 = call << 8;
    const int wr = wid >> 2, wc = wid & 3;
    for (int job = 0;; ++job) {
        int nsteps, cid, fl_rank;
        unsigned ids, fl_idx, out_off = 0;
        if (b < NGEMM) {
            if (job) break;
            const int xcd = b & 7, kk = b >> 3, li = (xcd & 1) * 4 + (kk & 3), j = (xcd >> 1) * 4 + (kk >> 2);
            const int own = li < 2 ? rank : (rank ^ li), i = own * 2 + (li < 2 ? li : (rank & 1)), J = (rank >> 1) * 16 + j;
            nsteps = li < 2 ? 64 : 56;
            unsigned mine; int pos;
            if (lane >= 56) { mine = (unsigned)(pb + PO_X + i * 8 + lane - 56) | ((unsigned)(pb + PO_W + J * 8 + lane - 56) << 16); pos = lane - 56; }
            else {
                const unsigned code = p.kord[li < 2][j][lane], d = code >> 3, st = ((rank ^ d) & (NR - 1)) * 8 + (code & 7u);
                mine = (unsigned)(ID_XR + li * 64 + st) | ((unsigned)(pb + PO_WR + j * 64 + st) << 16);
                const bool use = li < 2 || d != (unsigned)li;
                const u64 um = __ballot(use);
                const int below = __builtin_popcountll(um & ((1ull << lane) - 1ull));
                pos = use ? 8 + below : 56 + lane - below;
            }
            ids = (unsigned)__builtin_amdgcn_ds_permute(pos * 4, (int)mine);
            cid = li < 2 ? -1 : pb + PO_C + own * 128 + j * 8;
            fl_rank = own; fl_idx = ID_P + rank * 128 + j * 8;
            out_off = (unsigned)(li * 256) * (NG * 4) + (unsigned)(J * 256) * 4;
        } else {
            const int pt = (b - NGEMM - NPULL) + job * NHELP;
            if (pt >= 96) break;
            const int j = pt / 6, q = (rank ^ (2 + pt % 6)) & (NR - 1), m = lane & 7;
            nsteps = 8;
            ids = (unsigned)(pb + PO_X + (rank * 2 + (q & 1)) * 8 + m) | ((unsigned)(pb + PO_W + ((q >> 1) * 16 + j) * 8 + m) << 16);
            cid = ID_P + q * 128 + j * 8;
            fl_rank = rank; fl_idx = cid;
        }

        u64 mask = 0;
        unsigned cl = 0;
        int umax = -64;
        bool dead = false;
        auto fetch = [&](int n, int uprev, int& unit) -> unsigned {
            if (!((mask >> n) & 1ull)) {
                const u64 t0 = __builtin_amdgcn_s_memrealtime();
                for (unsigned spins = 1;; ++spins) {
                    const int vo = (unsigned)(lane - n) < (spins == 1 ? CFG_WIN : 8u) ? 0 : VOFF_OOB;
                    MEMFENCE();
                    const u32x4 fa = __builtin_amdgcn_raw_buffer_load_b128(rsl, 16 * (ids & 0xffffu) + vo, OFF_LC, AUX_SC1);
                    const u32x4 fb = __builtin_amdgcn_raw_buffer_load_b128(rsl, 16 * (ids >> 16) + vo, OFF_LC, AUX_SC1);
                    const bool ok = min(min(min(fa[0], fa[1]), min(fa[2], fa[3])), min(min(fb[0], fb[1]), min(fb[2], fb[3]))) >= tag0;
                    if (ok) cl = (fa[0] & 255u) + (fb[0] & 255u);
                    mask |= __ballot(ok);
                    if ((mask >> n) & 1ull) break;
                    if ((spins & 63u) == 0 && give_up(wsl, call, t0)) { mask = ~0ull; dead = true; break; }
                    __builtin_amdgcn_s_sleep(8);
                }
                MEMFENCE();
            }
            const int u = __builtin_amdgcn_readlane((int)cl, n);
            dead = dead || u >= 255;
            const bool tiny = u < umax - 5;
            unit = tiny ? uprev : u;
            umax = max(umax, u);
            return tiny ? (unsigned)ID_ZERO * 0x10001u : (unsigned)__builtin_amdgcn_readlane((int)ids, n);
        };
        const int lane16 = lane * 16;
        const unsigned wofs = OFF_DATA + wid * 2048;
#define SA(bf, h) (((bf) * 2 + (h)) * 16384)
#define SB(bf, h) ((4 + (bf) * 2 + (h)) * 16384)
#define STAGEV(bufoff, goff, vo) do { \
        __builtin_amdgcn_raw_ptr_buffer_load_lds(rsl, (LAS void*)(lds + (bufoff) + wid * 2048), 16, (vo), (goff), 0, AUX_SC1); \
        __builtin_amdgcn_raw_ptr_buffer_load_lds(rsl, (LAS void*)(lds + (bufoff) + wid * 2048), 16, (vo), (goff), 1024, AUX_SC1); } while (0)
#define STAGE(bufoff, goff) STAGEV(bufoff, goff, lane16)
#define PEEK_ISSUE(n) do { peek = ~mask & (0xffffull << (n)); const int vo_ = ((peek >> lane) & 1ull) ? 0 : VOFF_OOB; \
        __builtin_amdgcn_raw_ptr_buffer_load_lds(rsl, (LAS void*)(lds + LDS_FLAGS + wid * 2048), 16, 16 * (ids & 0xffffu) + vo_, OFF_LC, 0, AUX_SC1); \
        __builtin_amdgcn_raw_ptr_buffer_load_lds(rsl, (LAS void*)(lds + LDS_FLAGS + wid * 2048 + 1024), 16, 16 * (ids >> 16) + vo_, OFF_LC, 0, AUX_SC1); } while (0)
#define PEEK_READ() const u32x4 pa_ = *(const LAS u32x4*)(lds + LDS_FLAGS + wid * 2048 + lane16), pb_ = *(const LAS u32x4*)(lds + LDS_FLAGS + wid * 2048 + 1024 + lane16)
#define PEEK_USE() do { const bool ok_ = ((peek >> lane) & 1ull) && \
            min(min(min(pa_[0], pa_[1]), min(pa_[2], pa_[3])), min(min(pb_[0], pb_[1]), min(pb_[2], pb_[3]))) >= tag0; \
        if (ok_) cl = (pa_[0] & 255u) + (pb_[0] & 255u); mask |= __ballot(ok_); } while (0)
#define PEEK1(n)
#define PEEK2
#define PEEK3
#define LDA(dst, bf, h) do { _Pragma("unroll") for (int m = 0; m < 4; ++m) _Pragma("unroll") for (int k = 0; k < 2; ++k) \
        dst[m][k] = *(const LAS h8*)(lds + SA(bf, h) + aoff + m * 2048 + k * 1024); } while (0)
#define LDB(dst, bf, h) do { _Pragma("unroll") for (int n = 0; n < 2; ++n) _Pragma("unroll") for (int k = 0; k < 2; ++k) \
        dst[n][k] = *(const LAS h8*)(lds + SB(bf, h) + boff + n * 2048 + k * 1024); } while (0)
#define MMA(ai, bj, At, Bt) do { __builtin_amdgcn_s_setprio(1); \
        _Pragma("unroll") for (int m = 0; m < 4; ++m) _Pragma("unroll") for (int n = 0; n < 2; ++n) _Pragma("unroll") for (int k = 0; k < 2; ++k) \
            acc[ai][bj][m][n] = __builtin_amdgcn_mfma_f32_16x16x32_f16(At[m][k], Bt[n][k], acc[ai][bj][m][n], 0, 0, 0); \
        __builtin_amdgcn_s_setprio(0); } while (0)
#define WAIT_V(n) asm volatile("s_waitcnt vmcnt(" #n ")" ::: "memory")
#define WAIT_L(n) asm volatile("s_waitcnt lgkmcnt(" #n ")" ::: "memory")
#define BAR __builtin_amdgcn_s_barrier()
#define SCHED __builtin_amdgcn_sched_barrier(0)
#define RESCALE(from, to) do { if ((from) != (to)) { const float f_ = __builtin_ldexpf(1.0f, 16 * ((from) - (to))); \
        _Pragma("unroll") for (int a_ = 0; a_ < 2; ++a_) _Pragma("unroll") for (int c_ = 0; c_ < 2; ++c_) _Pragma("unroll") for (int m_ = 0; m_ < 4; ++m_) \
            _Pragma("unroll") for (int n_ = 0; n_ < 2; ++n_) acc[a_][c_][m_][n_] *= f_; } } while (0)
#define OFFA(c) (wofs + ((c) & 0xffffu) * BLOB)
#define OFFB(c) (wofs + ((c) >> 16) * BLOB)
        const int aoff = wr * 8192 + lane16, boff = wc * 4096 + lane16;
        f32x4 acc[2][2][4][2];
#pragma unroll
        for (int a = 0; a < 2; ++a)
#pragma unroll
            for (int c = 0; c < 2; ++c)
#pragma unroll
                for (int m = 0; m < 4; ++m)
#pragma unroll
                    for (int n = 0; n < 2; ++n) acc[a][c][m][n] = (f32x4){0.f, 0.f, 0.f, 0.f};
        h8 At[4][2], B0[2][2], B1[2][2];
        unsigned a1;
        int ucur, u0, u1;
        {
            const unsigned c0 = fetch(0, 0, u0), c1 = fetch(1, u0, u1);
            ucur = u0;
            STAGE(SB(0, 0), OFFB(c0)); STAGE(SA(0, 0), OFFA(c0)); STAGE(SB(0, 1), OFFB(c0) + 16384); STAGE(SA(0, 1), OFFA(c0) + 16384);
            if (wr == 1) BAR;
            WAIT_V(4); BAR;
            STAGE(SB(1, 0), OFFB(c1)); STAGE(SA(1, 0), OFFA(c1)); STAGE(SB(1, 1), OFFB(c1) + 16384);
            WAIT_V(6); BAR;
            a1 = OFFA(c1);
        }
        STAMP(2);
        for (int t = 0; t < nsteps; t += 2) {
            const int t2 = t + 2 < nsteps ? t + 2 : 0;
            const int vs = t + 2 < nsteps ? lane16 : VOFF_OOB;
            int u2, u3;
            u64 peek; (void)peek;
            RESCALE(ucur, u0);
            LDB(B0, 0, 0); SCHED; LDA(At, 0, 0); STAGE(SA(1, 1), a1 + 16384); PEEK1(t2 + 1)
            WAIT_L(8); BAR; WAIT_L(0); MMA(0, 0, At, B0); BAR; SCHED;
            const unsigned c2 = fetch(t2, u1, u2), a2 = OFFA(c2), b2 = OFFB(c2);
            LDB(B1, 0, 1); STAGEV(SB(0, 0), b2, vs);
            BAR; WAIT_L(0); MMA(0, 1, At, B1); BAR;
            LDA(At, 0, 1); STAGEV(SA(0, 0), a2, vs);
            BAR; WAIT_L(0); MMA(1, 0, At, B0); BAR; SCHED;
            STAGEV(SB(0, 1), b2 + 16384, vs);
            WAIT_V(6); BAR; MMA(1, 1, At, B1); BAR;
            RESCALE(u0, u1);
            PEEK2
            LDB(B0, 1, 0); SCHED; LDA(At, 1, 0); STAGEV(SA(0, 1), a2 + 16384, vs);
            WAIT_L(8); BAR; WAIT_L(0); MMA(0, 0, At, B0); BAR; SCHED;
            PEEK3
            const unsigned c3 = fetch(t2 + 1, u2, u3), a3 = OFFA(c3), b3 = OFFB(c3);
            LDB(B1, 1, 1); STAGEV(SB(1, 0), b3, vs);
            BAR; WAIT_L(0); MMA(0, 1, At, B1); BAR;
            LDA(At, 1, 1); STAGEV(SA(1, 0), a3, vs);
            BAR; WAIT_L(0); MMA(1, 0, At, B0); BAR; SCHED;
            STAGEV(SB(1, 1), b3 + 16384, vs);
            WAIT_V(6); BAR; MMA(1, 1, At, B1); BAR;
            a1 = a3; ucur = u1; u0 = u2; u1 = u3;
        }
        if (wr == 0) BAR;
        STAMP(3);
        const float qn = __builtin_nanf("");
        const int sh = 16 * (ucur - 16);
        if (cid >= 0) {
            const unsigned base = OFF_DATA + (unsigned)(cid + wid) * BLOB;
#pragma unroll
            for (int ai = 0; ai < 2; ++ai)
#pragma unroll
                for (int m = 0; m < 4; ++m)
#pragma unroll
                    for (int r = 0; r < 4; ++r) {
                        f32x4 v = {__builtin_ldexpf(acc[ai][0][m][0][r], sh), __builtin_ldexpf(acc[ai][0][m][1][r], sh),
                                   __builtin_ldexpf(acc[ai][1][m][0][r], sh), __builtin_ldexpf(acc[ai][1][m][1][r], sh)};
                        if (dead) v = (f32x4){qn, qn, qn, qn};
                        st128<AUX_SYS>(__builtin_bit_cast(u32x4, v), rsl, lane16, base + (ai * 16 + m * 4 + r) * 1024);
                    }
            WAIT_V(0);
            if (lane < 4) {
                if (fl_rank == rank) st_agent(wsl + OFF_LC + 16 * (fl_idx + wid) + 4 * lane, tag0);
                else if ((unsigned)fl_rank < (unsigned)NR) st_sys(p.ws[fl_rank] + off_rf(call) + 16 * (fl_idx + wid) + 4 * lane, tag0);
            }
        } else {
            const int lane_out = (lane >> 4) * 4 * (NG * 4) + (lane & 15) * 16;
            const unsigned base = out_off + (unsigned)(wr * 64) * (NG * 4) + wc * 256;
#pragma unroll
            for (int ai = 0; ai < 2; ++ai)
#pragma unroll
                for (int m = 0; m < 4; ++m)
#pragma unroll
                    for (int r = 0; r < 4; ++r) {
                        f32x4 v = {__builtin_ldexpf(acc[ai][0][m][0][r], sh), __builtin_ldexpf(acc[ai][0][m][1][r], sh),
                                   __builtin_ldexpf(acc[ai][1][m][0][r], sh), __builtin_ldexpf(acc[ai][1][m][1][r], sh)};
                        if (dead) v = (f32x4){qn, qn, qn, qn};
                        st128<AUX_SC1>(__builtin_bit_cast(u32x4, v), rso, lane_out, base + (unsigned)(ai * 128 + m * 16 + r) * (NG * 4));
                    }
            WAIT_V(0);
        }
        STAMP(4);
        BAR;
    }
}

void build_orders(Params& p, int e) {
    auto rt = [e](int vt) { return vt == 1 ? 2 - e : vt == 2 ? 1 + e : vt; };
    int dw[16][8], dx[2][8], pw[16][8], pf[16][8], px[2][8];
    int n = 0;
    for (int late = 0; late < 2; ++late)
        for (int vt = 0; vt < 8; ++vt) {
            if (!late) for (int u = 2; u < 7; ++u) { dx[0][vt] = n; p.dord[n++] = (unsigned char)((u << 3) | rt(vt)); }
            if ((vt >= 8 - CFG_D3) == late) for (int h = 0; h < 2; ++h) { dx[1][vt] = n; p.dord[n++] = (unsigned char)((h << 3) | rt(vt)); }
            for (int vj = 0; vj < 16; ++vj) {
                if (vt == 0 && vj % 2 == 0) { dw[vj][vt] = 0; continue; }
                if (vt == 2 && (vj ^ e) < PHI) continue;
                if ((vj >= CFG_J2 && vt >= 8 - CFG_D2) != late) continue;
                dw[vj][vt] = n; p.dord[n++] = (unsigned char)(0x80 | ((vj ^ e) << 3) | rt(vt));
            }
        }
    int key[N_INP], code[N_INP], m = 0;
    for (int vj = 0; vj < 16; ++vj) {
        const int cnt = (vj % 2 == 0 ? 7 : 8) + ((vj ^ e) < PHI ? 6 : 0);
        int k = 0;
        for (int vt = (vj % 2 == 0 ? 1 : 0); vt < 8; ++vt, ++k) { key[m] = 32 * vj - CFG_SP * (cnt - 1 - k); code[m++] = 0x100 | (vj << 3) | vt; }
        if ((vj ^ e) < PHI) for (int d = 2; d < 8; ++d, ++k) { key[m] = 32 * vj - CFG_SP * (cnt - 1 - k); code[m++] = 0x200 | (vj << 3) | d; }
        pw[vj][0] = 0;
    }
    for (int vt = 0; vt < 8; ++vt) {
        for (int li = 2; li < 8; ++li) { key[m] = CFG_XL * (vt - 8) - (vt < 3 ? 1000 : 0); code[m++] = (li << 3) | vt; }
        for (int h = 0; h < 2; ++h) { key[m] = CFG_OWN0 + CFG_OWNSP * vt; code[m++] = (h << 3) | vt; }
    }
    for (int i = 1; i < m; ++i) {
        const int k = key[i], c = code[i]; int q = i;
        for (; q > 0 && key[q - 1] > k; --q) { key[q] = key[q - 1]; code[q] = code[q - 1]; }
        key[q] = k; code[q] = c;
    }
    for (int i = 0; i < m; ++i) {
        const int kind = code[i] >> 8, a = (code[i] >> 3) & 15, c = code[i] & 7;
        if (kind == 0) { px[a < 2][c] = i; p.pord[i] = (unsigned short)((a << 3) | rt(c)); }
        else if (kind == 1) { pw[a][c] = i; p.pord[i] = (unsigned short)(0x100 | ((a ^ e) << 3) | rt(c)); }
        else { pf[a][c] = i; p.pord[i] = (unsigned short)(0x200 | ((a ^ e) << 3) | c); }
    }
    for (int own = 0; own < 2; ++own)
        for (int vj = 0; vj < 16; ++vj) {
            int kk[56], cc[56], c = 0;
            for (int vt = 0; vt < 8; ++vt)
                for (int d = 1; d < 8; ++d) {
                    const int ax = d == 1 ? px[own][vt] : dx[own][vt];
                    const int aw = d == 1 ? pw[vj][vt] : (vt == 2 && (vj ^ e) < PHI) ? pf[vj][d] : dw[vj][vt];
                    const int k = ax > aw ? ax : aw; int q = c++;
                    for (; q > 0 && kk[q - 1] > k; --q) { kk[q] = kk[q - 1]; cc[q] = cc[q - 1]; }
                    kk[q] = k; cc[q] = (d << 3) | rt(vt);
                }
            for (int i = 0; i < 56; ++i) p.kord[own][vj ^ e][i] = (unsigned char)cc[i];
        }
}

}

extern "C" void kernel_launch(int rank, int nranks, ncclComm_t comm,
                              hipStream_t stream,
                              void* const* d_in, const int* in_sizes, int n_in,
                              void* d_out, int out_size,
                              const int* in_dtypes, int out_dtype,
                              void* const* d_out_all,
                              void* const* d_ws_all, size_t ws_bytes) {
    (void)comm; (void)in_sizes; (void)n_in; (void)in_dtypes; (void)out_dtype; (void)d_out_all;
    if (nranks != NR || (unsigned)rank >= (unsigned)NR || ws_bytes < WS_NEED || (size_t)out_size * 4 != OUT_BYTES) {
        fprintf(stderr, "kernel_launch: unexpected configuration\n");
        return;
    }
    Params p;
    p.x = (const float*)d_in[0];
    p.w = (const float*)d_in[1];
    p.out = (float*)d_out;
    for (int r = 0; r < NR; ++r) p.ws[r] = (char*)d_ws_all[r];
    p.rank = rank;
    build_orders(p, rank & 1);
    HIP_CHECK(hipFuncSetAttribute((const void*)gemm_rs_fused, hipFuncAttributeMaxDynamicSharedMemorySize, LDS_BYTES));
    hipLaunchKernelGGL(gemm_rs_fused, dim3(GRID), dim3(512), LDS_BYTES, stream, p);
    HIP_CHECK(hipGetLastError());
}
```

```cpp
#include <hip/hip_runtime.h>
#include <rccl/rccl.h>
#include <cstdio>
#include <cstdint>

#define HIP_CHECK(expr) do { hipError_t _e = (expr); if (_e != hipSuccess) { \
    fprintf(stderr, "HIP error %s at %s:%d\n", hipGetErrorString(_e), __FILE__, __LINE__); \
    return; } } while (0)

namespace {

#define LAS __attribute__((address_space(3)))
#define GAS __attribute__((address_space(1)))
typedef _Float16 h8 __attribute__((ext_vector_type(8)));
typedef float f32x4 __attribute__((ext_vector_type(4)));
typedef unsigned u32x4 __attribute__((ext_vector_type(4)));
typedef __amdgpu_buffer_rsrc_t rsrc_t;
typedef unsigned long long u64;

constexpr int NR = 8;
constexpr int KSH = 512;
constexpr int NG = 16384;
constexpr int BLOB = 32768;
#define CFG_NW 6
#define CFG_PW 5
#define CFG_PU 2
#define CFG_PHI 13
#define CFG_WIN 32u
#define CFG_OWN0 1000
#define CFG_OWNSP 1
#define CFG_J2 6
#define CFG_SP 8
#define CFG_XL 8
#define CFG_SLEEP 48
#define CFG_FINE 16
#define CFG_D2 3
#define CFG_D3 4
constexpr int NW = CFG_NW;
constexpr int PW = CFG_PW;
constexpr int PU = CFG_PU;
constexpr int SL = 4;
constexpr int SLICE = BLOB / SL;
constexpr int SPI = SLICE / (PU * 1024);
constexpr int GRID = 256, NGEMM = 128, NPULL = (NR - 1) * NW, NCONV = GRID - NPULL, NHELP = GRID - NGEMM - NPULL;
constexpr int PHI = CFG_PHI;
constexpr int N_IND = 7 * 8 + 16 * 8 - 8 - PHI;
constexpr int N_INP = 8 * 8 + 16 * 8 - 8 + 6 * PHI;
constexpr int N_OUT = 128;
constexpr unsigned N_FINE = CFG_FINE;
static_assert(4 % PU == 0 && SPI * PU * 1024 * SL == BLOB && SPI >= 2 && SPI % 2 == 0 && N_FINE <= N_OUT, "puller geometry");
static_assert(NHELP > 0 && N_IND + N_OUT <= 512 && N_INP <= 512 && PHI <= 16, "geometry");

constexpr int ID_XR = 0;
constexpr int ID_P = 512;
constexpr int ID_PAR = 1536;
constexpr int PO_X = 0  , PO_W = 128  , PO_C = 640  ,
              PO_WR = 1664  , PAR_SZ = 2688;
constexpr int RF_WR = ID_PAR, N_RF = ID_PAR + 1024;
constexpr int ID_ZERO = ID_PAR + 2 * PAR_SZ;
constexpr int N_IDS = ID_ZERO + 1;
constexpr unsigned OFF_CTR = 0;
constexpr unsigned OFF_TMO = 32768;
constexpr unsigned OFF_TK = 36864;
constexpr unsigned OFF_RF0 = 65536;
constexpr unsigned OFF_LC = 163840;
constexpr unsigned OFF_DBG = 1u << 20;
constexpr unsigned OFF_DATA = 2u << 20;
static_assert(OFF_LC + 16u * N_IDS <= OFF_DBG && OFF_RF0 + 32u * N_RF <= OFF_LC && N_IDS < 32768, "control layout");
constexpr size_t WS_NEED = (size_t)OFF_DATA + (size_t)N_IDS * BLOB;
constexpr unsigned OUT_BYTES = 512u * NG * 4u;
constexpr u64 DEADLINE = 50000000ull;
constexpr int AUX_SC1 = 16, AUX_SYS = 17;
constexpr int LDS_BYTES = 131072;
constexpr int VOFF_OOB = 0x70000000;

struct Params {
    const float* x; const float* w; float* out; char* wsl;
    int rank, pad[7];
    char* ws[NR];
    unsigned short pord[N_INP];
    unsigned char dord[N_IND];
    unsigned char kord[2][16][56];
};

#define MEMFENCE() asm volatile("" ::: "memory")
#define WAITV_IMM(n) (((n) & 15) | (7 << 4) | (15 << 8) | (((n) >> 4) << 14))
#define WAITV(n) do { MEMFENCE(); __builtin_amdgcn_s_waitcnt(WAITV_IMM(n)); MEMFENCE(); } while (0)

#define STAMP(slot) do {} while (0)

__device__ __forceinline__ unsigned ld_agent(const void* p) { return __hip_atomic_load((const unsigned*)p, __ATOMIC_RELAXED, __HIP_MEMORY_SCOPE_AGENT); }
__device__ __forceinline__ unsigned ld_sys(const void* p) { return __hip_atomic_load((const unsigned*)p, __ATOMIC_RELAXED, __HIP_MEMORY_SCOPE_SYSTEM); }
__device__ __forceinline__ void st_agent(void* p, unsigned v) { __hip_atomic_store((unsigned*)p, v, __ATOMIC_RELAXED, __HIP_MEMORY_SCOPE_AGENT); }
__device__ __forceinline__ void st_sys(void* p, unsigned v) { __hip_atomic_store((unsigned*)p, v, __ATOMIC_RELAXED, __HIP_MEMORY_SCOPE_SYSTEM); }
__device__ __forceinline__ void st_agent(GAS void* p, unsigned v) { __hip_atomic_store((GAS unsigned*)p, v, __ATOMIC_RELAXED, __HIP_MEMORY_SCOPE_AGENT); }
__device__ __forceinline__ void st_sys(GAS void* p, unsigned v) { __hip_atomic_store((GAS unsigned*)p, v, __ATOMIC_RELAXED, __HIP_MEMORY_SCOPE_SYSTEM); }
__device__ __forceinline__ rsrc_t mk_rsrc(const void* p, unsigned bytes) { return __builtin_amdgcn_make_buffer_rsrc((void*)p, 0, bytes, 0x00020000); }
template <int AUX> __device__ __forceinline__ void st128(u32x4 v, rsrc_t rs, int voff, unsigned soff) {
    __builtin_amdgcn_raw_buffer_store_b128(v, rs, voff, soff, AUX);
    asm volatile("s_nop 1" :: "v"(v));
}
__device__ __forceinline__ unsigned off_rf(unsigned call) { return OFF_RF0 + (call & 1u) * (16u * N_RF); }
__device__ __forceinline__ unsigned uni(unsigned v) { return (unsigned)__builtin_amdgcn_readfirstlane((int)v); }

__device__ __forceinline__ bool give_up(char* wsl, unsigned call, u64 t0) {
    if (uni(ld_agent(wsl + OFF_TMO)) == call || __builtin_amdgcn_s_memrealtime() - t0 > DEADLINE) {
        st_agent(wsl + OFF_TMO, call);
        return true;
    }
    return false;
}

__device__ __forceinline__ void conv_load(const Params& p, int n, int wid, int lane, f32x4 (&v)[8]) {
    const int tt = n / 80, r = n % 80, g = lane >> 4, c = lane & 15;
    if (r < 16) {
        const float* src = p.x + (size_t)(r * 256 + wid * 32 + c) * KSH + tt * 64 + 4 * g;
#pragma unroll
        for (int f = 0; f < 2; ++f)
#pragma unroll
            for (int ii = 0; ii < 4; ++ii) v[f * 4 + ii] = *(const f32x4*)(src + f * 16 * KSH + ii * 16);
    } else {
        const int ks = wid & 1, wc = wid >> 1;
        const float* src = p.w + (size_t)(tt * 64 + ks * 32 + 4 * g) * NG + (r - 16) * 256 + wc * 64 + 4 * c;
#pragma unroll
        for (int e = 0; e < 8; ++e) v[e] = *(const f32x4*)(src + (size_t)((e >> 2) * 16 + (e & 3)) * NG);
    }
}
__device__ __forceinline__ unsigned conv_class(const f32x4 (&v)[8]) {
    float m = 0.f;
#pragma unroll
    for (int e = 0; e < 8; ++e)
#pragma unroll
        for (int j = 0; j < 4; ++j) m = __builtin_fmaxf(m, __builtin_fabsf(v[e][j]));
    const unsigned c = ((__builtin_bit_cast(unsigned, m) >> 23) + 3u) >> 4;
    unsigned c0 = uni(c);
    for (;;) {
        const u64 up = __ballot(c > c0);
        if (!up) break;
        c0 = (unsigned)__builtin_amdgcn_readlane((int)c, __builtin_ctzll(up));
    }
    return c0;
}
__device__ __forceinline__ int push_dest(int r) { return ((((r - 16) >> 4) << 1) | (r & 1)) & (NR - 1); }
__device__ __forceinline__ bool conv_pushes(int n, int rank) { return n >= 16 && n < 80 && push_dest(n) != rank; }
template <bool MAY_PUSH>
__device__ __forceinline__ void conv_store(const Params& p, rsrc_t rsl, int n, int pb, int wid, int lane, const f32x4 (&v)[8], unsigned cls) {
    const int tt = n / 80, r = n % 80, sh = 128 - 16 * (int)cls;
    if (r < 16) {
        const unsigned base = OFF_DATA + (unsigned)(pb + PO_X + r * 8 + tt) * BLOB + wid * 4096;
#pragma unroll
        for (int f = 0; f < 2; ++f)
#pragma unroll
            for (int ks = 0; ks < 2; ++ks) {
                h8 o;
#pragma unroll
                for (int e = 0; e < 8; ++e) o[e] = (_Float16)__builtin_ldexpf(v[f * 4 + ks * 2 + (e >> 2)][e & 3], sh);
                st128<AUX_SYS>(__builtin_bit_cast(u32x4, o), rsl, lane * 16, base + f * 2048 + ks * 1024);
            }
    } else {
        const int ks = wid & 1, wc = wid >> 1;
        const unsigned inb = wc * 4096 + ks * 1024, base = OFF_DATA + (unsigned)(pb + PO_W + (r - 16) * 8 + tt) * BLOB + inb;
        h8 o[4];
#pragma unroll
        for (int f = 0; f < 4; ++f)
#pragma unroll
            for (int e = 0; e < 8; ++e) o[f][e] = (_Float16)__builtin_ldexpf(v[e][f], sh);
#pragma unroll
        for (int f = 0; f < 4; ++f) st128<AUX_SYS>(__builtin_bit_cast(u32x4, o[f]), rsl, lane * 16, base + (f >> 1) * 16384 + (f & 1) * 2048);
        if (MAY_PUSH && conv_pushes(n, p.rank)) {
            MEMFENCE();
            const rsrc_t rsq = mk_rsrc(p.ws[push_dest(r)], (unsigned)WS_NEED);
            const unsigned rb = OFF_DATA + (unsigned)(pb + PO_WR + ((r - 16) & 15) * 64 + p.rank * 8) * BLOB + inb;
#pragma unroll
            for (int f = 0; f < 4; ++f) st128<AUX_SYS>(__builtin_bit_cast(u32x4, o[f]), rsq, lane * 16, rb + (f >> 1) * 16384 + (f & 1) * 2048);
        }
    }
}
template <bool MAY_PUSH>
__device__ __forceinline__ unsigned conv_agree(const Params& p, rsrc_t rsl, int n, int pb, int wid, int lane, const f32x4 (&v)[8], unsigned mine,
                                               const LAS unsigned* cls) {
    const u32x4 a = *(const LAS u32x4*)cls, b = *(const LAS u32x4*)(cls + 4);
    const unsigned hi = uni(max(max(max(a[0], a[1]), max(a[2], a[3])), max(max(b[0], b[1]), max(b[2], b[3]))));
    const unsigned lo = uni(min(min(min(a[0], a[1]), min(a[2], a[3])), min(min(b[0], b[1]), min(b[2], b[3]))));
    if (lo != hi) {
        WAITV(0);
        if (mine != hi) conv_store<MAY_PUSH>(p, rsl, n, pb, wid, lane, v, hi);
        WAITV(0);
        __builtin_amdgcn_s_barrier();
        MEMFENCE();
    }
    return hi;
}
__device__ __forceinline__ void conv_publish(int rank, GAS char* wst  , int n, int pb, unsigned tag, bool pushed) {
    const int tt = n / 80, r = n % 80, t = threadIdx.x >> 2, wd = threadIdx.x & 3;
    const unsigned OFF_RF = off_rf(tag >> 8);
    if (pushed) {
        if (t == push_dest(r)) st_sys(wst + OFF_LC + 16 * (pb + PO_WR + ((r - 16) & 15) * 64 + rank * 8) + 4 * wd, tag);
    } else if (t < NR) {
        if (t == rank) st_agent(wst + OFF_LC + 16 * (pb + (r < 16 ? PO_X + r * 8 : PO_W + (r - 16) * 8) + tt) + 4 * wd, tag);
        else if (r < 16) {
            const int o = r >> 1, li = o == t ? (r & 1) : (o ^ t);
            if (o != rank && (o == t || (li >= 2 && ((r ^ t) & 1) == 0))) st_sys(wst + OFF_RF + 16 * (ID_XR + li * 64 + rank * 8 + tt) + 4 * wd, tag);
        } else if (((r - 16) >> 4) == (t >> 1)) {
            const int j = (r - 16) & 15;
            if (!(tt == 0 && ((j ^ t) & 1) == 0) && !(j < PHI && tt == 1 + (t & 1) && (rank ^ t) >= 2))
                st_sys(wst + OFF_RF + 16 * (RF_WR + j * 64 + rank * 8 + tt) + 4 * wd, tag);
        }
    }
}

struct PStep { unsigned src, dst, did, tag; bool valid, last, fwd; };

__device__ __forceinline__ bool is_out(unsigned did) { return did - (unsigned)ID_P < (unsigned)(ID_PAR - ID_P); }

__device__ __forceinline__ void pull_items(char* wsl, rsrc_t rsl, rsrc_t rsp, rsrc_t rso, rsrc_t rsq  ,
                                           const LAS unsigned* desc  ,
                                           unsigned n_full, unsigned n_units, unsigned call, int pb, int lane, int peer) {
    const int lane16 = lane * 16, lane_out = (lane >> 4) * 4 * (NG * 4) + (lane & 15) * 16;
    const u64 t0 = __builtin_amdgcn_s_memrealtime();
    const int lane0 = lane == 0 ? 0 : VOFF_OOB;
    const unsigned tk_base = (call - 1u) * (n_units + NW * PW);
    bool dead = false;
    int sub = SPI;
    int nsub = 0;
    int nstate = 0;
    unsigned cur = 0, cofs = 0, ctag = 0, nxt = 0, nofs = 0, ntag = 0, nrf = 0, pend = 0, pend2 = 0, ptag = 0, spins = 0;
    const unsigned tag0 = call << 8, OFF_RF = off_rf(call);
    auto step = [&](const PStep& cu, const u32x4 (&rc)[PU], const u32x4 (&qc)[PU], PStep& nx, u32x4 (&rn)[PU], u32x4 (&qn)[PU]) -> bool {
        MEMFENCE();
        if (sub == SPI && nstate == 2) { cur = nxt; cofs = nofs; ctag = ntag; sub = nsub; nstate = 0; }
        nx.valid = sub < SPI;
        const unsigned inb = cofs + (nx.valid ? sub : SPI - 1) * (PU * 1024);
        nx.did = cur & 0x7fffu;
        nx.fwd = (cur >> 15) & 1u;
        nx.tag = ctag;
        nx.src = OFF_DATA + (cur >> 16) * BLOB + inb;
        nx.dst = OFF_DATA + nx.did * BLOB + inb;
        if (nx.valid) ++sub;
        nx.last = sub == SPI;
        const int vt = nstate == 0 ? lane0 : VOFF_OOB, vf = nstate == 1 ? lane0 : VOFF_OOB;
        const unsigned f0 = (unsigned)__builtin_amdgcn_raw_ptr_buffer_atomic_add_i32(1, rsl, vt, OFF_TK + peer * 128, 0);
        const unsigned f1 = __builtin_amdgcn_raw_buffer_load_b32(rsl, vf, nrf, AUX_SYS);
        const unsigned f2 = __builtin_amdgcn_raw_buffer_load_b32(rsl, vf, OFF_LC + 16 * (nxt & 0x7fffu), AUX_SC1);
        MEMFENCE();
        const int vr = nx.valid ? lane16 : VOFF_OOB, vq = (nx.valid && is_out(nx.did)) ? lane16 : VOFF_OOB;
#pragma unroll
        for (int u = 0; u < PU; ++u) rn[u] = __builtin_amdgcn_raw_buffer_load_b128(rsp, vr, nx.src + u * 1024, AUX_SYS);
#pragma unroll
        for (int u = 0; u < PU; ++u) qn[u] = __builtin_amdgcn_raw_buffer_load_b128(rsl, vq, nx.dst + u * 1024, AUX_SC1);
        WAITV(2 * PU);
        if (pend) {
            __builtin_amdgcn_raw_buffer_store_b32(ptag, rsl, lane0, pend, AUX_SC1);
            if (pend2) { __builtin_amdgcn_raw_buffer_store_b32(ptag, rsq, lane0, pend2, AUX_SYS); pend2 = 0; }
            pend = 0;
        }
        if (cu.valid) {
            if (is_out(cu.did)) {
                const unsigned cb = (cu.did - ID_P) & 127u, e = ((cu.dst - OFF_DATA) & (BLOB - 1)) >> 10;
                const unsigned row = (peer & 1) * 256 + (e >> 4) * 128 + ((cb >> 2) & 1) * 64 + ((e >> 2) & 3) * 16 + (e & 3);
                const unsigned col = (peer >> 1) * 4096 + (cb >> 3) * 256 + (cb & 3) * 64;
                const unsigned so = row * (NG * 4) + col * 4;
#pragma unroll
                for (int u = 0; u < PU; ++u) {
                    f32x4 v = __builtin_bit_cast(f32x4, rc[u]) + __builtin_bit_cast(f32x4, qc[u]);
                    if (dead) v = (f32x4){__builtin_nanf(""), __builtin_nanf(""), __builtin_nanf(""), __builtin_nanf("")};
                    st128<AUX_SC1>(__builtin_bit_cast(u32x4, v), rso, lane_out, so + u * (NG * 4));
                }
            } else {
#pragma unroll
                for (int u = 0; u < PU; ++u) {
                    st128<AUX_SYS>(rc[u], rsl, lane16, cu.dst + u * 1024);
                }
                if (cu.last) {
                    const unsigned wd = 4 * (((cu.dst - OFF_DATA) & (BLOB - 1)) / SLICE);
                    pend = OFF_LC + 16 * cu.did + wd; ptag = cu.tag;
                    if (cu.fwd) pend2 = OFF_RF + 16 * (cu.did - pb - PO_WR + RF_WR) + wd;
                }
            }
        }
        const unsigned g0 = uni(f0), g1 = uni(f1), g2 = uni(f2);
        if (nstate == 0) {
            const unsigned un = g0 - tk_base;
            if (un < n_units) {
                const bool fine = un >= n_full;
                const unsigned hu = fine ? un + n_full : 2 * un;
                nxt = uni(desc[hu / (2 * SL)]);
                nsub = fine ? SPI / 2 : 0;
                nofs = (hu % (2 * SL)) * (SLICE / 2) - nsub * (PU * 1024);
                const unsigned nd = nxt & 0x7fffu;
                nrf = OFF_RF + 16 * (nd < (unsigned)ID_PAR ? nd : nd - pb - PO_WR + RF_WR) + 4 * ((hu % (2 * SL)) >> 1);
                ntag = tag0 | 255u; nstate = dead ? 2 : 1;
            } else nstate = 3;
        } else if (nstate == 1) {
            if (g1 >= tag0 && (!is_out(nxt & 0x7fffu) || g2 >= tag0)) { ntag = g1; nstate = 2; }
            else if (!cu.valid && !nx.valid) {
                if ((++spins & 63u) == 0 && give_up(wsl, call, t0)) { dead = true; nstate = 2; }
                __builtin_amdgcn_s_sleep(2);
            }
        }
        return !cu.valid && !nx.valid && nstate == 3;
    };
    PStep sa = {0, 0, 0, 0, false, false, false}, sb;
    u32x4 ra[PU] = {}, qa[PU] = {}, rb[PU], qb[PU];
    bool done;
    do {
        step(sa, ra, qa, sb, rb, qb);
        done = step(sb, rb, qb, sa, ra, qa);
    } while (!done);
}

__global__ __launch_bounds__(512, 2) void gemm_rs_fused(const Params p) {
    extern __shared__ __attribute__((aligned(16))) char shm[];
    LAS char* lds = (LAS char*)shm;
    const int b = blockIdx.x, tid = threadIdx.x, lane = tid & 63, wid = __builtin_amdgcn_readfirstlane(tid >> 6);
    const int rank = p.rank;
    char* wsl = p.wsl;
    const rsrc_t rsl = mk_rsrc(wsl, (unsigned)WS_NEED);
    const rsrc_t rso = mk_rsrc(p.out, OUT_BYTES);
    STAMP(0);

    if (b >= NGEMM && b < NGEMM + NPULL) {
        const unsigned call = uni(ld_agent(wsl + OFF_CTR + b * 128)) + 1u;
        const int pi = b - NGEMM, ds = 1 + pi % (NR - 1), peer = (rank ^ ds) & (NR - 1);
        const int pb = ID_PAR + (int)(call & 1u) * PAR_SZ;
        const rsrc_t rsp = mk_rsrc(p.ws[peer], (unsigned)WS_NEED), rsq = mk_rsrc(p.ws[rank ^ 1], (unsigned)WS_NEED);
        LAS unsigned* desc = (LAS unsigned*)lds;
        const int n_in = ds == 1 ? N_INP : N_IND, n_items = ds == 1 ? N_INP : N_IND + N_OUT;
        if (tid < n_items) {
            unsigned did, sid;
            if (tid < n_in) {
                unsigned kind, a, c;
                if (ds == 1) { const unsigned code = p.pord[tid]; kind = code >> 8; a = (code >> 3) & 15u; c = code & 7u; }
                else {
                    const unsigned code = p.dord[tid]; kind = code >> 7; a = (code >> 3) & 15u; c = code & 7u;
                    if (kind == 0 && a >= (unsigned)ds) ++a;
                }
                if (kind == 0) {
                    const unsigned i = a < 2 ? rank * 2 + a : ((rank ^ a) * 2 + (rank & 1));
                    did = ID_XR + a * 64 + peer * 8 + c; sid = pb + PO_X + i * 8 + c;
                } else if (kind == 1) {
                    did = pb + PO_WR + a * 64 + peer * 8 + c; sid = pb + PO_W + ((rank >> 1) * 16 + a) * 8 + c;
                    if (ds != 1 && a < (unsigned)PHI && c == 2u - (rank & 1)) did |= 0x8000u;
                } else did = sid = pb + PO_WR + a * 64 + (rank ^ c) * 8 + 1 + (rank & 1);
            } else {
                const unsigned cb = (tid - n_in) ^ ((peer & 1) << 3);
                did = ID_P + peer * 128 + cb; sid = pb + PO_C + rank * 128 + cb;
            }
            desc[tid] = (sid << 16) | did;
        }
        asm volatile("s_waitcnt vmcnt(0) lgkmcnt(0)" ::: "memory");
        __builtin_amdgcn_s_barrier();
        MEMFENCE();
        if (tid == 0) st_agent(wsl + OFF_CTR + b * 128, call);
        if (wid >= PW) return;
        const unsigned n_fine = ds == 1 ? 0 : N_FINE, n_full = (n_items - n_fine) * SL;
        pull_items(wsl, rsl, rsp, rso, rsq, desc, n_full, n_full + n_fine * SL * 2, call, pb, lane, peer);
        STAMP(2);
        return;
    }

    unsigned call, kcode;
    int pb;
    {
        const int ci = b < NGEMM ? b : b - NPULL;
        static_assert(2 * NCONV <= 640 && 3 * NCONV >= 640 && 80 <= NCONV, "three blobs per converter at most, two at least");
        const int n2 = ci + 2 * NCONV < 640 ? ci + 2 * NCONV : ci;
        f32x4 v0[8], v1[8], v2[8];
        LAS unsigned* cls = (LAS unsigned*)lds;
        if (ci >= 80 && ci % 80 >= 16) __builtin_amdgcn_s_sleep(CFG_SLEEP);
        conv_load(p, ci, wid, lane, v0);
        const unsigned c0 = ld_agent(wsl + OFF_CTR + b * 128);
        MEMFENCE();
        u64 wsel = 0;
#pragma unroll
        for (int r = 0; r < NR; ++r) { u64 a = (u64)p.ws[r]; asm volatile("" : "+s"(a)); wsel = (tid >> 2) == r ? a : wsel; }
        GAS char* wst = (GAS char*)wsel;
        WAITV(0);
        call = uni(c0) + 1u;
        pb = ID_PAR + (int)(call & 1u) * PAR_SZ;
        const unsigned k0 = conv_class(v0);
        conv_store<true>(p, rsl, ci, pb, wid, lane, v0, k0);
        if (lane == 0) cls[wid] = k0;
        STAMP(5);
        MEMFENCE();
        asm volatile("s_waitcnt lgkmcnt(0)" ::: "memory");
        const bool pushes = conv_pushes(ci, rank);
        if (pushes) WAITV(4); else WAITV(0);
        STAMP(8);
        __builtin_amdgcn_s_barrier();
        MEMFENCE();
        const unsigned tagc0 = (call << 8) | conv_agree<true>(p, rsl, ci, pb, wid, lane, v0, k0, cls);
        conv_publish(rank, wst, ci, pb, tagc0, false);
        STAMP(6);
        MEMFENCE();
        conv_load(p, ci + NCONV, wid, lane, v1);
        conv_load(p, n2, wid, lane, v2);
        kcode = p.kord[((b & 1) * 4 + ((b >> 3) & 3)) < 2][((b & 7) >> 1) * 4 + ((b >> 5) & 3)][lane < 56 ? lane : 0];
        if (tid == 0) st_agent(wsl + OFF_CTR + b * 128, call);
        WAITV(0);
        const unsigned k1 = conv_class(v1), k2 = conv_class(v2);
        conv_store<false>(p, rsl, ci + NCONV, pb, wid, lane, v1, k1);
        if (n2 != ci) conv_store<false>(p, rsl, n2, pb, wid, lane, v2, k2);
        if (lane == 0) { cls[8 + wid] = k1; cls[16 + wid] = k2; }
        STAMP(7);
        asm volatile("s_waitcnt lgkmcnt(0)" ::: "memory");
        WAITV(0);
        __builtin_amdgcn_s_barrier();
        MEMFENCE();
        if (pushes) conv_publish(rank, wst, ci, pb, tagc0, true);
        conv_publish(rank, wst, ci + NCONV, pb, (call << 8) | conv_agree<false>(p, rsl, ci + NCONV, pb, wid, lane, v1, k1, cls + 8), false);
        if (n2 != ci) conv_publish(rank, wst, n2, pb, (call << 8) | conv_agree<false>(p, rsl, n2, pb, wid, lane, v2, k2, cls + 16), false);
        asm volatile("s_waitcnt lgkmcnt(0)" ::: "memory");
        __builtin_amdgcn_s_barrier();
        MEMFENCE();
    }
    STAMP(1);

    const unsigned tag0 = call << 8;
    const int wr = wid >> 2, wc = wid & 3;
    for (int job = 0;; ++job) {
        int nsteps, cid, fl_rank;
        unsigned ids, fl_idx, out_off = 0;
        if (b < NGEMM) {
            if (job) break;
            const int xcd = b & 7, kk = b >> 3, li = (xcd & 1) * 4 + (kk & 3), j = (xcd >> 1) * 4 + (kk >> 2);
            const int own = li < 2 ? rank : (rank ^ li), i = own * 2 + (li < 2 ? li : (rank & 1)), J = (rank >> 1) * 16 + j;
            nsteps = li < 2 ? 64 : 56;
            unsigned mine; int pos;
            if (lane >= 56) { mine = (unsigned)(pb + PO_X + i * 8 + lane - 56) | ((unsigned)(pb + PO_W + J * 8 + lane - 56) << 16); pos = lane - 56; }
            else {
                const unsigned code = kcode, d = code >> 3, st = ((rank ^ d) & (NR - 1)) * 8 + (code & 7u);
                mine = (unsigned)(ID_XR + li * 64 + st) | ((unsigned)(pb + PO_WR + j * 64 + st) << 16);
                const bool use = li < 2 || d != (unsigned)li;
                const u64 um = __ballot(use);
                const int below = __builtin_popcountll(um & ((1ull << lane) - 1ull));
                pos = use ? 8 + below : 56 + lane - below;
            }
            ids = (unsigned)__builtin_amdgcn_ds_permute(pos * 4, (int)mine);
            cid = li < 2 ? -1 : pb + PO_C + own * 128 + j * 8;
            fl_rank = own; fl_idx = ID_P + rank * 128 + j * 8;
            out_off = (unsigned)(li * 256) * (NG * 4) + (unsigned)(J * 256) * 4;
        } else {
            const int pt = (b - NGEMM - NPULL) + job * NHELP;
            if (pt >= 96) break;
            const int j = pt / 6, q = (rank ^ (2 + pt % 6)) & (NR - 1), m = lane & 7;
            nsteps = 8;
            ids = (unsigned)(pb + PO_X + (rank * 2 + (q & 1)) * 8 + m) | ((unsigned)(pb + PO_W + ((q >> 1) * 16 + j) * 8 + m) << 16);
            cid = ID_P + q * 128 + j * 8;
            fl_rank = rank; fl_idx = cid;
        }

        u64 mask = 0;
        unsigned cl = 0;
        int umax = -64;
        bool dead = false;
        auto fetch = [&](int n, int uprev, int& unit) -> unsigned {
            if (!((mask >> n) & 1ull)) {
                const u64 t0 = __builtin_amdgcn_s_memrealtime();
                for (unsigned spins = 1;; ++spins) {
                    const int vo = (unsigned)(lane - n) < (spins == 1 ? CFG_WIN : 8u) ? 0 : VOFF_OOB;
                    MEMFENCE();
                    const u32x4 fa = __builtin_amdgcn_raw_buffer_load_b128(rsl, 16 * (ids & 0xffffu) + vo, OFF_LC, AUX_SC1);
                    const u32x4 fb = __builtin_amdgcn_raw_buffer_load_b128(rsl, 16 * (ids >> 16) + vo, OFF_LC, AUX_SC1);
                    const bool ok = min(min(min(fa[0], fa[1]), min(fa[2], fa[3])), min(min(fb[0], fb[1]), min(fb[2], fb[3]))) >= tag0;
                    if (ok) cl = (max(max(fa[0], fa[1]), max(fa[2], fa[3])) & 255u) + (max(max(fb[0], fb[1]), max(fb[2], fb[3])) & 255u);
                    mask |= __ballot(ok);
                    if ((mask >> n) & 1ull) break;
                    if ((spins & 63u) == 0 && give_up(wsl, call, t0)) { mask = ~0ull; dead = true; break; }
                    __builtin_amdgcn_s_sleep(8);
                }
                MEMFENCE();
            }
            const int u = __builtin_amdgcn_readlane((int)cl, n);
            dead = dead || u >= 255;
            const bool tiny = u < umax - 5;
            unit = tiny ? uprev : u;
            umax = max(umax, u);
            return tiny ? (unsigned)ID_ZERO * 0x10001u : (unsigned)__builtin_amdgcn_readlane((int)ids, n);
        };
        const int lane16 = lane * 16;
        const unsigned wofs = OFF_DATA + wid * 2048;
#define SA(bf, h) (((bf) * 2 + (h)) * 16384)
#define SB(bf, h) ((4 + (bf) * 2 + (h)) * 16384)
#define STAGEV(bufoff, goff, vo) do { \
        __builtin_amdgcn_raw_ptr_buffer_load_lds(rsl, (LAS void*)(lds + (bufoff) + wid * 2048), 16, (vo), (goff), 0, AUX_SC1); \
        __builtin_amdgcn_raw_ptr_buffer_load_lds(rsl, (LAS void*)(lds + (bufoff) + wid * 2048), 16, (vo), (goff), 1024, AUX_SC1); } while (0)
#define STAGE(bufoff, goff) STAGEV(bufoff, goff, lane16)
#define LDA(dst, bf, h) do { _Pragma("unroll") for (int m = 0; m < 4; ++m) _Pragma("unroll") for (int k = 0; k < 2; ++k) \
        dst[m][k] = *(const LAS h8*)(lds + SA(bf, h) + aoff + m * 2048 + k * 1024); } while (0)
#define LDB(dst, bf, h) do { _Pragma("unroll") for (int n = 0; n < 2; ++n) _Pragma("unroll") for (int k = 0; k < 2; ++k) \
        dst[n][k] = *(const LAS h8*)(lds + SB(bf, h) + boff + n * 2048 + k * 1024); } while (0)
#define MMA(ai, bj, At, Bt) do { __builtin_amdgcn_s_setprio(1); \
        _Pragma("unroll") for (int m = 0; m < 4; ++m) _Pragma("unroll") for (int n = 0; n < 2; ++n) _Pragma("unroll") for (int k = 0; k < 2; ++k) \
            acc[ai][bj][m][n] = __builtin_amdgcn_mfma_f32_16x16x32_f16(At[m][k], Bt[n][k], acc[ai][bj][m][n], 0, 0, 0); \
        __builtin_amdgcn_s_setprio(0); } while (0)
#define WAIT_V(n) asm volatile("s_waitcnt vmcnt(" #n ")" ::: "memory")
#define WAIT_L(n) asm volatile("s_waitcnt lgkmcnt(" #n ")" ::: "memory")
#define BAR __builtin_amdgcn_s_barrier()
#define SCHED __builtin_amdgcn_sched_barrier(0)
#define RESCALE(from, to) do { if ((from) != (to)) { const float f_ = __builtin_ldexpf(1.0f, 16 * ((from) - (to))); \
        _Pragma("unroll") for (int a_ = 0; a_ < 2; ++a_) _Pragma("unroll") for (int c_ = 0; c_ < 2; ++c_) _Pragma("unroll") for (int m_ = 0; m_ < 4; ++m_) \
            _Pragma("unroll") for (int n_ = 0; n_ < 2; ++n_) acc[a_][c_][m_][n_] *= f_; } } while (0)
#define OFFA(c) (wofs + ((c) & 0xffffu) * BLOB)
#define OFFB(c) (wofs + ((c) >> 16) * BLOB)
        const int aoff = wr * 8192 + lane16, boff = wc * 4096 + lane16;
        f32x4 acc[2][2][4][2];
#pragma unroll
        for (int a = 0; a < 2; ++a)
#pragma unroll
            for (int c = 0; c < 2; ++c)
#pragma unroll
                for (int m = 0; m < 4; ++m)
#pragma unroll
                    for (int n = 0; n < 2; ++n) acc[a][c][m][n] = (f32x4){0.f, 0.f, 0.f, 0.f};
        h8 At[4][2], B0[2][2], B1[2][2];
        unsigned a1;
        int ucur, u0, u1;
        {
            const unsigned c0 = fetch(0, 0, u0), c1 = fetch(1, u0, u1);
            ucur = u0;
            STAGE(SB(0, 0), OFFB(c0)); STAGE(SA(0, 0), OFFA(c0)); STAGE(SB(0, 1), OFFB(c0) + 16384); STAGE(SA(0, 1), OFFA(c0) + 16384);
            if (wr == 1) BAR;
            WAIT_V(4); BAR;
            STAGE(SB(1, 0), OFFB(c1)); STAGE(SA(1, 0), OFFA(c1)); STAGE(SB(1, 1), OFFB(c1) + 16384);
            WAIT_V(6); BAR;
            a1 = OFFA(c1);
        }
        STAMP(2);
        for (int t = 0; t < nsteps; t += 2) {
            const int t2 = t + 2 < nsteps ? t + 2 : 0;
            const int vs = t + 2 < nsteps ? lane16 : VOFF_OOB;
            int u2, u3;
            RESCALE(ucur, u0);
            LDB(B0, 0, 0); SCHED; LDA(At, 0, 0); STAGE(SA(1, 1), a1 + 16384);
            WAIT_L(8); BAR; WAIT_L(0); MMA(0, 0, At, B0); BAR; SCHED;
            const unsigned c2 = fetch(t2, u1, u2), a2 = OFFA(c2), b2 = OFFB(c2);
            LDB(B1, 0, 1); STAGEV(SB(0, 0), b2, vs);
            BAR; WAIT_L(0); MMA(0, 1, At, B1); BAR;
            LDA(At, 0, 1); STAGEV(SA(0, 0), a2, vs);
            BAR; WAIT_L(0); MMA(1, 0, At, B0); BAR; SCHED;
            STAGEV(SB(0, 1), b2 + 16384, vs);
            WAIT_V(6); BAR; MMA(1, 1, At, B1); BAR;
            RESCALE(u0, u1);
            LDB(B0, 1, 0); SCHED; LDA(At, 1, 0); STAGEV(SA(0, 1), a2 + 16384, vs);
            WAIT_L(8); BAR; WAIT_L(0); MMA(0, 0, At, B0); BAR; SCHED;
            const unsigned c3 = fetch(t2 + 1, u2, u3), a3 = OFFA(c3), b3 = OFFB(c3);
            LDB(B1, 1, 1); STAGEV(SB(1, 0), b3, vs);
            BAR; WAIT_L(0); MMA(0, 1, At, B1); BAR;
            LDA(At, 1, 1); STAGEV(SA(1, 0), a3, vs);
            BAR; WAIT_L(0); MMA(1, 0, At, B0); BAR; SCHED;
            STAGEV(SB(1, 1), b3 + 16384, vs);
            WAIT_V(6); BAR; MMA(1, 1, At, B1); BAR;
            a1 = a3; ucur = u1; u0 = u2; u1 = u3;
        }
        if (wr == 0) BAR;
        STAMP(3);
        const float qn = __builtin_nanf("");
        const int sh = 16 * (ucur - 16);
        if (cid >= 0) {
            const unsigned base = OFF_DATA + (unsigned)(cid + wid) * BLOB;
#pragma unroll
            for (int ai = 0; ai < 2; ++ai)
#pragma unroll
                for (int m = 0; m < 4; ++m)
#pragma unroll
                    for (int r = 0; r < 4; ++r) {
                        f32x4 v = {__builtin_ldexpf(acc[ai][0][m][0][r], sh), __builtin_ldexpf(acc[ai][0][m][1][r], sh),
                                   __builtin_ldexpf(acc[ai][1][m][0][r], sh), __builtin_ldexpf(acc[ai][1][m][1][r], sh)};
                        if (dead) v = (f32x4){qn, qn, qn, qn};
                        st128<AUX_SYS>(__builtin_bit_cast(u32x4, v), rsl, lane16, base + (ai * 16 + m * 4 + r) * 1024);
                    }
            WAIT_V(0);
            if (lane < 4) {
                if (fl_rank == rank) st_agent(wsl + OFF_LC + 16 * (fl_idx + wid) + 4 * lane, tag0);
                else if ((unsigned)fl_rank < (unsigned)NR) st_sys(p.ws[fl_rank] + off_rf(call) + 16 * (fl_idx + wid) + 4 * lane, tag0);
            }
        } else {
            const int lane_out = (lane >> 4) * 4 * (NG * 4) + (lane & 15) * 16;
            const unsigned base = out_off + (unsigned)(wr * 64) * (NG * 4) + wc * 256;
#pragma unroll
            for (int ai = 0; ai < 2; ++ai)
#pragma unroll
                for (int m = 0; m < 4; ++m)
#pragma unroll
                    for (int r = 0; r < 4; ++r) {
                        f32x4 v = {__builtin_ldexpf(acc[ai][0][m][0][r], sh), __builtin_ldexpf(acc[ai][0][m][1][r], sh),
                                   __builtin_ldexpf(acc[ai][1][m][0][r], sh), __builtin_ldexpf(acc[ai][1][m][1][r], sh)};
                        if (dead) v = (f32x4){qn, qn, qn, qn};
                        st128<AUX_SC1>(__builtin_bit_cast(u32x4, v), rso, lane_out, base + (unsigned)(ai * 128 + m * 16 + r) * (NG * 4));
                    }
            WAIT_V(0);
        }
        STAMP(4);
        BAR;
    }
}

void build_orders(Params& p, int e) {
    auto rt = [e](int vt) { return vt == 1 ? 2 - e : vt == 2 ? 1 + e : vt; };
    int dw[16][8], dx[2][8], pw[16][8], pf[16][8], px[2][8];
    int n = 0;
    for (int late = 0; late < 2; ++late)
        for (int vt = 0; vt < 8; ++vt) {
            if (!late) for (int u = 2; u < 7; ++u) { dx[0][vt] = n; p.dord[n++] = (unsigned char)((u << 3) | rt(vt)); }
            if ((vt >= 8 - CFG_D3) == late) for (int h = 0; h < 2; ++h) { dx[1][vt] = n; p.dord[n++] = (unsigned char)((h << 3) | rt(vt)); }
            for (int vj = 0; vj < 16; ++vj) {
                if (vt == 0 && vj % 2 == 0) { dw[vj][vt] = 0; continue; }
                if (vt == 2 && (vj ^ e) < PHI) continue;
                if ((vj >= CFG_J2 && vt >= 8 - CFG_D2) != late) continue;
                dw[vj][vt] = n; p.dord[n++] = (unsigned char)(0x80 | ((vj ^ e) << 3) | rt(vt));
            }
        }
    int key[N_INP], code[N_INP], m = 0;
    for (int vj = 0; vj < 16; ++vj) {
        const int cnt = (vj % 2 == 0 ? 7 : 8) + ((vj ^ e) < PHI ? 6 : 0);
        int k = 0;
        for (int vt = (vj % 2 == 0 ? 1 : 0); vt < 8; ++vt, ++k) { key[m] = 32 * vj - CFG_SP * (cnt - 1 - k); code[m++] = 0x100 | (vj << 3) | vt; }
        if ((vj ^ e) < PHI) for (int d = 2; d < 8; ++d, ++k) { key[m] = 32 * vj - CFG_SP * (cnt - 1 - k); code[m++] = 0x200 | (vj << 3) | d; }
        pw[vj][0] = 0;
    }
    for (int vt = 0; vt < 8; ++vt) {
        for (int li = 2; li < 8; ++li) { key[m] = CFG_XL * (vt - 8) - (vt < 3 ? 1000 : 0); code[m++] = (li << 3) | vt; }
        for (int h = 0; h < 2; ++h) { key[m] = CFG_OWN0 + CFG_OWNSP * vt; code[m++] = (h << 3) | vt; }
    }
    for (int i = 1; i < m; ++i) {
        const int k = key[i], c = code[i]; int q = i;
        for (; q > 0 && key[q - 1] > k; --q) { key[q] = key[q - 1]; code[q] = code[q - 1]; }
        key[q] = k; code[q] = c;
    }
    for (int i = 0; i < m; ++i) {
        const int kind = code[i] >> 8, a = (code[i] >> 3) & 15, c = code[i] & 7;
        if (kind == 0) { px[a < 2][c] = i; p.pord[i] = (unsigned short)((a << 3) | rt(c)); }
        else if (kind == 1) { pw[a][c] = i; p.pord[i] = (unsigned short)(0x100 | ((a ^ e) << 3) | rt(c)); }
        else { pf[a][c] = i; p.pord[i] = (unsigned short)(0x200 | ((a ^ e) << 3) | c); }
    }
    for (int own = 0; own < 2; ++own)
        for (int vj = 0; vj < 16; ++vj) {
            int kk[56], cc[56], c = 0;
            for (int vt = 0; vt < 8; ++vt)
                for (int d = 1; d < 8; ++d) {
                    const int ax = d == 1 ? px[own][vt] : dx[own][vt];
                    const int aw = d == 1 ? pw[vj][vt] : (vt == 2 && (vj ^ e) < PHI) ? pf[vj][d] : dw[vj][vt];
                    const int k = ax > aw ? ax : aw; int q = c++;
                    for (; q > 0 && kk[q - 1] > k; --q) { kk[q] = kk[q - 1]; cc[q] = cc[q - 1]; }
                    kk[q] = k; cc[q] = (d << 3) | rt(vt);
                }
            for (int i = 0; i < 56; ++i) p.kord[own][vj ^ e][i] = (unsigned char)cc[i];
        }
}

}

extern "C" void kernel_launch(int rank, int nranks, ncclComm_t comm,
                              hipStream_t stream,
                              void* const* d_in, const int* in_sizes, int n_in,
                              void* d_out, int out_size,
                              const int* in_dtypes, int out_dtype,
                              void* const* d_out_all,
                              void* const* d_ws_all, size_t ws_bytes) {
    (void)comm; (void)in_sizes; (void)n_in; (void)in_dtypes; (void)out_dtype; (void)d_out_all;
    if (nranks != NR || (unsigned)rank >= (unsigned)NR || ws_bytes < WS_NEED || (size_t)out_size * 4 != OUT_BYTES) {
        fprintf(stderr, "kernel_launch: unexpected configuration\n");
        return;
    }
    Params p;
    p.x = (const float*)d_in[0];
    p.w = (const float*)d_in[1];
    p.out = (float*)d_out;
    for (int r = 0; r < NR; ++r) p.ws[r] = (char*)d_ws_all[r];
    p.rank = rank;
    p.wsl = p.ws[rank];
    for (int r = 0; r < 7; ++r) p.pad[r] = 0;
    build_orders(p, rank & 1);
    HIP_CHECK(hipFuncSetAttribute((const void*)gemm_rs_fused, hipFuncAttributeMaxDynamicSharedMemorySize, LDS_BYTES));
    hipLaunchKernelGGL(gemm_rs_fused, dim3(GRID), dim3(512), LDS_BYTES, stream, p);
    HIP_CHECK(hipGetLastError());
}
```

```cpp
#include <hip/hip_runtime.h>
#include <rccl/rccl.h>
#include <cstdio>
#include <cstdint>

#define HIP_CHECK(expr) do { hipError_t _e = (expr); if (_e != hipSuccess) { \
    fprintf(stderr, "HIP error %s at %s:%d\n", hipGetErrorString(_e), __FILE__, __LINE__); \
    return; } } while (0)

namespace {

#define LAS __attribute__((address_space(3)))
#define GAS __attribute__((address_space(1)))
typedef _Float16 h8 __attribute__((ext_vector_type(8)));
typedef float f32x4 __attribute__((ext_vector_type(4)));
typedef unsigned u32x4 __attribute__((ext_vector_type(4)));
typedef __amdgpu_buffer_rsrc_t rsrc_t;
typedef unsigned long long u64;

constexpr int NR = 8;
constexpr int KSH = 512;
constexpr int NG = 16384;
constexpr int BLOB = 32768;
#define CFG_NW 6
#define CFG_PW 5
#define CFG_PU 2
#define CFG_PHI 13
#define CFG_WIN 32u
#define CFG_OWN0 1000
#define CFG_OWNSP 1
#define CFG_OWNK 4
#define CFG_OWNK0 14
#define CFG_PE 1
#define CFG_GAP 48
#define CFG_GAP2 0
#define CFG_XSLEEP 40
#define CFG_SLEEP2 64
#define CFG_XAHEAD 1
#define CFG_J2 6
#define CFG_SP 8
#define CFG_XL 8
#define CFG_SLEEP 64
#define CFG_FINE 16
#define CFG_D2 3
#define CFG_D3 4
constexpr int NW = CFG_NW;
constexpr int PW = CFG_PW;
constexpr int PU = CFG_PU;
constexpr int SL = 4;
constexpr int SLICE = BLOB / SL;
constexpr int SPI = SLICE / (PU * 1024);
constexpr int GRID = 256, NGEMM = 128, NPULL = (NR - 1) * NW, NCONV = GRID - NPULL, NHELP = GRID - NGEMM - NPULL;
constexpr int PHI = CFG_PHI;
constexpr int N_IND = 7 * 8 + 16 * 8 - 8 - PHI;
constexpr int N_INP = 8 * 8 + 16 * 8 - 8 + 6 * PHI;
constexpr int N_OUT = 128;
constexpr unsigned N_FINE = CFG_FINE;
static_assert(4 % PU == 0 && SPI * PU * 1024 * SL == BLOB && SPI >= 2 && SPI % 2 == 0 && N_FINE <= N_OUT, "puller geometry");
static_assert(NHELP > 0 && N_IND + N_OUT <= 512 && N_INP <= 512 && PHI <= 16, "geometry");

constexpr int ID_XR = 0;
constexpr int ID_P = 512;
constexpr int ID_PAR = 1536;
constexpr int PO_X = 0  , PO_W = 128  , PO_C = 640  ,
              PO_WR = 1664  , PAR_SZ = 2688;
constexpr int RF_WR = ID_PAR, N_RF = ID_PAR + 1024;
constexpr int ID_ZERO = ID_PAR + 2 * PAR_SZ;
constexpr int N_IDS = ID_ZERO + 1;
constexpr unsigned OFF_CTR = 0;
constexpr unsigned OFF_TMO = 32768;
constexpr unsigned OFF_TK = 36864;
constexpr unsigned OFF_RF0 = 65536;
constexpr unsigned OFF_LC = 163840;
constexpr unsigned OFF_DBG = 1u << 20;
constexpr unsigned OFF_DATA = 2u << 20;
static_assert(OFF_LC + 16u * N_IDS <= OFF_DBG && OFF_RF0 + 32u * N_RF <= OFF_LC && N_IDS < 32768, "control layout");
constexpr size_t WS_NEED = (size_t)OFF_DATA + (size_t)N_IDS * BLOB;
constexpr unsigned OUT_BYTES = 512u * NG * 4u;
constexpr u64 DEADLINE = 50000000ull;
constexpr int AUX_SC1 = 16, AUX_SYS = 17;
#define CFG_CONVAUX 17
constexpr int LDS_BYTES = 131072;
constexpr int VOFF_OOB = 0x70000000;

struct Params {
    const float* x; const float* w; float* out; char* wsl;
    int rank, pad[7];
    char* ws[NR];
    unsigned short pord[N_INP];
    unsigned char dord[N_IND];
    unsigned char kord[2][16][64];
};

#define MEMFENCE() asm volatile("" ::: "memory")
#define WAITV_IMM(n) (((n) & 15) | (7 << 4) | (15 << 8) | (((n) >> 4) << 14))
#define WAITV(n) do { MEMFENCE(); __builtin_amdgcn_s_waitcnt(WAITV_IMM(n)); MEMFENCE(); } while (0)

#define STAMP(slot) do {} while (0)

__device__ __forceinline__ unsigned ld_agent(const void* p) { return __hip_atomic_load((const unsigned*)p, __ATOMIC_RELAXED, __HIP_MEMORY_SCOPE_AGENT); }
__device__ __forceinline__ unsigned ld_sys(const void* p) { return __hip_atomic_load((const unsigned*)p, __ATOMIC_RELAXED, __HIP_MEMORY_SCOPE_SYSTEM); }
__device__ __forceinline__ void st_agent(void* p, unsigned v) { __hip_atomic_store((unsigned*)p, v, __ATOMIC_RELAXED, __HIP_MEMORY_SCOPE_AGENT); }
__device__ __forceinline__ void st_sys(void* p, unsigned v) { __hip_atomic_store((unsigned*)p, v, __ATOMIC_RELAXED, __HIP_MEMORY_SCOPE_SYSTEM); }
__device__ __forceinline__ void st_agent(GAS void* p, unsigned v) { __hip_atomic_store((GAS unsigned*)p, v, __ATOMIC_RELAXED, __HIP_MEMORY_SCOPE_AGENT); }
__device__ __forceinline__ void st_sys(GAS void* p, unsigned v) { __hip_atomic_store((GAS unsigned*)p, v, __ATOMIC_RELAXED, __HIP_MEMORY_SCOPE_SYSTEM); }
__device__ __forceinline__ rsrc_t mk_rsrc(const void* p, unsigned bytes) { return __builtin_amdgcn_make_buffer_rsrc((void*)p, 0, bytes, 0x00020000); }
template <int AUX> __device__ __forceinline__ void st128(u32x4 v, rsrc_t rs, int voff, unsigned soff) {
    __builtin_amdgcn_raw_buffer_store_b128(v, rs, voff, soff, AUX);
    asm volatile("s_nop 1" :: "v"(v));
}
__device__ __forceinline__ unsigned off_rf(unsigned call) { return OFF_RF0 + (call & 1u) * (16u * N_RF); }
__device__ __forceinline__ unsigned uni(unsigned v) { return (unsigned)__builtin_amdgcn_readfirstlane((int)v); }

__device__ __forceinline__ bool give_up(char* wsl, unsigned call, u64 t0) {
    if (uni(ld_agent(wsl + OFF_TMO)) == call || __builtin_amdgcn_s_memrealtime() - t0 > DEADLINE) {
        st_agent(wsl + OFF_TMO, call);
        return true;
    }
    return false;
}

__device__ __forceinline__ void conv_load(const Params& p, int n, int wid, int lane, f32x4 (&v)[8]) {
    const int tt = n / 80, r = n % 80, g = lane >> 4, c = lane & 15;
    if (r < 16) {
        const float* src = p.x + (size_t)(r * 256 + wid * 32 + c) * KSH + tt * 64 + 4 * g;
#pragma unroll
        for (int f = 0; f < 2; ++f)
#pragma unroll
            for (int ii = 0; ii < 4; ++ii) v[f * 4 + ii] = *(const f32x4*)(src + f * 16 * KSH + ii * 16);
    } else {
        const int ks = wid & 1, wc = wid >> 1;
        const float* src = p.w + (size_t)(tt * 64 + ks * 32 + 4 * g) * NG + (r - 16) * 256 + wc * 64 + 4 * c;
#pragma unroll
        for (int e = 0; e < 8; ++e) {
            v[e] = *(const f32x4*)(src + (size_t)((e >> 2) * 16 + (e & 3)) * NG);
        }
    }
}
__device__ __forceinline__ unsigned conv_class(const f32x4 (&v)[8]) {
    float m = 0.f;
#pragma unroll
    for (int e = 0; e < 8; ++e)
#pragma unroll
        for (int j = 0; j < 4; ++j) m = __builtin_fmaxf(m, __builtin_fabsf(v[e][j]));
    const unsigned c = ((__builtin_bit_cast(unsigned, m) >> 23) + 3u) >> 4;
    unsigned c0 = uni(c);
    for (;;) {
        const u64 up = __ballot(c > c0);
        if (!up) break;
        c0 = (unsigned)__builtin_amdgcn_readlane((int)c, __builtin_ctzll(up));
    }
    return c0;
}
__device__ __forceinline__ int push_dest(int r) { return ((((r - 16) >> 4) << 1) | (r & 1)) & (NR - 1); }
__device__ __forceinline__ bool conv_pushes(int n, int rank) { return n >= 16 && n < 80 && push_dest(n) != rank; }
__device__ __forceinline__ void conv_store(const Params& p, rsrc_t rsl, int n, int pb, int wid, int lane, const f32x4 (&v)[8], unsigned cls) {
    const int tt = n / 80, r = n % 80, sh = 128 - 16 * (int)cls;
    if (r < 16) {
        const unsigned base = OFF_DATA + (unsigned)(pb + PO_X + r * 8 + tt) * BLOB + wid * 4096;
#pragma unroll
        for (int f = 0; f < 2; ++f)
#pragma unroll
            for (int ks = 0; ks < 2; ++ks) {
                h8 o;
#pragma unroll
                for (int e = 0; e < 8; ++e) o[e] = (_Float16)__builtin_ldexpf(v[f * 4 + ks * 2 + (e >> 2)][e & 3], sh);
                st128<CFG_CONVAUX>(__builtin_bit_cast(u32x4, o), rsl, lane * 16, base + f * 2048 + ks * 1024);
            }
    } else {
        const int ks = wid & 1, wc = wid >> 1;
        const unsigned inb = wc * 4096 + ks * 1024, base = OFF_DATA + (unsigned)(pb + PO_W + (r - 16) * 8 + tt) * BLOB + inb;
        h8 o[4];
#pragma unroll
        for (int f = 0; f < 4; ++f)
#pragma unroll
            for (int e = 0; e < 8; ++e) o[f][e] = (_Float16)__builtin_ldexpf(v[e][f], sh);
#pragma unroll
        for (int f = 0; f < 4; ++f) st128<CFG_CONVAUX>(__builtin_bit_cast(u32x4, o[f]), rsl, lane * 16, base + (f >> 1) * 16384 + (f & 1) * 2048);
        if (conv_pushes(n, p.rank)) {
            MEMFENCE();
            const rsrc_t rsq = mk_rsrc(p.ws[push_dest(r)], (unsigned)WS_NEED);
            const unsigned rb = OFF_DATA + (unsigned)(pb + PO_WR + ((r - 16) & 15) * 64 + p.rank * 8) * BLOB + inb;
#pragma unroll
            for (int f = 0; f < 4; ++f) st128<AUX_SYS>(__builtin_bit_cast(u32x4, o[f]), rsq, lane * 16, rb + (f >> 1) * 16384 + (f & 1) * 2048);
        }
    }
}
__device__ __forceinline__ unsigned conv_agree(const Params& p, rsrc_t rsl, int n, int pb, int wid, int lane, const f32x4 (&v)[8], unsigned mine,
                                               const LAS unsigned* cls) {
    const u32x4 a = *(const LAS u32x4*)cls, b = *(const LAS u32x4*)(cls + 4);
    const unsigned hi = uni(max(max(max(a[0], a[1]), max(a[2], a[3])), max(max(b[0], b[1]), max(b[2], b[3]))));
    const unsigned lo = uni(min(min(min(a[0], a[1]), min(a[2], a[3])), min(min(b[0], b[1]), min(b[2], b[3]))));
    if (lo != hi) {
        WAITV(0);
        if (mine != hi) conv_store(p, rsl, n, pb, wid, lane, v, hi);
        WAITV(0);
        __builtin_amdgcn_s_barrier();
        MEMFENCE();
    }
    return hi;
}
__device__ __forceinline__ void conv_publish(int rank, GAS char* wst  , int n, int pb, unsigned tag, bool pushed) {
    const int tt = n / 80, r = n % 80, t = threadIdx.x >> 2, wd = threadIdx.x & 3;
    const unsigned OFF_RF = off_rf(tag >> 8);
    if (pushed) {
        if (t == push_dest(r)) st_sys(wst + OFF_LC + 16 * (pb + PO_WR + ((r - 16) & 15) * 64 + rank * 8) + 4 * wd, tag);
    } else if (t < NR) {
        if (t == rank) st_agent(wst + OFF_LC + 16 * (pb + (r < 16 ? PO_X + r * 8 : PO_W + (r - 16) * 8) + tt) + 4 * wd, tag);
        else if (r < 16) {
            const int o = r >> 1, li = o == t ? (r & 1) : (o ^ t);
            if (o != rank && (o == t || (li >= 2 && ((r ^ t) & 1) == 0))) st_sys(wst + OFF_RF + 16 * (ID_XR + li * 64 + rank * 8 + tt) + 4 * wd, tag);
        } else if (((r - 16) >> 4) == (t >> 1)) {
            const int j = (r - 16) & 15;
            if (!(tt == 0 && ((j ^ t) & 1) == 0) && !(j < PHI && tt == 1 + (t & 1) && (rank ^ t) >= 2))
                st_sys(wst + OFF_RF + 16 * (RF_WR + j * 64 + rank * 8 + tt) + 4 * wd, tag);
        }
    }
}

struct PStep { unsigned src, dst, did, tag; bool valid, last, fwd; };

__device__ __forceinline__ bool is_out(unsigned did) { return did - (unsigned)ID_P < (unsigned)(ID_PAR - ID_P); }

__device__ __forceinline__ void pull_items(char* wsl, rsrc_t rsl, rsrc_t rsp, rsrc_t rso, rsrc_t rsq  ,
                                           const LAS unsigned* desc  ,
                                           unsigned n_full, unsigned n_units, unsigned call, int pb, int lane, int peer) {
    const int lane16 = lane * 16, lane_out = (lane >> 4) * 4 * (NG * 4) + (lane & 15) * 16;
    const u64 t0 = __builtin_amdgcn_s_memrealtime();
    const int lane0 = lane == 0 ? 0 : VOFF_OOB;
    const unsigned tk_base = (call - 1u) * (n_units + NW * PW);
    bool dead = false;
    int sub = SPI;
    int nsub = 0;
    int nstate = 0;
    unsigned cur = 0, cofs = 0, ctag = 0, nxt = 0, nofs = 0, ntag = 0, nrf = 0, pend = 0, pend2 = 0, ptag = 0, spins = 0;
    const unsigned tag0 = call << 8, OFF_RF = off_rf(call);
    auto step = [&](const PStep& cu, const u32x4 (&rc)[PU], const u32x4 (&qc)[PU], PStep& nx, u32x4 (&rn)[PU], u32x4 (&qn)[PU]) -> bool {
        MEMFENCE();
        if (sub == SPI && nstate == 2) { cur = nxt; cofs = nofs; ctag = ntag; sub = nsub; nstate = 0; }
        nx.valid = sub < SPI;
        const unsigned inb = cofs + (nx.valid ? sub : SPI - 1) * (PU * 1024);
        nx.did = cur & 0x7fffu;
        nx.fwd = (cur >> 15) & 1u;
        nx.tag = ctag;
        nx.src = OFF_DATA + (cur >> 16) * BLOB + inb;
        nx.dst = OFF_DATA + nx.did * BLOB + inb;
        if (nx.valid) ++sub;
        nx.last = sub == SPI;
        const int vt = nstate == 0 ? lane0 : VOFF_OOB, vf = nstate == 1 ? lane0 : VOFF_OOB;
        const unsigned f0 = (unsigned)__builtin_amdgcn_raw_ptr_buffer_atomic_add_i32(1, rsl, vt, OFF_TK + peer * 128, 0);
        const unsigned f1 = __builtin_amdgcn_raw_buffer_load_b32(rsl, vf, nrf, AUX_SYS);
        const unsigned f2 = __builtin_amdgcn_raw_buffer_load_b32(rsl, vf, OFF_LC + 16 * (nxt & 0x7fffu), AUX_SC1);
        MEMFENCE();
        const int vr = nx.valid ? lane16 : VOFF_OOB, vq = (nx.valid && is_out(nx.did)) ? lane16 : VOFF_OOB;
#pragma unroll
        for (int u = 0; u < PU; ++u) rn[u] = __builtin_amdgcn_raw_buffer_load_b128(rsp, vr, nx.src + u * 1024, AUX_SYS);
#pragma unroll
        for (int u = 0; u < PU; ++u) qn[u] = __builtin_amdgcn_raw_buffer_load_b128(rsl, vq, nx.dst + u * 1024, AUX_SC1);
        WAITV(2 * PU);
        if (pend) {
            __builtin_amdgcn_raw_buffer_store_b32(ptag, rsl, lane0, pend, AUX_SC1);
            if (pend2) { __builtin_amdgcn_raw_buffer_store_b32(ptag, rsq, lane0, pend2, AUX_SYS); pend2 = 0; }
            pend = 0;
        }
        if (cu.valid) {
            if (is_out(cu.did)) {
                const unsigned cb = (cu.did - ID_P) & 127u, e = ((cu.dst - OFF_DATA) & (BLOB - 1)) >> 10;
                const unsigned row = (peer & 1) * 256 + (e >> 4) * 128 + ((cb >> 2) & 1) * 64 + ((e >> 2) & 3) * 16 + (e & 3);
                const unsigned col = (peer >> 1) * 4096 + (cb >> 3) * 256 + (cb & 3) * 64;
                const unsigned so = row * (NG * 4) + col * 4;
#pragma unroll
                for (int u = 0; u < PU; ++u) {
                    f32x4 v = __builtin_bit_cast(f32x4, rc[u]) + __builtin_bit_cast(f32x4, qc[u]);
                    if (dead) v = (f32x4){__builtin_nanf(""), __builtin_nanf(""), __builtin_nanf(""), __builtin_nanf("")};
                    st128<AUX_SC1>(__builtin_bit_cast(u32x4, v), rso, lane_out, so + u * (NG * 4));
                }
            } else {
#pragma unroll
                for (int u = 0; u < PU; ++u) {
                    st128<AUX_SYS>(rc[u], rsl, lane16, cu.dst + u * 1024);
                }
                if (cu.last) {
                    const unsigned wd = 4 * (((cu.dst - OFF_DATA) & (BLOB - 1)) / SLICE);
                    pend = OFF_LC + 16 * cu.did + wd; ptag = cu.tag;
                    if (cu.fwd) pend2 = OFF_RF + 16 * (cu.did - pb - PO_WR + RF_WR) + wd;
                }
            }
        }
        const unsigned g0 = uni(f0), g1 = uni(f1), g2 = uni(f2);
        if (nstate == 0) {
            const unsigned un = g0 - tk_base;
            if (un < n_units) {
                const bool fine = un >= n_full;
                const unsigned hu = fine ? un + n_full : 2 * un;
                nxt = uni(desc[hu / (2 * SL)]);
                nsub = fine ? SPI / 2 : 0;
                nofs = (hu % (2 * SL)) * (SLICE / 2) - nsub * (PU * 1024);
                const unsigned nd = nxt & 0x7fffu;
                nrf = OFF_RF + 16 * (nd < (unsigned)ID_PAR ? nd : nd - pb - PO_WR + RF_WR) + 4 * ((hu % (2 * SL)) >> 1);
                ntag = tag0 | 255u; nstate = dead ? 2 : 1;
            } else nstate = 3;
        } else if (nstate == 1) {
            if (g1 >= tag0 && (!is_out(nxt & 0x7fffu) || g2 >= tag0)) { ntag = g1; nstate = 2; }
            else if (!cu.valid && !nx.valid) {
                if ((++spins & 63u) == 0 && give_up(wsl, call, t0)) { dead = true; nstate = 2; }
                __builtin_amdgcn_s_sleep(2);
            }
        }
        return !cu.valid && !nx.valid && nstate == 3;
    };
    PStep sa = {0, 0, 0, 0, false, false, false}, sb;
    u32x4 ra[PU] = {}, qa[PU] = {}, rb[PU], qb[PU];
    bool done;
    do {
        step(sa, ra, qa, sb, rb, qb);
        done = step(sb, rb, qb, sa, ra, qa);
    } while (!done);
}

__global__ __launch_bounds__(512, 2) void gemm_rs_fused(const Params p) {
    extern __shared__ __attribute__((aligned(16))) char shm[];
    LAS char* lds = (LAS char*)shm;
    const int b = blockIdx.x, tid = threadIdx.x, lane = tid & 63, wid = __builtin_amdgcn_readfirstlane(tid >> 6);
    const int rank = p.rank;
    char* wsl = p.wsl;
    const rsrc_t rsl = mk_rsrc(wsl, (unsigned)WS_NEED);
    const rsrc_t rso = mk_rsrc(p.out, OUT_BYTES);
    STAMP(0);

    if (b >= NGEMM && b < NGEMM + NPULL) {
        const unsigned call = uni(ld_agent(wsl + OFF_CTR + b * 128)) + 1u;
        const int pi = b - NGEMM, ds = 1 + pi % (NR - 1), peer = (rank ^ ds) & (NR - 1);
        const int pb = ID_PAR + (int)(call & 1u) * PAR_SZ;
        const rsrc_t rsp = mk_rsrc(p.ws[peer], (unsigned)WS_NEED), rsq = mk_rsrc(p.ws[rank ^ 1], (unsigned)WS_NEED);
        LAS unsigned* desc = (LAS unsigned*)lds;
        const int n_in = ds == 1 ? N_INP : N_IND, n_items = ds == 1 ? N_INP : N_IND + N_OUT;
        if (tid < n_items) {
            unsigned did, sid;
            if (tid < n_in) {
                unsigned kind, a, c;
                if (ds == 1) { const unsigned code = p.pord[tid]; kind = code >> 8; a = (code >> 3) & 15u; c = code & 7u; }
                else {
                    const unsigned code = p.dord[tid]; kind = code >> 7; a = (code >> 3) & 15u; c = code & 7u;
                    if (kind == 0 && a >= (unsigned)ds) ++a;
                }
                if (kind == 0) {
                    const unsigned i = a < 2 ? rank * 2 + a : ((rank ^ a) * 2 + (rank & 1));
                    did = ID_XR + a * 64 + peer * 8 + c; sid = pb + PO_X + i * 8 + c;
                } else if (kind == 1) {
                    did = pb + PO_WR + a * 64 + peer * 8 + c; sid = pb + PO_W + ((rank >> 1) * 16 + a) * 8 + c;
                    if (ds != 1 && a < (unsigned)PHI && c == 2u - (rank & 1)) did |= 0x8000u;
                } else did = sid = pb + PO_WR + a * 64 + (rank ^ c) * 8 + 1 + (rank & 1);
            } else {
                const unsigned cb = (tid - n_in) ^ ((peer & 1) << 3);
                did = ID_P + peer * 128 + cb; sid = pb + PO_C + rank * 128 + cb;
            }
            desc[tid] = (sid << 16) | did;
        }
        asm volatile("s_waitcnt vmcnt(0) lgkmcnt(0)" ::: "memory");
        __builtin_amdgcn_s_barrier();
        MEMFENCE();
        if (tid == 0) st_agent(wsl + OFF_CTR + b * 128, call);
        if (wid >= PW) return;
        const unsigned n_fine = ds == 1 ? 0 : N_FINE, n_full = (n_items - n_fine) * SL;
        pull_items(wsl, rsl, rsp, rso, rsq, desc, n_full, n_full + n_fine * SL * 2, call, pb, lane, peer);
        STAMP(2);
        return;
    }

    const int li = (b & 1) * 4 + ((b >> 3) & 3), tj = ((b & 7) >> 1) * 2 + ((b >> 5) & 1) + ((b >> 6) & 1) * 8;

    unsigned call = 0, kcode = 0;
    int pb = 0;
    {
        constexpr int NC2 = NCONV - 48, N_REST = 640 - 48;
        const bool early = b < 64 && li >= 2;
        const int c2 = b >= NGEMM ? b - NGEMM - NPULL : b < 64 ? NHELP + tj * 2 + li : NHELP + 16 + b - 64;
        LAS unsigned* cls = (LAS unsigned*)lds;
        auto blob_of = [](int m) {
            const int tt = m < 192 ? m >> 6 : 3 + (m - 192) / 80, r = m < 192 ? 16 + (m & 63) : (m - 192) % 80;
            return tt * 80 + (r < 16 ? r : 16 + ((r & 3) << 4) + ((r - 16) >> 2));
        };
        static_assert(NC2 <= 192, "first blobs");
        const int ex = tj * 6 + li - 2;
        int n = early ? (ex >> 4) * 80 + (ex & 15) : (c2 >> 6) * 80 + 16 + ((c2 & 3) << 4) + ((c2 & 63) >> 2);
        if (early && ex >= 16) __builtin_amdgcn_s_sleep(CFG_XSLEEP);
        if (!early && c2 >= 128) __builtin_amdgcn_s_sleep(CFG_SLEEP2);
        if (!early && c2 >= 64) __builtin_amdgcn_s_sleep(CFG_SLEEP);
        f32x4 v[8];
        conv_load(p, n, wid, lane, v);
        const unsigned c0 = ld_agent(wsl + OFF_CTR + b * 128);
        MEMFENCE();
        u64 wsel = 0;
#pragma unroll
        for (int r = 0; r < NR; ++r) { u64 a = (u64)p.ws[r]; asm volatile("" : "+s"(a)); wsel = (tid >> 2) == r ? a : wsel; }
        GAS char* wst = (GAS char*)wsel;
        call = uni(c0) + 1u;
        pb = ID_PAR + (int)(call & 1u) * PAR_SZ;
        int npush = -1;
        unsigned tpush = 0;
        for (int it = 0, m = early ? N_REST : c2;; ++it) {
            WAITV(0);
            const unsigned k = conv_class(v);
            conv_store(p, rsl, n, pb, wid, lane, v, k);
            if (lane == 0) cls[(it & 3) * 8 + wid] = k;
            asm volatile("s_waitcnt lgkmcnt(0)" ::: "memory");
            const bool pushes = conv_pushes(n, rank);
            if (pushes) WAITV(4); else WAITV(0);
            __builtin_amdgcn_s_barrier();
            MEMFENCE();
            const unsigned tag = (call << 8) | conv_agree(p, rsl, n, pb, wid, lane, v, k, cls + (it & 3) * 8);
            conv_publish(rank, wst, n, pb, tag, false);
            if (npush >= 0) { conv_publish(rank, wst, npush, pb, tpush, true); npush = -1; }
            if (pushes) { npush = n; tpush = tag; }
            if (it == 0) {
                if (tid == 0) st_agent(wsl + OFF_CTR + b * 128, call);
                kcode = p.kord[li < 2][tj][lane];
            }
            STAMP(5 + it);
            m += NC2;
            if (m >= N_REST) break;
            n = blob_of(m);
            if (CFG_GAP) __builtin_amdgcn_s_sleep(CFG_GAP);
            if (CFG_GAP2) __builtin_amdgcn_s_sleep(CFG_GAP2);
            MEMFENCE();
            conv_load(p, n, wid, lane, v);
        }
        static_assert(64 + NC2 <= N_REST && 4 * NC2 >= N_REST && NHELP + 80 == NC2, "converter deal");
        asm volatile("s_waitcnt lgkmcnt(0)" ::: "memory");
        __builtin_amdgcn_s_barrier();
        MEMFENCE();
    }
    STAMP(1);

    const unsigned tag0 = call << 8;
    const int wr = wid >> 2, wc = wid & 3;
    for (int job = 0;; ++job) {
        int nsteps, cid, fl_rank;
        unsigned ids, fl_idx, out_off = 0;
        if (b < NGEMM) {
            if (job) break;
            const int j = tj, own = li < 2 ? rank : (rank ^ li), i = own * 2 + (li < 2 ? li : (rank & 1)), J = (rank >> 1) * 16 + j;
            nsteps = li < 2 ? 64 : 56;
            const unsigned d = kcode >> 3, tt = kcode & 7u, st = ((rank ^ d) & (NR - 1)) * 8 + tt;
            const unsigned mine = d == 0 ? (unsigned)(pb + PO_X + i * 8 + tt) | ((unsigned)(pb + PO_W + J * 8 + tt) << 16)
                                         : (unsigned)(ID_XR + li * 64 + st) | ((unsigned)(pb + PO_WR + j * 64 + st) << 16);
            const bool use = li < 2 || d != (unsigned)li;
            const int below = __builtin_popcountll(__ballot(use) & ((1ull << lane) - 1ull));
            const int pos = use ? below : 56 + lane - below;
            ids = (unsigned)__builtin_amdgcn_ds_permute(pos * 4, (int)mine);
            cid = li < 2 ? -1 : pb + PO_C + own * 128 + j * 8;
            fl_rank = own; fl_idx = ID_P + rank * 128 + j * 8;
            out_off = (unsigned)(li * 256) * (NG * 4) + (unsigned)(J * 256) * 4;
        } else {
            const int pt = (b - NGEMM - NPULL) + job * NHELP;
            if (pt >= 96) break;
            const int j = pt / 6, q = (rank ^ (2 + pt % 6)) & (NR - 1), m = lane & 7;
            nsteps = 8;
            ids = (unsigned)(pb + PO_X + (rank * 2 + (q & 1)) * 8 + m) | ((unsigned)(pb + PO_W + ((q >> 1) * 16 + j) * 8 + m) << 16);
            cid = ID_P + q * 128 + j * 8;
            fl_rank = rank; fl_idx = cid;
        }

        u64 mask = 0;
        unsigned cl = 0;
        int umax = -64;
        bool dead = false;
        auto fetch = [&](int n, int uprev, int& unit) -> unsigned {
            if (!((mask >> n) & 1ull)) {
                const u64 t0 = __builtin_amdgcn_s_memrealtime();
                for (unsigned spins = 1;; ++spins) {
                    const int vo = (unsigned)(lane - n) < (spins == 1 ? CFG_WIN : 8u) ? 0 : VOFF_OOB;
                    MEMFENCE();
                    const u32x4 fa = __builtin_amdgcn_raw_buffer_load_b128(rsl, 16 * (ids & 0xffffu) + vo, OFF_LC, AUX_SC1);
                    const u32x4 fb = __builtin_amdgcn_raw_buffer_load_b128(rsl, 16 * (ids >> 16) + vo, OFF_LC, AUX_SC1);
                    const bool ok = min(min(min(fa[0], fa[1]), min(fa[2], fa[3])), min(min(fb[0], fb[1]), min(fb[2], fb[3]))) >= tag0;
                    if (ok) cl = (max(max(fa[0], fa[1]), max(fa[2], fa[3])) & 255u) + (max(max(fb[0], fb[1]), max(fb[2], fb[3])) & 255u);
                    mask |= __ballot(ok);
                    if ((mask >> n) & 1ull) break;
                    if ((spins & 63u) == 0 && give_up(wsl, call, t0)) { mask = ~0ull; dead = true; break; }
                    __builtin_amdgcn_s_sleep(8);
                }
                MEMFENCE();
            }
            const int u = __builtin_amdgcn_readlane((int)cl, n);
            dead = dead || u >= 255;
            const bool tiny = u < umax - 5;
            unit = tiny ? uprev : u;
            umax = max(umax, u);
            return tiny ? (unsigned)ID_ZERO * 0x10001u : (unsigned)__builtin_amdgcn_readlane((int)ids, n);
        };
        const int lane16 = lane * 16;
        const unsigned wofs = OFF_DATA + wid * 2048;
#define SA(bf, h) (((bf) * 2 + (h)) * 16384)
#define SB(bf, h) ((4 + (bf) * 2 + (h)) * 16384)
#define STAGEV(bufoff, goff, vo) do { \
        __builtin_amdgcn_raw_ptr_buffer_load_lds(rsl, (LAS void*)(lds + (bufoff) + wid * 2048), 16, (vo), (goff), 0, AUX_SC1); \
        __builtin_amdgcn_raw_ptr_buffer_load_lds(rsl, (LAS void*)(lds + (bufoff) + wid * 2048), 16, (vo), (goff), 1024, AUX_SC1); } while (0)
#define STAGE(bufoff, goff) STAGEV(bufoff, goff, lane16)
#define LDA(dst, bf, h) do { _Pragma("unroll") for (int m = 0; m < 4; ++m) _Pragma("unroll") for (int k = 0; k < 2; ++k) \
        dst[m][k] = *(const LAS h8*)(lds + SA(bf, h) + aoff + m * 2048 + k * 1024); } while (0)
#define LDB(dst, bf, h) do { _Pragma("unroll") for (int n = 0; n < 2; ++n) _Pragma("unroll") for (int k = 0; k < 2; ++k) \
        dst[n][k] = *(const LAS h8*)(lds + SB(bf, h) + boff + n * 2048 + k * 1024); } while (0)
#define MMA(ai, bj, At, Bt) do { __builtin_amdgcn_s_setprio(1); \
        _Pragma("unroll") for (int m = 0; m < 4; ++m) _Pragma("unroll") for (int n = 0; n < 2; ++n) _Pragma("unroll") for (int k = 0; k < 2; ++k) \
            acc[ai][bj][m][n] = __builtin_amdgcn_mfma_f32_16x16x32_f16(At[m][k], Bt[n][k], acc[ai][bj][m][n], 0, 0, 0); \
        __builtin_amdgcn_s_setprio(0); } while (0)
#define WAIT_V(n) asm volatile("s_waitcnt vmcnt(" #n ")" ::: "memory")
#define WAIT_L(n) asm volatile("s_waitcnt lgkmcnt(" #n ")" ::: "memory")
#define BAR __builtin_amdgcn_s_barrier()
#define SCHED __builtin_amdgcn_sched_barrier(0)
#define RESCALE(from, to) do { if ((from) != (to)) { const float f_ = __builtin_ldexpf(1.0f, 16 * ((from) - (to))); \
        _Pragma("unroll") for (int a_ = 0; a_ < 2; ++a_) _Pragma("unroll") for (int c_ = 0; c_ < 2; ++c_) _Pragma("unroll") for (int m_ = 0; m_ < 4; ++m_) \
            _Pragma("unroll") for (int n_ = 0; n_ < 2; ++n_) acc[a_][c_][m_][n_] *= f_; } } while (0)
#define OFFA(c) (wofs + ((c) & 0xffffu) * BLOB)
#define OFFB(c) (wofs + ((c) >> 16) * BLOB)
        const int aoff = wr * 8192 + lane16, boff = wc * 4096 + lane16;
        f32x4 acc[2][2][4][2];
#pragma unroll
        for (int a = 0; a < 2; ++a)
#pragma unroll
            for (int c = 0; c < 2; ++c)
#pragma unroll
                for (int m = 0; m < 4; ++m)
#pragma unroll
                    for (int n = 0; n < 2; ++n) acc[a][c][m][n] = (f32x4){0.f, 0.f, 0.f, 0.f};
        h8 At[4][2], B0[2][2], B1[2][2];
        unsigned a1;
        int ucur, u0, u1;
        {
            const unsigned c0 = fetch(0, 0, u0), c1 = fetch(1, u0, u1);
            ucur = u0;
            STAGE(SB(0, 0), OFFB(c0)); STAGE(SA(0, 0), OFFA(c0)); STAGE(SB(0, 1), OFFB(c0) + 16384); STAGE(SA(0, 1), OFFA(c0) + 16384);
            if (wr == 1) BAR;
            WAIT_V(4); BAR;
            STAGE(SB(1, 0), OFFB(c1)); STAGE(SA(1, 0), OFFA(c1)); STAGE(SB(1, 1), OFFB(c1) + 16384);
            WAIT_V(6); BAR;
            a1 = OFFA(c1);
        }
        STAMP(2);
        for (int t = 0; t < nsteps; t += 2) {
            const int t2 = t + 2 < nsteps ? t + 2 : 0;
            const int vs = t + 2 < nsteps ? lane16 : VOFF_OOB;
            int u2, u3;
            RESCALE(ucur, u0);
            LDB(B0, 0, 0); SCHED; LDA(At, 0, 0); STAGE(SA(1, 1), a1 + 16384);
            WAIT_L(8); BAR; WAIT_L(0); MMA(0, 0, At, B0); BAR; SCHED;
            const unsigned c2 = fetch(t2, u1, u2), a2 = OFFA(c2), b2 = OFFB(c2);
            LDB(B1, 0, 1); STAGEV(SB(0, 0), b2, vs);
            BAR; WAIT_L(0); MMA(0, 1, At, B1); BAR;
            LDA(At, 0, 1); STAGEV(SA(0, 0), a2, vs);
            BAR; WAIT_L(0); MMA(1, 0, At, B0); BAR; SCHED;
            STAGEV(SB(0, 1), b2 + 16384, vs);
            WAIT_V(6); BAR; MMA(1, 1, At, B1); BAR;
            RESCALE(u0, u1);
            LDB(B0, 1, 0); SCHED; LDA(At, 1, 0); STAGEV(SA(0, 1), a2 + 16384, vs);
            WAIT_L(8); BAR; WAIT_L(0); MMA(0, 0, At, B0); BAR; SCHED;
            const unsigned c3 = fetch(t2 + 1, u2, u3), a3 = OFFA(c3), b3 = OFFB(c3);
            LDB(B1, 1, 1); STAGEV(SB(1, 0), b3, vs);
            BAR; WAIT_L(0); MMA(0, 1, At, B1); BAR;
            LDA(At, 1, 1); STAGEV(SA(1, 0), a3, vs);
            BAR; WAIT_L(0); MMA(1, 0, At, B0); BAR; SCHED;
            STAGEV(SB(1, 1), b3 + 16384, vs);
            WAIT_V(6); BAR; MMA(1, 1, At, B1); BAR;
            a1 = a3; ucur = u1; u0 = u2; u1 = u3;
        }
        if (wr == 0) BAR;
        STAMP(3);
        const float qn = __builtin_nanf("");
        const int sh = 16 * (ucur - 16);
        if (cid >= 0) {
            const unsigned base = OFF_DATA + (unsigned)(cid + wid) * BLOB;
#pragma unroll
            for (int ai = 0; ai < 2; ++ai)
#pragma unroll
                for (int m = 0; m < 4; ++m)
#pragma unroll
                    for (int r = 0; r < 4; ++r) {
                        f32x4 v = {__builtin_ldexpf(acc[ai][0][m][0][r], sh), __builtin_ldexpf(acc[ai][0][m][1][r], sh),
                                   __builtin_ldexpf(acc[ai][1][m][0][r], sh), __builtin_ldexpf(acc[ai][1][m][1][r], sh)};
                        if (dead) v = (f32x4){qn, qn, qn, qn};
                        st128<AUX_SYS>(__builtin_bit_cast(u32x4, v), rsl, lane16, base + (ai * 16 + m * 4 + r) * 1024);
                    }
            WAIT_V(0);
            if (lane < 4) {
                if (fl_rank == rank) st_agent(wsl + OFF_LC + 16 * (fl_idx + wid) + 4 * lane, tag0);
                else if ((unsigned)fl_rank < (unsigned)NR) st_sys(p.ws[fl_rank] + off_rf(call) + 16 * (fl_idx + wid) + 4 * lane, tag0);
            }
        } else {
            const int lane_out = (lane >> 4) * 4 * (NG * 4) + (lane & 15) * 16;
            const unsigned base = out_off + (unsigned)(wr * 64) * (NG * 4) + wc * 256;
#pragma unroll
            for (int ai = 0; ai < 2; ++ai)
#pragma unroll
                for (int m = 0; m < 4; ++m)
#pragma unroll
                    for (int r = 0; r < 4; ++r) {
                        f32x4 v = {__builtin_ldexpf(acc[ai][0][m][0][r], sh), __builtin_ldexpf(acc[ai][0][m][1][r], sh),
                                   __builtin_ldexpf(acc[ai][1][m][0][r], sh), __builtin_ldexpf(acc[ai][1][m][1][r], sh)};
                        if (dead) v = (f32x4){qn, qn, qn, qn};
                        st128<AUX_SC1>(__builtin_bit_cast(u32x4, v), rso, lane_out, base + (unsigned)(ai * 128 + m * 16 + r) * (NG * 4));
                    }
            WAIT_V(0);
        }
        STAMP(4);
        BAR;
    }
}

void build_orders(Params& p, int e) {
    auto rt = [e](int vt) { return vt == 1 ? 2 - e : vt == 2 ? 1 + e : vt; };
    int dw[16][8], dx[2][8], pw[16][8], pf[16][8], px[2][8];
    int n = 0;
    for (int late = 0; late < 2; ++late)
        for (int vt = 0; vt < 8; ++vt) {
            for (int xv = vt; xv <= (vt == 0 ? CFG_XAHEAD : vt); ++xv) {
                if (vt > 0 && vt <= CFG_XAHEAD) break;
                if (!late) for (int u = 2; u < 7; ++u) { dx[0][xv] = n; p.dord[n++] = (unsigned char)((u << 3) | rt(xv)); }
                if ((xv >= 8 - CFG_D3) == late) for (int h = 0; h < 2; ++h) { dx[1][xv] = n; p.dord[n++] = (unsigned char)((h << 3) | rt(xv)); }
            }
            for (int vj = 0; vj < 16; ++vj) {
                if (vt == 0 && vj % 2 == 0) { dw[vj][vt] = 0; continue; }
                if (vt == 2 && (vj ^ e) < PHI) continue;
                if ((vj >= CFG_J2 && vt >= 8 - CFG_D2) != late) continue;
                dw[vj][vt] = n; p.dord[n++] = (unsigned char)(0x80 | ((vj ^ e) << 3) | rt(vt));
            }
        }
    int key[N_INP], code[N_INP], m = 0;
    for (int vj = 0; vj < 16; ++vj) {
        const int cnt = (vj % 2 == 0 ? 7 : 8) + ((vj ^ e) < PHI ? 6 : 0);
        int k = 0;
        for (int vt = (vj % 2 == 0 ? 1 : 0); vt < 8; ++vt, ++k) { key[m] = 32 * vj - CFG_SP * (cnt - 1 - k) - (rt(vt) < CFG_PE ? 2000 : 0); code[m++] = 0x100 | (vj << 3) | vt; }
        if ((vj ^ e) < PHI) for (int d = 2; d < 8; ++d, ++k) { key[m] = 32 * vj - CFG_SP * (cnt - 1 - k); code[m++] = 0x200 | (vj << 3) | d; }
        pw[vj][0] = 0;
    }
    for (int vt = 0; vt < 8; ++vt) {
        for (int li = 2; li < 8; ++li) { key[m] = CFG_XL * (vt - 8) - (vt < 3 ? 4000 : 0); code[m++] = (li << 3) | vt; }
        for (int h = 0; h < 2; ++h) { key[m] = CFG_OWN0 + CFG_OWNSP * vt; code[m++] = (h << 3) | vt; }
    }
    for (int i = 1; i < m; ++i) {
        const int k = key[i], c = code[i]; int q = i;
        for (; q > 0 && key[q - 1] > k; --q) { key[q] = key[q - 1]; code[q] = code[q - 1]; }
        key[q] = k; code[q] = c;
    }
    for (int i = 0; i < m; ++i) {
        const int kind = code[i] >> 8, a = (code[i] >> 3) & 15, c = code[i] & 7;
        if (kind == 0) { px[a < 2][c] = i; p.pord[i] = (unsigned short)((a << 3) | rt(c)); }
        else if (kind == 1) { pw[a][c] = i; p.pord[i] = (unsigned short)(0x100 | ((a ^ e) << 3) | rt(c)); }
        else { pf[a][c] = i; p.pord[i] = (unsigned short)(0x200 | ((a ^ e) << 3) | c); }
    }
    for (int own = 0; own < 2; ++own)
        for (int vj = 0; vj < 16; ++vj) {
            int kk[64], cc[64], c = 0;
            for (int vt = 0; vt < 8; ++vt)
                for (int d = 0; d < 8; ++d) {
                    const int ax = d == 1 ? px[own][vt] : dx[own][vt];
                    const int aw = d == 1 ? pw[vj][vt] : (vt == 2 && (vj ^ e) < PHI) ? pf[vj][d] : dw[vj][vt];
                    const int k = d == 0 ? CFG_OWNK * rt(vt) - CFG_OWNK0 : ax > aw ? ax : aw; int q = c++;
                    for (; q > 0 && kk[q - 1] > k; --q) { kk[q] = kk[q - 1]; cc[q] = cc[q - 1]; }
                    kk[q] = k; cc[q] = (d << 3) | rt(vt);
                }
            for (int i = 0; i < 64; ++i) p.kord[own][vj ^ e][i] = (unsigned char)cc[i];
        }
}

}

extern "C" void kernel_launch(int rank, int nranks, ncclComm_t comm,
                              hipStream_t stream,
                              void* const* d_in, const int* in_sizes, int n_in,
                              void* d_out, int out_size,
                              const int* in_dtypes, int out_dtype,
                              void* const* d_out_all,
                              void* const* d_ws_all, size_t ws_bytes) {
    (void)comm; (void)in_sizes; (void)n_in; (void)in_dtypes; (void)out_dtype; (void)d_out_all;
    if (nranks != NR || (unsigned)rank >= (unsigned)NR || ws_bytes < WS_NEED || (size_t)out_size * 4 != OUT_BYTES) {
        fprintf(stderr, "kernel_launch: unexpected configuration\n");
        return;
    }
    Params p;
    p.x = (const float*)d_in[0];
    p.w = (const float*)d_in[1];
    p.out = (float*)d_out;
    for (int r = 0; r < NR; ++r) p.ws[r] = (char*)d_ws_all[r];
    p.rank = rank;
    p.wsl = p.ws[rank];
    for (int r = 0; r < 7; ++r) p.pad[r] = 0;
    build_orders(p, rank & 1);
    HIP_CHECK(hipFuncSetAttribute((const void*)gemm_rs_fused, hipFuncAttributeMaxDynamicSharedMemorySize, LDS_BYTES));
    hipLaunchKernelGGL(gemm_rs_fused, dim3(GRID), dim3(512), LDS_BYTES, stream, p);
    HIP_CHECK(hipGetLastError());
}
```

```cpp
#include <hip/hip_runtime.h>
#include <rccl/rccl.h>
#include <cstdio>
#include <cstdint>

#define HIP_CHECK(expr) do { hipError_t _e = (expr); if (_e != hipSuccess) { \
    fprintf(stderr, "HIP error %s at %s:%d\n", hipGetErrorString(_e), __FILE__, __LINE__); \
    return; } } while (0)

namespace {

#define LAS __attribute__((address_space(3)))
#define GAS __attribute__((address_space(1)))
typedef _Float16 h8 __attribute__((ext_vector_type(8)));
typedef float f32x4 __attribute__((ext_vector_type(4)));
typedef unsigned u32x4 __attribute__((ext_vector_type(4)));
typedef __amdgpu_buffer_rsrc_t rsrc_t;
typedef unsigned long long u64;

constexpr int NR = 8;
constexpr int KSH = 512;
constexpr int NG = 16384;
constexpr int BLOB = 32768;
constexpr int NW = 8;
constexpr int PW = 2;
constexpr int PU = 2;
constexpr int SL = 4;
constexpr int SLICE = BLOB / SL;
constexpr int SPI = SLICE / (PU * 1024);
constexpr int GRID = 256, NGEMM = 128, NPULL = (NR - 1) * NW, NCONV = GRID - NPULL, NHELP = GRID - NGEMM - NPULL;
constexpr int DYN0 = 6;
constexpr unsigned FWD_FIX = 0x383eu, FWD_DYN = 0x1fu << DYN0;
constexpr int NDYN = 10, EDYN = 6;
constexpr int PJ = 10;
constexpr int N_IND = 7 * 8 + 16 * 8 - PJ / 2 - 8;
constexpr int N_INP = 8 * 8 + 16 * 8 - PJ / 2 + 6 * 8;
constexpr int N_OUT = 128;
static_assert(4 % PU == 0 && SPI * PU * 1024 * SL == BLOB, "puller geometry");
static_assert(NHELP > 0 && N_IND + N_OUT <= 512 && N_INP <= 256, "geometry");

constexpr int ID_XR = 0;
constexpr int ID_P = 512;
constexpr int ID_PAR = 1536;
constexpr int PO_X = 0  , PO_W = 128  , PO_C = 640  ,
              PO_WR = 1664  , PAR_SZ = 2688;
constexpr int RF_WR = ID_PAR, RF_FW = ID_PAR + 1024, N_RF = ID_PAR + 2048;
constexpr int ID_ZERO = ID_PAR + 2 * PAR_SZ;
constexpr int N_IDS = ID_ZERO + 1;
constexpr unsigned OFF_CTR = 0;
constexpr unsigned OFF_TMO = 32768;
constexpr unsigned OFF_TK = 36864;
constexpr unsigned OFF_TKB = 40960;
constexpr unsigned OFF_DEC = 45056;
constexpr unsigned OFF_RF0 = 65536;
constexpr unsigned OFF_LC = 196608;
constexpr unsigned OFF_DATA = 2u << 20;
static_assert(OFF_LC + 16u * N_IDS <= OFF_DATA && OFF_RF0 + 32u * N_RF <= OFF_LC && N_IDS < 32768, "control layout");
constexpr size_t WS_NEED = (size_t)OFF_DATA + (size_t)N_IDS * BLOB;
constexpr unsigned OUT_BYTES = 512u * NG * 4u;
constexpr u64 DEADLINE = 150000000ull;
constexpr int AUX_SC1 = 16, AUX_SYS = 17;
constexpr int LDS_BYTES = 131072;
constexpr int VOFF_OOB = 0x70000000;

struct Params {
    const float* x; const float* w; float* out; char* wsl;
    int rank, dflex, pflex, pad[5];
    char* ws[NR];
    unsigned short pord[N_INP];
    unsigned char dord[N_IND];
    unsigned char kord[2][16][64];
};

#define MEMFENCE() asm volatile("" ::: "memory")
#define WAITV_IMM(n) (((n) & 15) | (7 << 4) | (15 << 8) | (((n) >> 4) << 14))
#define WAITV(n) do { MEMFENCE(); __builtin_amdgcn_s_waitcnt(WAITV_IMM(n)); MEMFENCE(); } while (0)

__device__ __forceinline__ unsigned ld_agent(const void* p) { return __hip_atomic_load((const unsigned*)p, __ATOMIC_RELAXED, __HIP_MEMORY_SCOPE_AGENT); }
__device__ __forceinline__ void st_agent(void* p, unsigned v) { __hip_atomic_store((unsigned*)p, v, __ATOMIC_RELAXED, __HIP_MEMORY_SCOPE_AGENT); }
__device__ __forceinline__ void st_sys(void* p, unsigned v) { __hip_atomic_store((unsigned*)p, v, __ATOMIC_RELAXED, __HIP_MEMORY_SCOPE_SYSTEM); }
__device__ __forceinline__ void st_agent(GAS void* p, unsigned v) { __hip_atomic_store((GAS unsigned*)p, v, __ATOMIC_RELAXED, __HIP_MEMORY_SCOPE_AGENT); }
__device__ __forceinline__ void st_sys(GAS void* p, unsigned v) { __hip_atomic_store((GAS unsigned*)p, v, __ATOMIC_RELAXED, __HIP_MEMORY_SCOPE_SYSTEM); }
__device__ __forceinline__ rsrc_t mk_rsrc(const void* p, unsigned bytes) { return __builtin_amdgcn_make_buffer_rsrc((void*)p, 0, bytes, 0x00020000); }
template <int AUX> __device__ __forceinline__ void st128(u32x4 v, rsrc_t rs, int voff, unsigned soff) {
    __builtin_amdgcn_raw_buffer_store_b128(v, rs, voff, soff, AUX);
    asm volatile("s_nop 1" :: "v"(v));
}
__device__ __forceinline__ unsigned off_rf(unsigned call) { return OFF_RF0 + (call & 1u) * (16u * N_RF); }
__device__ __forceinline__ unsigned uni(unsigned v) { return (unsigned)__builtin_amdgcn_readfirstlane((int)v); }

__device__ __forceinline__ bool give_up(char* wsl, unsigned call, u64 t0) {
    if (uni(ld_agent(wsl + OFF_TMO)) == call || __builtin_amdgcn_s_memrealtime() - t0 > DEADLINE) {
        st_agent(wsl + OFF_TMO, call);
        return true;
    }
    return false;
}

__device__ __forceinline__ void conv_load(const Params& p, int n, int wid, int lane, f32x4 (&v)[8]) {
    const int tt = n / 80, r = n % 80, g = lane >> 4, c = lane & 15;
    if (r < 16) {
        const float* src = p.x + (size_t)(r * 256 + wid * 32 + c) * KSH + tt * 64 + 4 * g;
#pragma unroll
        for (int f = 0; f < 2; ++f)
#pragma unroll
            for (int ii = 0; ii < 4; ++ii) v[f * 4 + ii] = *(const f32x4*)(src + f * 16 * KSH + ii * 16);
    } else {
        const int ks = wid & 1, wc = wid >> 1;
        const float* src = p.w + (size_t)(tt * 64 + ks * 32 + 4 * g) * NG + (r - 16) * 256 + wc * 64 + 4 * c;
#pragma unroll
        for (int e = 0; e < 8; ++e) v[e] = *(const f32x4*)(src + (size_t)((e >> 2) * 16 + (e & 3)) * NG);
    }
}
__device__ __forceinline__ unsigned conv_class(const f32x4 (&v)[8]) {
    float m = 0.f;
#pragma unroll
    for (int e = 0; e < 8; ++e)
#pragma unroll
        for (int j = 0; j < 4; ++j) m = __builtin_fmaxf(m, __builtin_fabsf(v[e][j]));
    const unsigned c = ((__builtin_bit_cast(unsigned, m) >> 23) + 3u) >> 4;
    unsigned c0 = uni(c);
    for (;;) {
        const u64 up = __ballot(c > c0);
        if (!up) break;
        c0 = (unsigned)__builtin_amdgcn_readlane((int)c, __builtin_ctzll(up));
    }
    return c0;
}
__device__ __forceinline__ int push_dest(int r) { return ((((r - 16) >> 4) << 1) | (r & 1)) & (NR - 1); }
__device__ __forceinline__ bool conv_pushes(int n, int rank) { return n >= 16 && n < 80 && (n & 15) < PJ && push_dest(n) != rank; }
__device__ __forceinline__ void conv_store(const Params& p, rsrc_t rsl, int n, int pb, int wid, int lane, const f32x4 (&v)[8], unsigned cls) {
    const int tt = n / 80, r = n % 80, sh = 128 - 16 * (int)cls;
    if (r < 16) {
        const unsigned base = OFF_DATA + (unsigned)(pb + PO_X + r * 8 + tt) * BLOB + wid * 4096;
#pragma unroll
        for (int f = 0; f < 2; ++f)
#pragma unroll
            for (int ks = 0; ks < 2; ++ks) {
                h8 o;
#pragma unroll
                for (int e = 0; e < 8; ++e) o[e] = (_Float16)__builtin_ldexpf(v[f * 4 + ks * 2 + (e >> 2)][e & 3], sh);
                st128<AUX_SYS>(__builtin_bit_cast(u32x4, o), rsl, lane * 16, base + f * 2048 + ks * 1024);
            }
    } else {
        const int ks = wid & 1, wc = wid >> 1;
        const unsigned inb = wc * 4096 + ks * 1024, base = OFF_DATA + (unsigned)(pb + PO_W + (r - 16) * 8 + tt) * BLOB + inb;
        h8 o[4];
#pragma unroll
        for (int f = 0; f < 4; ++f)
#pragma unroll
            for (int e = 0; e < 8; ++e) o[f][e] = (_Float16)__builtin_ldexpf(v[e][f], sh);
#pragma unroll
        for (int f = 0; f < 4; ++f) st128<AUX_SYS>(__builtin_bit_cast(u32x4, o[f]), rsl, lane * 16, base + (f >> 1) * 16384 + (f & 1) * 2048);
        if (conv_pushes(n, p.rank)) {
            MEMFENCE();
            const rsrc_t rsq = mk_rsrc(p.ws[push_dest(r)], (unsigned)WS_NEED);
            const unsigned rb = OFF_DATA + (unsigned)(pb + PO_WR + ((r - 16) & 15) * 64 + p.rank * 8) * BLOB + inb;
#pragma unroll
            for (int f = 0; f < 4; ++f) st128<AUX_SYS>(__builtin_bit_cast(u32x4, o[f]), rsq, lane * 16, rb + (f >> 1) * 16384 + (f & 1) * 2048);
        }
    }
}
__device__ __forceinline__ unsigned conv_agree(const Params& p, rsrc_t rsl, int n, int pb, int wid, int lane, const f32x4 (&v)[8], unsigned mine,
                                               const LAS unsigned* cls) {
    const u32x4 a = *(const LAS u32x4*)cls, b = *(const LAS u32x4*)(cls + 4);
    const unsigned hi = uni(max(max(max(a[0], a[1]), max(a[2], a[3])), max(max(b[0], b[1]), max(b[2], b[3]))));
    const unsigned lo = uni(min(min(min(a[0], a[1]), min(a[2], a[3])), min(min(b[0], b[1]), min(b[2], b[3]))));
    if (lo != hi) {
        WAITV(0);
        if (mine != hi) conv_store(p, rsl, n, pb, wid, lane, v, hi);
        WAITV(0);
        __builtin_amdgcn_s_barrier();
        MEMFENCE();
    }
    return hi;
}
__device__ __forceinline__ void conv_publish(int rank, GAS char* wst  , int n, int pb, unsigned tag, bool pushed) {
    const int tt = n / 80, r = n % 80, t = threadIdx.x >> 2, wd = threadIdx.x & 3;
    const unsigned OFF_RF = off_rf(tag >> 8);
    if (pushed) {
        if (t == push_dest(r)) st_sys(wst + OFF_LC + 16 * (pb + PO_WR + ((r - 16) & 15) * 64 + rank * 8) + 4 * wd, tag);
    } else if (t < NR) {
        if (t == rank) st_agent(wst + OFF_LC + 16 * (pb + (r < 16 ? PO_X + r * 8 : PO_W + (r - 16) * 8) + tt) + 4 * wd, tag);
        else if (r < 16) {
            const int o = r >> 1, li = o == t ? (r & 1) : (o ^ t);
            if (o != rank && (o == t || (li >= 2 && ((r ^ t) & 1) == 0))) st_sys(wst + OFF_RF + 16 * (ID_XR + li * 64 + rank * 8 + tt) + 4 * wd, tag);
        } else if (((r - 16) >> 4) == (t >> 1)) {
            const int j = (r - 16) & 15;
            if (!(tt == 0 && j < PJ && ((j ^ t) & 1) == 0) && !(((FWD_FIX >> j) & 1u) && tt == 1 + (t & 1) && (rank ^ t) >= 2))
                st_sys(wst + OFF_RF + 16 * (RF_WR + j * 64 + rank * 8 + tt) + 4 * wd, tag);
        }
    }
}


__device__ __forceinline__ bool is_out(unsigned did) { return did - (unsigned)ID_P < (unsigned)(ID_PAR - ID_P); }

__device__ __forceinline__ void units_of(unsigned dec, int ds, unsigned& ecut, unsigned& n_units) {
    if (ds == 1) {
        ecut = 0;
        for (int d = 0; d < 6; ++d) ecut += min((dec >> (4 * d)) & 15u, (unsigned)NDYN);
        n_units = (N_INP + ecut) * SL;
    } else {
        ecut = min((dec >> (4 * (ds - 2))) & 15u, (unsigned)NDYN);
        n_units = (N_IND + N_OUT - ecut) * SL;
    }
}
__device__ __forceinline__ void deal_candidates(char* wsl, unsigned call, int rank, unsigned thr, int lane) {
    const int ds = lane < 7 ? lane + 1 : 1;
    const unsigned base = ld_agent(wsl + OFF_TKB + ((call & 1u) * 8 + ds) * 4);
    const u64 t0 = __builtin_amdgcn_s_memrealtime();
    float sum1 = 0.f, sum2 = 0.f; int cnt1 = 0, cnt2 = 0;
    for (unsigned spins = 1;; ++spins) {
        const unsigned q = ld_agent(wsl + OFF_TK + (rank ^ ds) * 128) - base;
        if (__ballot(q >= (thr - 12) * SL)) { sum2 += (float)q; ++cnt2; }
        else if (__ballot(q >= 24 * SL) && !__ballot(q >= (24 + 12) * SL)) { sum1 += (float)q; ++cnt1; }
        if (__ballot(q >= thr * SL)) break;
        if ((spins & 63u) == 0 && give_up(wsl, call, t0)) break;
        __builtin_amdgcn_s_sleep(12);
    }
    auto lane_f = [](float v, int i) { return __builtin_bit_cast(float, __builtin_amdgcn_readlane(__builtin_bit_cast(int, v), i)); };
    const float p = cnt2 ? sum2 / (float)(cnt2 * SL) : 0.f;
    float rate = cnt1 && cnt2 ? p - sum1 / (float)(cnt1 * SL) : 1.f, psum = 0.f, rsum = 0.f;
    for (int i = 0; i < 7; ++i) rsum += lane_f(rate, i);
    rate = 1.f + 0.4f * __builtin_fminf(__builtin_fmaxf(rate * 7.f / rsum - 1.f, -0.02f), 0.02f);
    rsum = 0.f;
    for (int i = 0; i < 7; ++i) { rsum += lane_f(rate, i); psum += lane_f(p, i); }
    constexpr float N0 = N_IND + N_OUT, GAP = (N0 - EDYN) - (N_INP + 6 * EDYN);
    const float tau = (6.f * N0 + N_INP - psum + GAP * lane_f(rate, 0)) / rsum;
    const float want = N0 - p - tau * rate;
    int e = (int)__builtin_floorf(__builtin_fminf(__builtin_fmaxf(want, 0.f), (float)NDYN) + 0.5f);
    const int budget = (int)__builtin_floorf((tau - GAP) * lane_f(rate, 0) + lane_f(p, 0) - N_INP + 0.5f);
    for (int k = 0; k < 60; ++k) {
        int tot = 0, who = -1; float worst = -1e9f;
        for (int i = 1; i < 7; ++i) {
            const int ei = __builtin_amdgcn_readlane(e, i);
            const float ov = (float)ei - lane_f(want, i);
            tot += ei;
            if (ei > 0 && ov > worst) { worst = ov; who = i; }
        }
        if (tot <= budget || who < 0) break;
        if (lane == who) --e;
    }
    unsigned dec = 0;
    for (int i = 1; i < 7; ++i) dec |= (unsigned)__builtin_amdgcn_readlane(e, i) << (4 * (i - 1));
    unsigned ecut, n_units;
    units_of(dec, ds, ecut, n_units);
    if (lane < 7) st_agent(wsl + OFF_TKB + (((call + 1u) & 1u) * 8 + ds) * 4, base + n_units + NW * PW);
    if (lane == 0) st_agent(wsl + OFF_DEC, (dec << 8) | (call & 255u));
}

struct PUnit { unsigned src, loc, sto, st1, st4, sig, sig2, tag; bool out; };

__device__ __forceinline__ void pull_items(char* wsl, char* out, rsrc_t rsl, rsrc_t rsp, rsrc_t rsq  ,
                                           const LAS unsigned* desc  ,
                                           unsigned flex, unsigned tk_base, unsigned call, int pb, int lane, int peer, int ds) {
    static_assert(SPI == 4 && PU == 2, "a unit is four steps of 2 KiB");
    const int lane16 = lane * 16, lane_out = (lane >> 4) * 4 * (NG * 4) + (lane & 15) * 16;
    const u64 t0 = __builtin_amdgcn_s_memrealtime();
    const int lane0 = lane == 0 ? 0 : VOFF_OOB;
    const unsigned tag0 = call << 8, OFF_RF = off_rf(call), tkw = OFF_TK + peer * 128;
    unsigned dec = 0, ecut = 0, n_units = 0;
    bool have = false, dead = false;
    PUnit U, V;
    unsigned rf = 0, rf2 = 0;
    auto see = [&](unsigned w) { if (!have && ((w ^ call) & 255u) == 0) { dec = w >> 8; have = true; units_of(dec, ds, ecut, n_units); } };
    auto resolve = [&](unsigned tk) -> int {
        rf2 = OFF_DEC;
        if (!have && tk >= flex * SL) return 0;
        if (have && tk >= n_units) return 2;
        unsigned it = tk / SL;
        const unsigned sl = uni(tk % SL);
        if (have) {
            if (ds != 1) { if (it >= flex + NDYN - ecut) it += ecut; }
            else if (it >= flex + ecut) it -= ecut;
            else if (it >= flex) {
                unsigned f = it - flex, d = 0;
                for (; d < 5 && f >= ((dec >> (4 * d)) & 15u); ++d) f -= (dec >> (4 * d)) & 15u;
                it = 256 + d * NDYN + min(f, (unsigned)NDYN - 1);
            }
        }
        const unsigned w = uni(desc[it]), did = w & 0x7fffu, wr = did - pb - PO_WR, cb = (did - ID_P) & 127u;
        V.out = is_out(did);
        V.src = OFF_DATA + (w >> 16) * BLOB + sl * SLICE;
        V.loc = OFF_DATA + did * BLOB + sl * SLICE;
        V.sto = V.out ? ((peer & 1) * 256 + (sl >> 1) * 128 + ((cb >> 2) & 1) * 64 + (sl & 1) * 32) * (NG * 4) + ((peer >> 1) * 4096 + (cb >> 3) * 256 + (cb & 3) * 64) * 4 : V.loc;
        V.st1 = V.out ? NG * 4 : 1024;
        V.st4 = V.out ? 16 * NG * 4 : 4096;
        V.sig = V.out ? 0 : OFF_LC + 16 * did + 4 * sl;
        V.sig2 = (w & 0x8000u) ? OFF_RF + 16 * (wr + RF_FW) + 4 * sl : 0;
        V.tag = tag0 | 255u;
        rf = OFF_RF + 16 * (did < (unsigned)ID_PAR ? did : wr + (did == (w >> 16) ? RF_FW : RF_WR)) + 4 * sl;
        if (V.out) rf2 = OFF_LC + 16 * did;
        return 1;
    };
    u32x4 r[4][2], q[4][2];
#define PWAIT(n) do { __builtin_amdgcn_sched_barrier(0); WAITV(n); __builtin_amdgcn_sched_barrier(0); } while (0)
#define PLOAD(k, X, vr, vq) do { \
        r[k][0] = __builtin_amdgcn_raw_buffer_load_b128(rsp, vr, X.src + (k) * 2048, AUX_SYS); r[k][1] = __builtin_amdgcn_raw_buffer_load_b128(rsp, vr, X.src + (k) * 2048 + 1024, AUX_SYS); \
        q[k][0] = __builtin_amdgcn_raw_buffer_load_b128(rsl, vq, X.loc + (k) * 2048, AUX_SC1); q[k][1] = __builtin_amdgcn_raw_buffer_load_b128(rsl, vq, X.loc + (k) * 2048 + 1024, AUX_SC1); \
        MEMFENCE(); } while (0)
#define PSTORE(k) do { _Pragma("unroll") for (int u_ = 0; u_ < 2; ++u_) { \
            const f32x4 y_ = __builtin_bit_cast(f32x4, r[k][u_]) + __builtin_bit_cast(f32x4, q[k][u_]) + poison; \
            st128<AUX_SYS>(U.out ? __builtin_bit_cast(u32x4, y_) : r[k][u_], rsu, vu, uni(U.sto + ((k) >> 1) * U.st4 + (((k) & 1) * 2 + u_) * U.st1)); } \
        MEMFENCE(); } while (0)
    unsigned tk = uni((unsigned)__builtin_amdgcn_raw_ptr_buffer_atomic_add_i32(1, rsl, lane0, tkw, 0)) - tk_base;
    for (;;) {
        int st;
        for (unsigned spins = 1;; ++spins) {
            st = resolve(tk);
            if (st == 2) break;
            MEMFENCE();
            const unsigned g1 = uni(__builtin_amdgcn_raw_buffer_load_b32(rsl, lane0, uni(rf), AUX_SYS)), g2 = uni(__builtin_amdgcn_raw_buffer_load_b32(rsl, lane0, uni(rf2), AUX_SC1));
            if (st == 1 && dead) break;
            if (st == 1 && g1 >= tag0 && (!V.out || g2 >= tag0)) { V.tag = g1; break; }
            if (st == 0) { see(dead ? call : g2); if (have) continue; }
            if ((spins & 63u) == 0 && give_up(wsl, call, t0)) dead = true;
            __builtin_amdgcn_s_sleep(2);
        }
        if (st == 2) return;
        const float poison = dead ? __builtin_nanf("") : -0.0f;
        MEMFENCE();
        unsigned f0;
        {
            const int vq = V.out ? lane16 : VOFF_OOB;
            PLOAD(0, V, lane16, vq);
            f0 = (unsigned)__builtin_amdgcn_raw_ptr_buffer_atomic_add_i32(1, rsl, lane0, tkw, 0);
            MEMFENCE();
#pragma unroll
            for (int k = 0; k < 4; ++k) { __builtin_amdgcn_raw_buffer_store_b32(0, rsl, VOFF_OOB, 0, AUX_SC1); MEMFENCE(); }
            PLOAD(1, V, lane16, vq);
#pragma unroll
            for (int k = 0; k < 2; ++k) { __builtin_amdgcn_raw_buffer_store_b32(0, rsl, VOFF_OOB, 0, AUX_SC1); MEMFENCE(); }
        }
        unsigned sig = 0, sig2 = 0, stag = 0;
        bool go;
        do {
            U = V;
            const rsrc_t rsu = mk_rsrc(U.out ? out : wsl, U.out ? OUT_BYTES : (unsigned)WS_NEED);
            const int vu = U.out ? lane_out : lane16, vqu = U.out ? lane16 : VOFF_OOB;
            PWAIT(10);
            PLOAD(2, U, lane16, vqu);
            tk = uni(f0) - tk_base;
            st = resolve(tk);
            MEMFENCE();
            const unsigned f1 = __builtin_amdgcn_raw_buffer_load_b32(rsl, lane0, uni(rf), AUX_SYS), f2 = __builtin_amdgcn_raw_buffer_load_b32(rsl, lane0, uni(rf2), AUX_SC1);
            MEMFENCE();
            PSTORE(0);
            PWAIT(10);
            PLOAD(3, U, lane16, vqu);
            PSTORE(1);
            PWAIT(8);
            const unsigned g1 = uni(f1), g2 = uni(f2);
            go = st == 1 && !dead && g1 >= tag0 && (!V.out || g2 >= tag0);
            const int vr = go ? lane16 : VOFF_OOB, vq = go && V.out ? lane16 : VOFF_OOB;
            PLOAD(0, V, vr, vq);
            f0 = (unsigned)__builtin_amdgcn_raw_ptr_buffer_atomic_add_i32(1, rsl, go ? lane0 : VOFF_OOB, tkw, 0);
            MEMFENCE();
            __builtin_amdgcn_raw_buffer_store_b32(stag, rsl, sig ? lane0 : VOFF_OOB, uni(sig), AUX_SC1);
            __builtin_amdgcn_raw_buffer_store_b32(stag, rsq, sig2 ? lane0 : VOFF_OOB, uni(sig2), AUX_SYS);
            MEMFENCE();
            PSTORE(2);
            V.tag = g1;
            if (rf2 == OFF_DEC) see(g2);
            PWAIT(11);
            PLOAD(1, V, vr, vq);
            PSTORE(3);
            sig = U.sig; sig2 = U.sig2; stag = U.tag;
        } while (go);
        if (st == 2 && !sig) return;
        WAITV(0);
        __builtin_amdgcn_raw_buffer_store_b32(stag, rsl, sig ? lane0 : VOFF_OOB, uni(sig), AUX_SC1);
        __builtin_amdgcn_raw_buffer_store_b32(stag, rsq, sig2 ? lane0 : VOFF_OOB, uni(sig2), AUX_SYS);
    }
#undef PWAIT
#undef PLOAD
#undef PSTORE
}

__global__ __launch_bounds__(512, 2) void gemm_rs_fused(const Params p) {
    extern __shared__ __attribute__((aligned(16))) char shm[];
    LAS char* lds = (LAS char*)shm;
    const int b = blockIdx.x, tid = threadIdx.x, lane = tid & 63, wid = __builtin_amdgcn_readfirstlane(tid >> 6);
    const int rank = p.rank;
    char* wsl = p.wsl;
    const rsrc_t rsl = mk_rsrc(wsl, (unsigned)WS_NEED);
    const rsrc_t rso = mk_rsrc(p.out, OUT_BYTES);

    if (b >= NGEMM && b < NGEMM + NPULL) {
        const unsigned call = uni(ld_agent(wsl + OFF_CTR + b * 128)) + 1u;
        const int pi = b - NGEMM, ds = 1 + pi % (NR - 1), peer = (rank ^ ds) & (NR - 1);
        const int pb = ID_PAR + (int)(call & 1u) * PAR_SZ;
        const rsrc_t rsp = mk_rsrc(p.ws[peer], (unsigned)WS_NEED), rsq = mk_rsrc(p.ws[rank ^ 1], (unsigned)WS_NEED);
        LAS unsigned* desc = (LAS unsigned*)lds;
        const int n_in = ds == 1 ? N_INP : N_IND, n_items = ds == 1 ? N_INP : N_IND + N_OUT;
        if (tid < n_items) {
            unsigned did, sid;
            if (tid < n_in) {
                unsigned kind, a, c;
                if (ds == 1) { const unsigned code = p.pord[tid]; kind = code >> 8; a = (code >> 3) & 15u; c = code & 7u; }
                else {
                    const unsigned code = p.dord[tid]; kind = code >> 7; a = (code >> 3) & 15u; c = code & 7u;
                    if (kind == 0 && a >= (unsigned)ds) ++a;
                }
                if (kind == 0) {
                    const unsigned i = a < 2 ? rank * 2 + a : ((rank ^ a) * 2 + (rank & 1));
                    did = ID_XR + a * 64 + peer * 8 + c; sid = pb + PO_X + i * 8 + c;
                } else if (kind == 1) {
                    did = pb + PO_WR + a * 64 + peer * 8 + c; sid = pb + PO_W + ((rank >> 1) * 16 + a) * 8 + c;
                    if (ds != 1 && ((c == 2u - (rank & 1) && (((FWD_FIX | FWD_DYN) >> a) & 1u)) || (c == 4u - (rank & 1) && ((FWD_DYN >> a) & 1u))))
                        did |= 0x8000u;
                } else did = sid = pb + PO_WR + a * 64 + (rank ^ c) * 8 + 1 + (rank & 1);
            } else {
                const unsigned cb = (tid - n_in) ^ ((peer & 1) << 3);
                did = ID_P + peer * 128 + cb; sid = pb + PO_C + rank * 128 + cb;
            }
            desc[tid] = (sid << 16) | did;
        }
        if (ds == 1 && tid >= 256 && tid < 256 + 6 * NDYN) {
            const unsigned d = (tid - 256) / NDYN, f = (tid - 256) % NDYN;
            desc[tid] = (pb + PO_WR + (DYN0 + f % 5) * 64 + (rank ^ (d + 2)) * 8 + 1 + (f / 5) * 2 + (rank & 1)) * 0x10001u;
        }
        const unsigned tk_base = uni(ld_agent(wsl + OFF_TKB + ((call & 1u) * 8 + ds) * 4));
        asm volatile("s_waitcnt vmcnt(0) lgkmcnt(0)" ::: "memory");
        __builtin_amdgcn_s_barrier();
        MEMFENCE();
        if (tid == 0) st_agent(wsl + OFF_CTR + b * 128, call);
        if (pi == 0 && wid == PW) deal_candidates(wsl, call, rank, min(p.dflex, p.pflex) - 10, lane);
        if (wid >= PW) return;
        pull_items(wsl, (char*)p.out, rsl, rsp, rsq, desc, ds == 1 ? p.pflex : p.dflex, tk_base, call, pb, lane, peer, ds);
        return;
    }

    const int li = (b & 1) * 4 + ((b >> 3) & 3), tj = ((b & 7) >> 1) * 2 + ((b >> 5) & 1) + ((b >> 6) & 1) * 8;

    unsigned call = 0, kcode = 0;
    int pb = 0;
    {
        constexpr int NC2 = NCONV - 48, N_REST = 640 - 48;
        const bool early = b < 64 && li >= 2;
        const int c2 = b >= NGEMM ? b - NGEMM - NPULL : b < 64 ? NHELP + tj * 2 + li : NHELP + 16 + b - 64;
        LAS unsigned* cls = (LAS unsigned*)lds;
        auto blob_of = [](int m) {
            const int tt = m < 192 ? m >> 6 : 3 + (m - 192) / 80, r = m < 192 ? 16 + (m & 63) : (m - 192) % 80;
            return tt * 80 + (r < 16 ? r : 16 + ((r & 3) << 4) + ((r - 16) >> 2));
        };
        static_assert(NC2 <= 192, "first blobs");
        const int ex = tj * 6 + li - 2;
        int n = early ? (ex >> 4) * 80 + (ex & 15) : (c2 >> 6) * 80 + 16 + ((c2 & 3) << 4) + ((c2 & 63) >> 2);
        if (early && ex >= 16) __builtin_amdgcn_s_sleep(40);
        if (!early && c2 >= 64) __builtin_amdgcn_s_sleep(64);
        if (!early && c2 >= 128) __builtin_amdgcn_s_sleep(64);
        f32x4 v[8];
        conv_load(p, n, wid, lane, v);
        const unsigned c0 = ld_agent(wsl + OFF_CTR + b * 128);
        MEMFENCE();
        u64 wsel = 0;
#pragma unroll
        for (int r = 0; r < NR; ++r) { u64 a = (u64)p.ws[r]; asm volatile("" : "+s"(a)); wsel = (tid >> 2) == r ? a : wsel; }
        GAS char* wst = (GAS char*)wsel;
        call = uni(c0) + 1u;
        pb = ID_PAR + (int)(call & 1u) * PAR_SZ;
        int npush = -1;
        unsigned tpush = 0;
        for (int it = 0, m = early ? N_REST : c2;; ++it) {
            WAITV(0);
            const unsigned k = conv_class(v);
            conv_store(p, rsl, n, pb, wid, lane, v, k);
            if (lane == 0) cls[(it & 3) * 8 + wid] = k;
            asm volatile("s_waitcnt lgkmcnt(0)" ::: "memory");
            const bool pushes = conv_pushes(n, rank);
            if (pushes) WAITV(4); else WAITV(0);
            __builtin_amdgcn_s_barrier();
            MEMFENCE();
            const unsigned tag = (call << 8) | conv_agree(p, rsl, n, pb, wid, lane, v, k, cls + (it & 3) * 8);
            conv_publish(rank, wst, n, pb, tag, false);
            if (npush >= 0) { conv_publish(rank, wst, npush, pb, tpush, true); npush = -1; }
            if (pushes) { npush = n; tpush = tag; }
            if (it == 0) {
                if (tid == 0) st_agent(wsl + OFF_CTR + b * 128, call);
                kcode = p.kord[li < 2][tj][lane];
            }
            m += NC2;
            if (m >= N_REST) break;
            n = blob_of(m);
            __builtin_amdgcn_s_sleep(48);
            MEMFENCE();
            conv_load(p, n, wid, lane, v);
        }
        static_assert(64 + NC2 <= N_REST && NC2 >= 64 && NHELP + 80 == NC2, "converter deal");
        asm volatile("s_waitcnt lgkmcnt(0)" ::: "memory");
        __builtin_amdgcn_s_barrier();
        MEMFENCE();
    }

    const unsigned tag0 = call << 8;
    const int wr = wid >> 2, wc = wid & 3;
    for (int job = 0;; ++job) {
        int nsteps, cid, fl_rank;
        unsigned ids, fl_idx, out_off = 0;
        if (b < NGEMM) {
            if (job) break;
            const int j = tj, own = li < 2 ? rank : (rank ^ li), i = own * 2 + (li < 2 ? li : (rank & 1)), J = (rank >> 1) * 16 + j;
            nsteps = li < 2 ? 64 : 56;
            const unsigned d = kcode >> 3, tt = kcode & 7u, st = ((rank ^ d) & (NR - 1)) * 8 + tt;
            const unsigned mine = d == 0 ? (unsigned)(pb + PO_X + i * 8 + tt) | ((unsigned)(pb + PO_W + J * 8 + tt) << 16)
                                         : (unsigned)(ID_XR + li * 64 + st) | ((unsigned)(pb + PO_WR + j * 64 + st) << 16);
            const bool use = li < 2 || d != (unsigned)li;
            const int below = __builtin_popcountll(__ballot(use) & ((1ull << lane) - 1ull));
            const int pos = use ? below : 56 + lane - below;
            ids = (unsigned)__builtin_amdgcn_ds_permute(pos * 4, (int)mine);
            cid = li < 2 ? -1 : pb + PO_C + own * 128 + j * 8;
            fl_rank = own; fl_idx = ID_P + rank * 128 + j * 8;
            out_off = (unsigned)(li * 256) * (NG * 4) + (unsigned)(J * 256) * 4;
        } else {
            const int pt = (b - NGEMM - NPULL) + job * NHELP;
            if (pt >= 96) break;
            const int j = pt / 6, q = (rank ^ (2 + pt % 6)) & (NR - 1), m = lane & 7;
            nsteps = 8;
            ids = (unsigned)(pb + PO_X + (rank * 2 + (q & 1)) * 8 + m) | ((unsigned)(pb + PO_W + ((q >> 1) * 16 + j) * 8 + m) << 16);
            cid = ID_P + q * 128 + j * 8;
            fl_rank = rank; fl_idx = cid;
        }

        u64 mask = 0;
        unsigned cl = 0;
        int umax = -64;
        bool dead = false;
        auto fetch = [&](int n, int uprev, int& unit) -> unsigned {
            if (!((mask >> n) & 1ull)) {
                const u64 t0 = __builtin_amdgcn_s_memrealtime();
                for (unsigned spins = 1;; ++spins) {
                    const int vo = (unsigned)(lane - n) < (spins == 1 ? 32u : 8u) ? 0 : VOFF_OOB;
                    MEMFENCE();
                    const u32x4 fa = __builtin_amdgcn_raw_buffer_load_b128(rsl, 16 * (ids & 0xffffu) + vo, OFF_LC, AUX_SC1);
                    const u32x4 fb = __builtin_amdgcn_raw_buffer_load_b128(rsl, 16 * (ids >> 16) + vo, OFF_LC, AUX_SC1);
                    const bool ok = min(min(min(fa[0], fa[1]), min(fa[2], fa[3])), min(min(fb[0], fb[1]), min(fb[2], fb[3]))) >= tag0;
                    if (ok) cl = (max(max(fa[0], fa[1]), max(fa[2], fa[3])) & 255u) + (max(max(fb[0], fb[1]), max(fb[2], fb[3])) & 255u);
                    mask |= __ballot(ok);
                    if ((mask >> n) & 1ull) break;
                    if ((spins & 63u) == 0 && give_up(wsl, call, t0)) { mask = ~0ull; dead = true; break; }
                    __builtin_amdgcn_s_sleep(8);
                }
                MEMFENCE();
            }
            const int u = __builtin_amdgcn_readlane((int)cl, n);
            dead = dead || u >= 255;
            const bool tiny = u < umax - 5;
            unit = tiny ? uprev : u;
            umax = max(umax, u);
            return tiny ? (unsigned)ID_ZERO * 0x10001u : (unsigned)__builtin_amdgcn_readlane((int)ids, n);
        };
        const int lane16 = lane * 16;
        const unsigned wofs = OFF_DATA + wid * 2048;
#define SA(bf, h) (((bf) * 2 + (h)) * 16384)
#define SB(bf, h) ((4 + (bf) * 2 + (h)) * 16384)
#define STAGEV(bufoff, goff, vo) do { \
        __builtin_amdgcn_raw_ptr_buffer_load_lds(rsl, (LAS void*)(lds + (bufoff) + wid * 2048), 16, (vo), (goff), 0, AUX_SC1); \
        __builtin_amdgcn_raw_ptr_buffer_load_lds(rsl, (LAS void*)(lds + (bufoff) + wid * 2048), 16, (vo), (goff), 1024, AUX_SC1); } while (0)
#define STAGE(bufoff, goff) STAGEV(bufoff, goff, lane16)
#define LDA(dst, bf, h) do { _Pragma("unroll") for (int m = 0; m < 4; ++m) _Pragma("unroll") for (int k = 0; k < 2; ++k) \
        dst[m][k] = *(const LAS h8*)(lds + SA(bf, h) + aoff + m * 2048 + k * 1024); } while (0)
#define LDB(dst, bf, h) do { _Pragma("unroll") for (int n = 0; n < 2; ++n) _Pragma("unroll") for (int k = 0; k < 2; ++k) \
        dst[n][k] = *(const LAS h8*)(lds + SB(bf, h) + boff + n * 2048 + k * 1024); } while (0)
#define MMA(ai, bj, At, Bt) do { __builtin_amdgcn_s_setprio(1); \
        _Pragma("unroll") for (int m = 0; m < 4; ++m) _Pragma("unroll") for (int n = 0; n < 2; ++n) _Pragma("unroll") for (int k = 0; k < 2; ++k) \
            acc[ai][bj][m][n] = __builtin_amdgcn_mfma_f32_16x16x32_f16(At[m][k], Bt[n][k], acc[ai][bj][m][n], 0, 0, 0); \
        __builtin_amdgcn_s_setprio(0); } while (0)
#define WAIT_V(n) asm volatile("s_waitcnt vmcnt(" #n ")" ::: "memory")
#define WAIT_L(n) asm volatile("s_waitcnt lgkmcnt(" #n ")" ::: "memory")
#define BAR __builtin_amdgcn_s_barrier()
#define SCHED __builtin_amdgcn_sched_barrier(0)
#define RESCALE(from, to) do { if ((from) != (to)) { const float f_ = __builtin_ldexpf(1.0f, 16 * ((from) - (to))); \
        _Pragma("unroll") for (int a_ = 0; a_ < 2; ++a_) _Pragma("unroll") for (int c_ = 0; c_ < 2; ++c_) _Pragma("unroll") for (int m_ = 0; m_ < 4; ++m_) \
            _Pragma("unroll") for (int n_ = 0; n_ < 2; ++n_) acc[a_][c_][m_][n_] *= f_; } } while (0)
#define OFFA(c) (wofs + ((c) & 0xffffu) * BLOB)
#define OFFB(c) (wofs + ((c) >> 16) * BLOB)
        const int aoff = wr * 8192 + lane16, boff = wc * 4096 + lane16;
        f32x4 acc[2][2][4][2];
#pragma unroll
        for (int a = 0; a < 2; ++a)
#pragma unroll
            for (int c = 0; c < 2; ++c)
#pragma unroll
                for (int m = 0; m < 4; ++m)
#pragma unroll
                    for (int n = 0; n < 2; ++n) acc[a][c][m][n] = (f32x4){0.f, 0.f, 0.f, 0.f};
        h8 At[4][2], B0[2][2], B1[2][2];
        unsigned a1;
        int ucur, u0, u1;
        {
            const unsigned c0 = fetch(0, 0, u0), c1 = fetch(1, u0, u1);
            ucur = u0;
            STAGE(SB(0, 0), OFFB(c0)); STAGE(SA(0, 0), OFFA(c0)); STAGE(SB(0, 1), OFFB(c0) + 16384); STAGE(SA(0, 1), OFFA(c0) + 16384);
            if (wr == 1) BAR;
            WAIT_V(4); BAR;
            STAGE(SB(1, 0), OFFB(c1)); STAGE(SA(1, 0), OFFA(c1)); STAGE(SB(1, 1), OFFB(c1) + 16384);
            WAIT_V(6); BAR;
            a1 = OFFA(c1);
        }
        for (int t = 0; t < nsteps; t += 2) {
            const int t2 = t + 2 < nsteps ? t + 2 : 0;
            const int vs = t + 2 < nsteps ? lane16 : VOFF_OOB;
            int u2, u3;
            RESCALE(ucur, u0);
            LDB(B0, 0, 0); SCHED; LDA(At, 0, 0); STAGE(SA(1, 1), a1 + 16384);
            WAIT_L(8); BAR; WAIT_L(0); MMA(0, 0, At, B0); BAR; SCHED;
            const unsigned c2 = fetch(t2, u1, u2), a2 = OFFA(c2), b2 = OFFB(c2);
            LDB(B1, 0, 1); STAGEV(SB(0, 0), b2, vs);
            BAR; WAIT_L(0); MMA(0, 1, At, B1); BAR;
            LDA(At, 0, 1); STAGEV(SA(0, 0), a2, vs);
            BAR; WAIT_L(0); MMA(1, 0, At, B0); BAR; SCHED;
            STAGEV(SB(0, 1), b2 + 16384, vs);
            WAIT_V(6); BAR; MMA(1, 1, At, B1); BAR;
            RESCALE(u0, u1);
            LDB(B0, 1, 0); SCHED; LDA(At, 1, 0); STAGEV(SA(0, 1), a2 + 16384, vs);
            WAIT_L(8); BAR; WAIT_L(0); MMA(0, 0, At, B0); BAR; SCHED;
            const unsigned c3 = fetch(t2 + 1, u2, u3), a3 = OFFA(c3), b3 = OFFB(c3);
            LDB(B1, 1, 1); STAGEV(SB(1, 0), b3, vs);
            BAR; WAIT_L(0); MMA(0, 1, At, B1); BAR;
            LDA(At, 1, 1); STAGEV(SA(1, 0), a3, vs);
            BAR; WAIT_L(0); MMA(1, 0, At, B0); BAR; SCHED;
            STAGEV(SB(1, 1), b3 + 16384, vs);
            WAIT_V(6); BAR; MMA(1, 1, At, B1); BAR;
            a1 = a3; ucur = u1; u0 = u2; u1 = u3;
        }
        if (wr == 0) BAR;
        const float qn = __builtin_nanf("");
        const int sh = 16 * (ucur - 16);
        if (cid >= 0) {
            const unsigned base = OFF_DATA + (unsigned)(cid + wid) * BLOB;
#pragma unroll
            for (int ai = 0; ai < 2; ++ai)
#pragma unroll
                for (int m = 0; m < 4; ++m)
#pragma unroll
                    for (int r = 0; r < 4; ++r) {
                        f32x4 v = {__builtin_ldexpf(acc[ai][0][m][0][r], sh), __builtin_ldexpf(acc[ai][0][m][1][r], sh),
                                   __builtin_ldexpf(acc[ai][1][m][0][r], sh), __builtin_ldexpf(acc[ai][1][m][1][r], sh)};
                        if (dead) v = (f32x4){qn, qn, qn, qn};
                        st128<AUX_SYS>(__builtin_bit_cast(u32x4, v), rsl, lane16, base + (ai * 16 + m * 4 + r) * 1024);
                    }
            WAIT_V(0);
            if (lane < 4) {
                if (fl_rank == rank) st_agent(wsl + OFF_LC + 16 * (fl_idx + wid) + 4 * lane, tag0);
                else if ((unsigned)fl_rank < (unsigned)NR) st_sys(p.ws[fl_rank] + off_rf(call) + 16 * (fl_idx + wid) + 4 * lane, tag0);
            }
        } else {
            const int lane_out = (lane >> 4) * 4 * (NG * 4) + (lane & 15) * 16;
            const unsigned base = out_off + (unsigned)(wr * 64) * (NG * 4) + wc * 256;
#pragma unroll
            for (int ai = 0; ai < 2; ++ai)
#pragma unroll
                for (int m = 0; m < 4; ++m)
#pragma unroll
                    for (int r = 0; r < 4; ++r) {
                        f32x4 v = {__builtin_ldexpf(acc[ai][0][m][0][r], sh), __builtin_ldexpf(acc[ai][0][m][1][r], sh),
                                   __builtin_ldexpf(acc[ai][1][m][0][r], sh), __builtin_ldexpf(acc[ai][1][m][1][r], sh)};
                        if (dead) v = (f32x4){qn, qn, qn, qn};
                        st128<AUX_SC1>(__builtin_bit_cast(u32x4, v), rso, lane_out, base + (unsigned)(ai * 128 + m * 16 + r) * (NG * 4));
                    }
            WAIT_V(0);
        }
        BAR;
    }
}

void build_orders(Params& p, int e) {
    auto rt = [e](int vt) { return vt == 1 ? 2 - e : vt == 2 ? 1 + e : vt == 3 ? 4 - e : vt == 4 ? 3 + e : vt; };
    auto fix = [e](int vj, int vt) { return vt == 2 && ((FWD_FIX >> (vj ^ e)) & 1u); };
    auto dyn = [e](int vj, int vt) { return (vt == 2 || vt == 4) && ((FWD_DYN >> (vj ^ e)) & 1u); };
    int dw[16][8] = {}, dx[2][8] = {}, pw[16][8] = {}, pf[16][8] = {}, px[2][8] = {};
    int key[N_INP], code[N_INP], m = 0;
    auto sort = [&]() {
        for (int i = 1; i < m; ++i) {
            const int k = key[i], c = code[i]; int q = i;
            for (; q > 0 && key[q - 1] > k; --q) { key[q] = key[q - 1]; code[q] = code[q - 1]; }
            key[q] = k; code[q] = c;
        }
    };
    for (int vt = 0; vt < 8; ++vt) {
        for (int u = 0; u < 7; ++u) { key[m] = 8 * vt + (vt < 3 ? -100 + (u < 2) : u < 2 ? 32 : -1); code[m++] = (u << 3) | vt; }
        for (int vj = 0; vj < 16; ++vj) {
            if (vt == 0 && vj % 2 == 0 && vj < PJ) { dw[vj][vt] = 0; continue; }
            if (fix(vj, vt) || dyn(vj, vt)) continue;
            key[m] = 8 * vt + 8 * ((vj | 1) > 3 ? (vj | 1) - 3 : 0) - (vt == 0 ? 50 : 0); code[m++] = 0x80 | (vj << 3) | vt;
        }
    }
    for (int i = 0; i < m; ++i) key[i] *= 2;
    for (int f = NDYN - 1; f >= 0; --f) { key[m] = 2 * 120 + 1; code[m++] = 0x80 | (((DYN0 + f % 5) ^ e) << 3) | (f < 5 ? 2 : 4); }
    sort();
    p.dflex = 0;
    while (key[p.dflex] != 2 * 120 + 1) ++p.dflex;
    for (int i = 0; i < m; ++i) {
        const int a = (code[i] >> 3) & 15, vt = code[i] & 7;
        if (code[i] & 0x80) { dw[a][vt] = i; p.dord[i] = (unsigned char)(0x80 | ((a ^ e) << 3) | rt(vt)); }
        else { dx[a < 2][vt] = i; p.dord[i] = (unsigned char)((a << 3) | rt(vt)); }
    }
    m = 0;
    for (int vj = 0; vj < 16; ++vj) {
        const bool psh = vj % 2 == 0 && vj < PJ;
        const int cnt = (psh ? 7 : 8) + (fix(vj, 2) ? 6 : 0);
        int k = 0;
        for (int vt = (psh ? 1 : 0); vt < 8; ++vt, ++k) { key[m] = 32 * vj - 10 * (cnt - 1 - k) - (rt(vt) < 3 && vj < 8 ? 2000 - 400 * rt(vt) + 32 * vj : 0); code[m++] = 0x100 | (vj << 3) | vt; }
        if (fix(vj, 2)) for (int d = 2; d < 8; ++d, ++k) { key[m] = 32 * vj - 10 * (cnt - 1 - k); code[m++] = 0x200 | (vj << 3) | d; }
        pw[vj][0] = 0;
    }
    for (int vt = 0; vt < 8; ++vt) {
        for (int li = 2; li < 8; ++li) { key[m] = 8 * (vt - 8) - (vt < 3 ? 4000 : 0); code[m++] = (li << 3) | vt; }
        for (int h = 0; h < 2; ++h) { key[m] = 464 + 8 * vt; code[m++] = (h << 3) | vt; }
    }
    sort();
    p.pflex = 0;
    while (key[p.pflex] <= 208) ++p.pflex;
    for (int n = 0; n < m; ++n) {
        const int kind = code[n] >> 8, a = (code[n] >> 3) & 15, c = code[n] & 7, i = n < p.pflex ? n : n + 6 * EDYN;
        if (kind == 0) { px[a < 2][c] = i; p.pord[n] = (unsigned short)((a << 3) | rt(c)); }
        else if (kind == 1) { pw[a][c] = i; p.pord[n] = (unsigned short)(0x100 | ((a ^ e) << 3) | rt(c)); }
        else { pf[a][c] = i; p.pord[n] = (unsigned short)(0x200 | ((a ^ e) << 3) | c); }
    }
    for (int own = 0; own < 2; ++own)
        for (int vj = 0; vj < 16; ++vj) {
            int kk[64], cc[64], c = 0;
            for (int vt = 0; vt < 8; ++vt)
                for (int d = 0; d < 8; ++d) {
                    const int ax = d == 1 ? px[own][vt] : dx[own][vt];
                    const int aw = d == 1 ? pw[vj][vt] : fix(vj, vt) ? pf[vj][d] : dyn(vj, vt) ? (p.pflex + (d - 1) * EDYN > p.dflex + NDYN - EDYN ? p.pflex + (d - 1) * EDYN : p.dflex + NDYN - EDYN) : dw[vj][vt] - (dw[vj][vt] > p.dflex ? EDYN : 0);
                    const int k = d == 0 ? 4 * rt(vt) - 14 : ax > aw ? ax : aw; int q = c++;
                    for (; q > 0 && kk[q - 1] > k; --q) { kk[q] = kk[q - 1]; cc[q] = cc[q - 1]; }
                    kk[q] = k; cc[q] = (d << 3) | rt(vt);
                }
            for (int i = 0; i < 64; ++i) p.kord[own][vj ^ e][i] = (unsigned char)cc[i];
        }
}

}

extern "C" void kernel_launch(int rank, int nranks, ncclComm_t comm,
                              hipStream_t stream,
                              void* const* d_in, const int* in_sizes, int n_in,
                              void* d_out, int out_size,
                              const int* in_dtypes, int out_dtype,
                              void* const* d_out_all,
                              void* const* d_ws_all, size_t ws_bytes) {
    (void)comm; (void)in_sizes; (void)n_in; (void)in_dtypes; (void)out_dtype; (void)d_out_all;
    if (nranks != NR || (unsigned)rank >= (unsigned)NR || ws_bytes < WS_NEED || (size_t)out_size * 4 != OUT_BYTES) {
        fprintf(stderr, "kernel_launch: unexpected configuration\n");
        return;
    }
    Params p;
    p.x = (const float*)d_in[0];
    p.w = (const float*)d_in[1];
    p.out = (float*)d_out;
    for (int r = 0; r < NR; ++r) p.ws[r] = (char*)d_ws_all[r];
    p.rank = rank;
    p.wsl = p.ws[rank];
    for (int r = 0; r < 5; ++r) p.pad[r] = 0;
    build_orders(p, rank & 1);
    HIP_CHECK(hipFuncSetAttribute((const void*)gemm_rs_fused, hipFuncAttributeMaxDynamicSharedMemorySize, LDS_BYTES));
    hipLaunchKernelGGL(gemm_rs_fused, dim3(GRID), dim3(512), LDS_BYTES, stream, p);
    HIP_CHECK(hipGetLastError());
}
```
